```python
import numpy as np

kernel_name = "gpu_ag_gemm_m8192_k8192_n8192_bf16_gelu"


ABSMAX_RTOL = 0.008

COMPUTE_DTYPE = "bfloat16"
EPILOGUE = "gelu"
IN_QUANT = None
OUT_QUANT = None
IN_DTYPES = {"x": "bfloat16", "w_mat": "bfloat16"}
WORLD_SIZE = 8

def _to_bf16(x):
    u = np.ascontiguousarray(x, dtype=np.float32).view(np.uint32)
    return ((u + 0x7FFF + ((u >> 16) & 1)) & 0xFFFF0000).view(np.float32)

def setup_inputs():

    x = _to_bf16(np.random.randn(8192, 8192).astype(np.float32))
    w_mat = _to_bf16((np.random.randn(8192, 8192) / np.sqrt(8192)).astype(np.float32))
    return {"x": x, "w_mat": w_mat}

def _epilogue(y):
    c = 0.7978845608028654
    return (0.5 * y * (1.0 + np.tanh(c * (y + 0.044715 * y ** 3)))).astype(np.float32)

def reference(x, w_mat):
    y = (x @ w_mat).astype(np.float32)
    return _epilogue(y)
```

```cpp
#include <hip/hip_runtime.h>
#include <rccl/rccl.h>
#include <cstdio>
#include <cstdint>

#define HIP_CHECK(expr) do { hipError_t _e = (expr); if (_e != hipSuccess) { \
    fprintf(stderr, "HIP error %s at %s:%d\n", hipGetErrorString(_e), __FILE__, __LINE__); \
    return; } } while (0)

namespace agk {

#define LAS __attribute__((address_space(3)))
typedef short bf16x8 __attribute__((ext_vector_type(8)));
typedef short s16x4 __attribute__((ext_vector_type(4)));
typedef float f32x4 __attribute__((ext_vector_type(4)));
typedef unsigned u32x4 __attribute__((ext_vector_type(4)));
typedef unsigned u32x2 __attribute__((ext_vector_type(2)));
typedef __amdgpu_buffer_rsrc_t rsrc_t;

constexpr int NR = 8, KDIM = 8192, MSH = 1024, NSH = 1024;
constexpr unsigned XPITCH = KDIM * 2;
constexpr unsigned WPITCH = NSH * 2;
constexpr unsigned QPITCH = 512;
constexpr unsigned UNITB = 64u * XPITCH;

#define CFG_U16 11
#define CFG_U2 2
#define CFG_NU 4
#define CFG_NJ 16
#define CFG_EPI_INFLIGHT 0
#define CFG_PU 2
constexpr int U16 = CFG_U16, V16 = 16 - U16;
constexpr int U2 = CFG_U2, U1 = U16 - U2;
constexpr int NU = CFG_NU;
constexpr int NJ = CFG_NJ;
constexpr int NPUSH = 7 * NJ;
constexpr int UA = 16 + 3 * U1 + 4 * V16;
constexpr int UB = U16 + 3 * U1 + 3 * V16;
constexpr int RBA = (UA + NU - 1) / NU, RBB = (UB + NU - 1) / NU;
constexpr int NWG = 256;
constexpr int NGWG = 8 * ((4 * RBA + 7) / 8 + (4 * RBB + 7) / 8);
static_assert(NPUSH + NGWG <= NWG, "grid");
constexpr int GEMM0 = NWG - NGWG;
constexpr int NT2X = 3 * U2 * 2;

#define CFG_PA 1
#define CFG_PA_CNT 1
#define CFG_PA_TAIL 0
#define CFG_POLL_SLEEP 2
#define CFG_LG1 2
#define CFG_LG2 2
constexpr int NT = KDIM / 64;
constexpr int LG1 = CFG_LG1, LG2 = CFG_LG2;
__host__ __device__ constexpr int ch_body(int lg) { return (NT >> lg) - 1; }
__host__ __device__ constexpr int ch_count(int lg) { return ch_body(lg) + lg + 1; }
__host__ __device__ constexpr int ch_lg(int lg, int c) { return c < ch_body(lg) ? lg : (c - ch_body(lg) < lg ? lg - 1 - (c - ch_body(lg)) : 0); }
constexpr int NCH1 = ch_count(LG1), NCH2 = ch_count(LG2), NCH = NCH1 + NCH2;
__host__ __device__ constexpr int CH_LOGF(int c) { return c < NCH1 ? ch_lg(LG1, c) : ch_lg(LG2, c - NCH1); }
constexpr int ch_sum(int a, int b) { int s = 0; for (int i = a; i < b; ++i) s += 1 << CH_LOGF(i); return s; }
static_assert(ch_sum(0, NCH1) == NT && ch_sum(NCH1, NCH) == NT, "chunks");
constexpr unsigned long long ch_mask(int gen, int half) { unsigned long long m = 0; int s = 0;
    for (int i = gen ? NCH1 : 0; i < (gen ? NCH : NCH1); ++i) { if (s / 64 == half) m |= 1ull << (s % 64); s += 1 << CH_LOGF(i); } return m; }
constexpr unsigned long long CHM0 = ch_mask(0, 0), CHM1 = ch_mask(0, 1), CH2M0 = ch_mask(1, 0), CH2M1 = ch_mask(1, 1);
constexpr int FROW = 256;
static_assert(NCH <= FROW, "flags");

#define CFG_WAVETK 1
constexpr int PB = (CFG_WAVETK ? 1 : 8) * CFG_PU;
constexpr int PBASE_S1 = U1 * 8, PBASE_S2 = U2 * 8, PBASE_X = V16 * 8 + 32;
__host__ __device__ constexpr int nbatch(int pieces) { return (pieces + PB - 1) / PB; }
constexpr int nb_total(bool same) { int s = 0; for (int c = 0; c < NCH; ++c) { const int pb = c < NCH1 ? (same ? PBASE_S1 : PBASE_X) : (same ? PBASE_S2 : 0); s += nbatch(pb << CH_LOGF(c)); } return s; }
constexpr int NB_S = nb_total(true), NB_X = nb_total(false);

constexpr size_t WS_CTR = 0;
constexpr size_t WS_TMO = 16384;
constexpr size_t WS_DONE = 16384 + 256;
constexpr size_t WS_QUEUE = 16384 + 512;
constexpr size_t WS_TICK = 16384 + 2048;
constexpr size_t WS_FLAG = 32768;
constexpr size_t WS_DBG = 65536;
constexpr size_t WS_STAGE = 1u << 20;
constexpr size_t XS_BYTES = (size_t)(3 * U16 + 4 * V16) * UNITB;
constexpr size_t WSQ_BYTES = (size_t)KDIM * QPITCH;
constexpr size_t PAR_BYTES = XS_BYTES + 4 * WSQ_BYTES;
constexpr size_t WS_NEED = WS_STAGE + 2 * PAR_BYTES;
constexpr unsigned long long DEADLINE = 100000000ull;

constexpr int LDS_STAGE = 2 * NU * 8192 + 65536;
constexpr int LDS_EPI = NU * 64 * 528;
constexpr int LDS_SMALL = 6 * 24576;
constexpr int cmax(int a, int b) { return a > b ? a : b; }
constexpr int LDS_CTL = cmax(cmax(LDS_STAGE, LDS_EPI), LDS_SMALL);
constexpr int LDS_BYTES = LDS_CTL + 256;

struct Params {
    const char* x; const char* w;
    char* out[NR]; char* ws[NR];
    int rank; int dbg;
};

__device__ __forceinline__ unsigned cvt_pk_bf16(float lo, float hi) { unsigned r; asm("v_cvt_pk_bf16_f32 %0, %1, %2" : "=v"(r) : "v"(lo), "v"(hi)); return r; }
__device__ __forceinline__ float gelu1(float y) {
    const float c2 = -2.0f * 0.7978845608028654f * 1.4426950408889634f;
    const float a = y * __builtin_fmaf(y * y, 0.044715f * c2, c2);
    return y * __builtin_amdgcn_rcpf(1.0f + __builtin_amdgcn_exp2f(a));
}
__device__ __forceinline__ rsrc_t mk_rsrc(const void* p, unsigned bytes) { return __builtin_amdgcn_make_buffer_rsrc((void*)p, 0, bytes, 0x00020000); }

#define WAIT_V_(n) asm volatile("s_waitcnt vmcnt(" #n ")" ::: "memory")
#define WAIT_V(n) WAIT_V_(n)
#define WAIT_L(n) asm volatile("s_waitcnt lgkmcnt(" #n ")" ::: "memory")
#define BAR() __builtin_amdgcn_s_barrier()
#define SCHED() __builtin_amdgcn_sched_barrier(0)
#define SYS_LOAD(p) __hip_atomic_load((p), __ATOMIC_RELAXED, __HIP_MEMORY_SCOPE_SYSTEM)
#define AGT_LOAD(p) __hip_atomic_load((p), __ATOMIC_RELAXED, __HIP_MEMORY_SCOPE_AGENT)
#define DMA(rs, ldsp, voff, soff) __builtin_amdgcn_raw_ptr_buffer_load_lds((rs), (LAS void*)(ldsp), 16, (voff), (soff), 0, 16)
#define TRR(dst, base, off) asm volatile("ds_read_b64_tr_b16 %0, %1 offset:%2" : "=v"(dst) : "v"(base), "i"(off) : "memory")

#define STAMP(slot) do {} while (0)

__device__ __forceinline__ bool wave_wait(const unsigned* p, unsigned need, unsigned* tmo, unsigned call) {
    const unsigned long long t0 = __builtin_amdgcn_s_memrealtime();
    for (;;) {
        const unsigned v = SYS_LOAD(p);
        if (__all((int)(v - need) >= 0 || need == 0u)) return true;
        if (__builtin_amdgcn_readfirstlane(AGT_LOAD(tmo)) == call || __builtin_amdgcn_s_memrealtime() - t0 > DEADLINE) {
            __hip_atomic_store(tmo, call, __ATOMIC_RELAXED, __HIP_MEMORY_SCOPE_AGENT);
            return false;
        }
        __builtin_amdgcn_s_sleep(4);
    }
}

template <int WN, int NS>
__device__ __forceinline__ int window_wait(const unsigned* p, unsigned need, unsigned* tmo, unsigned call) {
    const unsigned long long t0 = __builtin_amdgcn_s_memrealtime();
    for (;;) {
        const unsigned v = SYS_LOAD(p);
        unsigned long long m = __ballot((int)(v - need) >= 0 || need == 0u), a = m;
#pragma unroll
        for (int s = 1; s < NS; ++s) a &= m >> (s * WN);
        a &= (1ull << WN) - 1ull;
        if (a & 1ull) return (int)__builtin_ctzll(~a);
        if (__builtin_amdgcn_readfirstlane(AGT_LOAD(tmo)) == call || __builtin_amdgcn_s_memrealtime() - t0 > DEADLINE) {
            __hip_atomic_store(tmo, call, __ATOMIC_RELAXED, __HIP_MEMORY_SCOPE_AGENT);
            return WN;
        }
        __builtin_amdgcn_s_sleep(CFG_POLL_SLEEP);
    }
}

__device__ __forceinline__ void store_out(char* o, u32x4 v) { asm volatile("global_store_dwordx4 %0, %1, off sc0 sc1\n\ts_nop 1" :: "v"(o), "v"(v) : "memory"); }

struct TileDesc {
    const char* xa[NU];
    const char* wb; unsigned wpitch, wbytes;
    char* out[NU];
    const unsigned* flag;
    unsigned call1;
    int pbase;
    bool anywait, throttle;
};

__device__ __forceinline__ void gemm_tile(LAS unsigned char* lds, const TileDesc& d, const Params& P, unsigned call) {
    constexpr int MT = NU * 2;
    constexpr int ABUF = NU * 8192;
    constexpr int BOFF = 2 * ABUF;
    const int tid = threadIdx.x, wid = __builtin_amdgcn_readfirstlane(tid >> 6), lane = tid & 63;
    const int wr = wid >> 2, wc = wid & 3, fr = lane & 15, fq = lane >> 4;
    unsigned* tmo = (unsigned*)(P.ws[P.rank] + WS_TMO);
    volatile LAS unsigned* ready = (volatile LAS unsigned*)(lds + LDS_CTL);
    const unsigned voffA = (unsigned)(8 * wid + (lane >> 3)) * XPITCH + (unsigned)(((lane & 7) ^ (((wid & 1) << 2) | (lane >> 4))) * 16);
    const unsigned fB = (unsigned)((lane >> 4) | (((wid >> 1) & 1) << 2));
    const unsigned voffB = (unsigned)(4 * wid + (lane >> 4)) * d.wpitch + ((((unsigned)(lane & 15) >> 1) ^ fB) * 32u + (unsigned)(lane & 1) * 16u);
    const unsigned ldsw = (unsigned)wid * 1024u;
    const unsigned aoff0 = (unsigned)(wr * (NU * 32) + fr) * 128u + (unsigned)((fq ^ (fr >> 1)) * 16);
    const unsigned aoff1 = aoff0 ^ 64u;
    const unsigned q = (unsigned)(fr >> 2), p = (unsigned)(fr & 3);
    const unsigned boff0 = (unsigned)(size_t)lds + BOFF + (unsigned)(8 * fq + q) * 256u + (((unsigned)(2 * wc) ^ (q | ((unsigned)(fq & 1) << 2))) * 32u) + p * 8u;
    const unsigned boff1 = boff0 ^ 32u;
    rsrc_t ra[NU];
#pragma unroll
    for (int u = 0; u < NU; ++u) ra[u] = mk_rsrc(d.xa[u], UNITB);
    const rsrc_t rb = mk_rsrc(d.wb, d.wbytes);
    const unsigned wstep = 64u * d.wpitch, whalf = 32u * d.wpitch;

    f32x4 acc[2][MT][2];
#pragma unroll
    for (int b = 0; b < 2; ++b)
#pragma unroll
        for (int m = 0; m < MT; ++m)
#pragma unroll
            for (int n = 0; n < 2; ++n) acc[b][m][n] = (f32x4){0.f, 0.f, 0.f, 0.f};
    bf16x8 af[MT][2], bfr[2][2]; s16x4 tb[8];

#define STAGE_A(buf, t) do { _Pragma("unroll") for (int _i = 0; _i < NU; ++_i) DMA(ra[_i], lds + (buf) * ABUF + _i * 8192 + ldsw, voffA, (unsigned)(t) * 128u); } while (0)
#define STAGE_B(buf, h, t) do { _Pragma("unroll") for (int _i = 0; _i < 2; ++_i) \
        DMA(rb, lds + BOFF + ((buf) * 2 + (h)) * 16384 + _i * 8192 + ldsw, voffB, (unsigned)(t) * wstep + _i * whalf + (h) * 256u); } while (0)
#define LDA(buf) do { _Pragma("unroll") for (int m = 0; m < MT; ++m) { \
        af[m][0] = *(const LAS bf16x8*)(lds + (buf) * ABUF + aoff0 + m * 2048); \
        af[m][1] = *(const LAS bf16x8*)(lds + (buf) * ABUF + aoff1 + m * 2048); } } while (0)
#define LDB(buf, h) do { \
        TRR(tb[0], boff0, ((buf) * 2 + (h)) * 16384); TRR(tb[1], boff0, ((buf) * 2 + (h)) * 16384 + 1024); \
        TRR(tb[2], boff1, ((buf) * 2 + (h)) * 16384); TRR(tb[3], boff1, ((buf) * 2 + (h)) * 16384 + 1024); \
        TRR(tb[4], boff0, ((buf) * 2 + (h)) * 16384 + 8192); TRR(tb[5], boff0, ((buf) * 2 + (h)) * 16384 + 8192 + 1024); \
        TRR(tb[6], boff1, ((buf) * 2 + (h)) * 16384 + 8192); TRR(tb[7], boff1, ((buf) * 2 + (h)) * 16384 + 8192 + 1024); } while (0)
#define WAITB() do { asm volatile("s_waitcnt lgkmcnt(0)" : "+v"(tb[0]), "+v"(tb[1]), "+v"(tb[2]), "+v"(tb[3]), "+v"(tb[4]), "+v"(tb[5]), "+v"(tb[6]), "+v"(tb[7]) :: "memory"); \
        bfr[0][0] = __builtin_shufflevector(tb[0], tb[1], 0, 1, 2, 3, 4, 5, 6, 7); bfr[1][0] = __builtin_shufflevector(tb[2], tb[3], 0, 1, 2, 3, 4, 5, 6, 7); \
        bfr[0][1] = __builtin_shufflevector(tb[4], tb[5], 0, 1, 2, 3, 4, 5, 6, 7); bfr[1][1] = __builtin_shufflevector(tb[6], tb[7], 0, 1, 2, 3, 4, 5, 6, 7); } while (0)
#define MMA(bj) do { __builtin_amdgcn_s_setprio(1); \
        _Pragma("unroll") for (int kk = 0; kk < 2; ++kk) _Pragma("unroll") for (int m = 0; m < MT; ++m) _Pragma("unroll") for (int n = 0; n < 2; ++n) \
            acc[bj][m][n] = __builtin_amdgcn_mfma_f32_16x16x32_bf16(bfr[n][kk], af[m][kk], acc[bj][m][n], 0, 0, 0); \
        __builtin_amdgcn_s_setprio(0); } while (0)
#define WAITN() do { if constexpr (NU == 3) { WAIT_V(7); } else if constexpr (NU == 2) { WAIT_V(6); } else { WAIT_V(8); } } while (0)
    constexpr int NS = NU + 1, WN = 64 / NS;
    int cnext = 0, known = 0;
#define POLL() do { const int ck_ = cnext + lane % WN; const bool w_ = ck_ < NCH1 && lane < NS * WN; const int cc_ = w_ ? ck_ : 0; \
        known = cnext + window_wait<WN, NS>(d.flag + cc_, w_ ? d.call1 * (unsigned)nbatch(d.pbase << CH_LOGF(cc_)) : 0u, tmo, call); } while (0)
#define GATE() do { \
        if (known <= cnext) { \
            if (wid == 0) { POLL(); if (lane == 0) *ready = (unsigned)known; } \
            else { unsigned r_; while ((r_ = *ready) <= (unsigned)cnext) __builtin_amdgcn_s_sleep(1); known = (int)__builtin_amdgcn_readfirstlane(r_); } \
        } \
        if ((cnext & 3) == 0) STAMP(8 + (cnext >> 2)); \
        ++cnext; } while (0)
#define CHECK(tt) do { if (d.anywait && (tt) < NT && ((((tt) < 64 ? CHM0 : CHM1) >> ((tt) & 63)) & 1ull)) GATE(); } while (0)

    if (d.anywait) {
        if (wid == 0) { POLL(); if (lane == 0) *ready = (unsigned)known; }
        WAIT_L(0); BAR();
        known = (int)__builtin_amdgcn_readfirstlane(*ready);
        cnext = 1;
    }
    static_assert(LG1 >= 1, "prologue loads K tiles 0, 1");
    STAMP(1);
    STAGE_A(0, 0); STAGE_B(0, 0, 0); STAGE_B(0, 1, 0);
    STAGE_A(1, 1); STAGE_B(1, 0, 1);
    if (wr == 1) BAR();
    if constexpr (NU == 3) { WAIT_V(5); } else if constexpr (NU == 2) { WAIT_V(4); } else { WAIT_V(6); }
    BAR(); BAR();
#define KTILE(b, t) do { \
        LDB(b, 0); LDA(b); \
        STAGE_B((b) ^ 1, 1, min((t) + 1, NT - 1)); \
        WAITN(); WAITB(); BAR(); SCHED(); MMA(0); SCHED(); BAR(); \
        CHECK((t) + 2); \
        LDB(b, 1); \
        STAGE_A(b, min((t) + 2, NT - 1)); STAGE_B(b, 0, min((t) + 2, NT - 1)); \
        WAITN(); WAITB(); BAR(); SCHED(); MMA(1); SCHED(); BAR(); } while (0)
    for (int t = 0; t < NT; t += 2) {
        KTILE(0, t);
        KTILE(1, t + 1);
    }
    WAIT_V(0);
    if (wr == 0) BAR();
    BAR();
    STAMP(2);
    constexpr int EP = 528;
    const bool bad = AGT_LOAD(tmo) == call;
#pragma unroll
    for (int bj = 0; bj < 2; ++bj)
#pragma unroll
        for (int m = 0; m < MT; ++m)
#pragma unroll
            for (int n = 0; n < 2; ++n) {
                const f32x4 v = acc[bj][m][n];
                u32x2 w; w.x = cvt_pk_bf16(gelu1(v[0]), gelu1(v[1])); w.y = cvt_pk_bf16(gelu1(v[2]), gelu1(v[3]));
                if (bad) { w.x = 0x7fc07fc0u; w.y = 0x7fc07fc0u; }
                *(LAS u32x2*)(lds + (wr * (NU * 32) + m * 16 + fr) * EP + (bj * 128 + wc * 32 + n * 16 + fq * 4) * 2) = w;
            }
    WAIT_L(0); BAR();
#pragma unroll
    for (int u = 0; u < NU; ++u) {
        if (d.out[u] != nullptr) {
#pragma unroll
            for (int i = 0; i < 4; ++i) {
                const int r = (i * 8 + wid) * 2 + (lane >> 5);
                const u32x4 v = *(const LAS u32x4*)(lds + (u * 64 + r) * EP + (lane & 31) * 16);
                store_out(d.out[u] + (size_t)r * WPITCH + (lane & 31) * 16, v);
                if (d.throttle) WAIT_V(CFG_EPI_INFLIGHT);
            }
        }
    }
#undef STAGE_A
#undef STAGE_B
#undef LDA
#undef LDB
#undef MMA
#undef KTILE
#undef WAITB
#undef WAITN
#undef GATE
#undef CHECK
#undef POLL
}

__device__ __forceinline__ void small_tile(LAS unsigned char* lds, const char* xa, const char* wb, unsigned wpitch, unsigned wbytes, char* out,
                                           const unsigned* xflag, unsigned xcall, const unsigned* wflag, unsigned wcall, const Params& P, unsigned call) {
    const int tid = threadIdx.x, wid = __builtin_amdgcn_readfirstlane(tid >> 6), lane = tid & 63;
    const int g = wid >> 2, wq = wid & 3, wm = wq >> 1, wn = wq & 1, fr = lane & 15, fq = lane >> 4;
    unsigned* tmo = (unsigned*)(P.ws[P.rank] + WS_TMO);
    volatile LAS unsigned* ready = (volatile LAS unsigned*)(lds + LDS_CTL + 32) + g;
    const unsigned voffA = (unsigned)(8 * wq + (lane >> 3)) * XPITCH + (unsigned)(((lane & 7) ^ (((wq & 1) << 2) | (lane >> 4))) * 16);
    const unsigned fB = (unsigned)((lane >> 4) | (((wq >> 1) & 1) << 2));
    const unsigned voffB = (unsigned)(4 * wq + (lane >> 4)) * wpitch + ((((unsigned)(lane & 15) >> 1) ^ fB) * 32u + (unsigned)(lane & 1) * 16u);
    const unsigned gbase = (unsigned)g * (3u * 24576u);
    const unsigned ldsw = gbase + (unsigned)wq * 1024u;
    const unsigned aoff0 = gbase + (unsigned)(wm * 32 + fr) * 128u + (unsigned)((fq ^ (fr >> 1)) * 16);
    const unsigned aoff1 = aoff0 ^ 64u;
    const unsigned q = (unsigned)(fr >> 2), p = (unsigned)(fr & 3);
    const unsigned boff = (unsigned)(size_t)lds + gbase + 8192u + (unsigned)(8 * fq + q) * 256u + (((unsigned)(4 * wn) ^ (q | ((unsigned)(fq & 1) << 2))) * 32u) + p * 8u;
    const unsigned bo0 = boff, bo1 = boff ^ 32u, bo2 = boff ^ 64u, bo3 = boff ^ 96u;
    const rsrc_t ra = mk_rsrc(xa, UNITB), rb = mk_rsrc(wb, wbytes);
    const unsigned wstep = 64u * wpitch, wq16 = 16u * wpitch;
    f32x4 acc[2][4];
#pragma unroll
    for (int m = 0; m < 2; ++m)
#pragma unroll
        for (int n = 0; n < 4; ++n) acc[m][n] = (f32x4){0.f, 0.f, 0.f, 0.f};
#define STAGE(s, t) do { DMA(ra, lds + (s) * 24576 + ldsw, voffA, (unsigned)(t) * 128u); DMA(ra, lds + (s) * 24576 + 4096 + ldsw, voffA, (unsigned)(t) * 128u + 32u * XPITCH); \
        _Pragma("unroll") for (int _j = 0; _j < 4; ++_j) DMA(rb, lds + (s) * 24576 + 8192 + _j * 4096 + ldsw, voffB, (unsigned)(t) * wstep + _j * wq16); } while (0)
    int known = 0;
#define CIDX(tt) ((tt) < 64 ? __builtin_popcountll(CH2M0 & ((2ull << (tt)) - 1ull)) - 1 : __builtin_popcountll(CH2M0) + __builtin_popcountll(CH2M1 & ((2ull << ((tt) - 64)) - 1ull)) - 1)
#define POLL(ci) do { const int ck_ = (ci) + lane; const bool w_ = ck_ < NCH2 && lane < 32; const int cc_ = w_ ? ck_ : 0; \
        known = (ci) + window_wait<32, 1>(xflag + cc_, w_ ? xcall * (unsigned)nbatch(PBASE_S2 << CH_LOGF(NCH1 + cc_)) : 0u, tmo, call); } while (0)
#define CHECK(tt) do { if ((tt) < NT) { const int ci_ = CIDX(tt); if (known <= ci_) { \
            if (wq == 0) { POLL(ci_); if (lane == 0) *ready = (unsigned)known; } \
            else { unsigned r_; while ((r_ = *ready) <= (unsigned)ci_) __builtin_amdgcn_s_sleep(1); known = (int)__builtin_amdgcn_readfirstlane(r_); } } } } while (0)
    BAR();
    if (wid == 0) {
        for (int c0 = 0; c0 < NCH1; c0 += 64) { const int lc = c0 + lane < NCH1 ? c0 + lane : 0;
            wave_wait(wflag + lc, c0 + lane < NCH1 ? wcall * (unsigned)nbatch(PBASE_X << CH_LOGF(lc)) : 0u, tmo, call); }
        POLL(0); if (lane == 0) { ready[0] = (unsigned)known; ready[1] = (unsigned)known; }
    }
    WAIT_L(0); BAR();
    known = (int)__builtin_amdgcn_readfirstlane(*ready);
    STAMP(5);
    static_assert(LG2 >= 2, "small tile prologue loads K tiles 0..3");
    STAGE(0, g); STAGE(1, g + 2);
    WAIT_V(6); BAR();
    if (g == 1) BAR();
    s16x4 tb[16]; bf16x8 af[2][2];
#define STEP(s, t) do { \
        CHECK((t) + 4); \
        TRR(tb[0], bo0, (s) * 24576); TRR(tb[1], bo0, (s) * 24576 + 1024); TRR(tb[2], bo1, (s) * 24576); TRR(tb[3], bo1, (s) * 24576 + 1024); \
        TRR(tb[4], bo2, (s) * 24576); TRR(tb[5], bo2, (s) * 24576 + 1024); TRR(tb[6], bo3, (s) * 24576); TRR(tb[7], bo3, (s) * 24576 + 1024); \
        TRR(tb[8], bo0, (s) * 24576 + 8192); TRR(tb[9], bo0, (s) * 24576 + 9216); TRR(tb[10], bo1, (s) * 24576 + 8192); TRR(tb[11], bo1, (s) * 24576 + 9216); \
        TRR(tb[12], bo2, (s) * 24576 + 8192); TRR(tb[13], bo2, (s) * 24576 + 9216); TRR(tb[14], bo3, (s) * 24576 + 8192); TRR(tb[15], bo3, (s) * 24576 + 9216); \
        _Pragma("unroll") for (int m = 0; m < 2; ++m) { af[m][0] = *(const LAS bf16x8*)(lds + (s) * 24576 + aoff0 + m * 2048); af[m][1] = *(const LAS bf16x8*)(lds + (s) * 24576 + aoff1 + m * 2048); } \
        STAGE(((s) + 2) % 3, min((t) + 4, NT - 2 + g)); \
        asm volatile("s_waitcnt lgkmcnt(0)" : "+v"(tb[0]), "+v"(tb[1]), "+v"(tb[2]), "+v"(tb[3]), "+v"(tb[4]), "+v"(tb[5]), "+v"(tb[6]), "+v"(tb[7]), \
                     "+v"(tb[8]), "+v"(tb[9]), "+v"(tb[10]), "+v"(tb[11]), "+v"(tb[12]), "+v"(tb[13]), "+v"(tb[14]), "+v"(tb[15]) :: "memory"); \
        BAR(); SCHED(); \
        __builtin_amdgcn_s_setprio(1); \
        _Pragma("unroll") for (int kk = 0; kk < 2; ++kk) _Pragma("unroll") for (int n = 0; n < 4; ++n) { \
            const bf16x8 b_ = __builtin_shufflevector(tb[kk * 8 + n * 2], tb[kk * 8 + n * 2 + 1], 0, 1, 2, 3, 4, 5, 6, 7); \
            _Pragma("unroll") for (int m = 0; m < 2; ++m) acc[m][n] = __builtin_amdgcn_mfma_f32_16x16x32_bf16(b_, af[m][kk], acc[m][n], 0, 0, 0); } \
        __builtin_amdgcn_s_setprio(0); \
        SCHED(); WAIT_V(6); BAR(); } while (0)
    for (int i = 0; i < 63; i += 3) {
        STEP(0, g + 2 * i); STEP(1, g + 2 * i + 2); STEP(2, g + 2 * i + 4);
    }
    STEP(0, g + 126);
    WAIT_V(0);
    if (g == 0) BAR();
    BAR();
    STAMP(6);
    constexpr int EP = 272;
    const bool bad = AGT_LOAD(tmo) == call;
    if (g == 1) {
#pragma unroll
        for (int m = 0; m < 2; ++m)
#pragma unroll
            for (int n = 0; n < 4; ++n) *(LAS f32x4*)(lds + 32768 + ((wq * 8 + m * 4 + n) * 64 + lane) * 16) = acc[m][n];
    }
    WAIT_L(0); BAR();
    if (g == 0) {
#pragma unroll
        for (int m = 0; m < 2; ++m)
#pragma unroll
            for (int n = 0; n < 4; ++n) {
                const f32x4 v = acc[m][n] + *(const LAS f32x4*)(lds + 32768 + ((wq * 8 + m * 4 + n) * 64 + lane) * 16);
                u32x2 w; w.x = cvt_pk_bf16(gelu1(v[0]), gelu1(v[1])); w.y = cvt_pk_bf16(gelu1(v[2]), gelu1(v[3]));
                if (bad) { w.x = 0x7fc07fc0u; w.y = 0x7fc07fc0u; }
                *(LAS u32x2*)(lds + (wm * 32 + m * 16 + fr) * EP + (wn * 64 + n * 16 + fq * 4) * 2) = w;
            }
    }
    WAIT_L(0); BAR();
#pragma unroll
    for (int i = 0; i < 2; ++i) {
        const int r = (i * 8 + wid) * 4 + (lane >> 4);
        const u32x4 v = *(const LAS u32x4*)(lds + r * EP + (lane & 15) * 16);
        store_out(out + (size_t)r * WPITCH + (lane & 15) * 16, v);
    }
#undef STAGE
#undef CIDX
#undef CHECK
#undef POLL
#undef STEP
}

__device__ __forceinline__ void push_role(LAS unsigned char* lds, const Params& P, unsigned call, int pw) {
    const int tid = threadIdx.x, wid = __builtin_amdgcn_readfirstlane(tid >> 6), lane = tid & 63;
    const int rank = P.rank, cls = rank >> 2, idx = rank & 3;
    const int di = pw % 7, j = pw / 7;
    const bool same = di < 3;
    const int q = same ? (cls * 4 + ((idx + 1 + di) & 3)) : ((1 - cls) * 4 + (di - 3));
    const int qidx = q & 3;
    const size_t par = WS_STAGE + (size_t)(call & 1u) * PAR_BYTES;
    const int slot = same ? ((idx - qidx - 1) & 3) : idx;
    const size_t xdst_off = par + (same ? (size_t)slot * U16 * UNITB : (size_t)(3 * U16 + slot * V16) * UNITB);
    const size_t wdst_off = par + XS_BYTES + (size_t)slot * WSQ_BYTES;
    const unsigned xbytes = (unsigned)(same ? U16 : V16) * UNITB;
    const char* xsrc = P.x + (same ? 0 : (size_t)U16 * UNITB);
    const char* wsrc = P.w + qidx * 512;
    char* wsq = P.ws[q];
    const rsrc_t rxs = mk_rsrc(xsrc, xbytes), rxd = mk_rsrc(wsq + xdst_off, xbytes);
    const rsrc_t rws = mk_rsrc(wsrc, (unsigned)KDIM * WPITCH - (unsigned)qidx * 512u), rwd = mk_rsrc(wsq + wdst_off, (unsigned)WSQ_BYTES);
    unsigned* flag = (unsigned*)(wsq + WS_FLAG) + rank * FROW;
    constexpr int U = CFG_PU;
    const int nch = same ? (U2 > 0 ? NCH : NCH1) : NCH1;
    const unsigned nbt = same ? NB_S : NB_X;
    unsigned* tick = (unsigned*)(P.ws[rank] + WS_TICK) + di * 32;
    const unsigned tbase = (call - 1u) * (nbt + (unsigned)NJ);
    const unsigned tbasew = (call - 1u) * nbt - (unsigned)NJ * 8u;
    (void)tbase; (void)lds;
    unsigned t = (unsigned)(j * 8 + wid);
    int c = 0, kt0 = 0, row0 = 0, pbase = same ? PBASE_S1 : PBASE_X, nx0 = same ? PBASE_S1 : V16 * 8;
    unsigned cb0 = 0;
#define LOCATE(tt) do { for (;;) { const unsigned nbc_ = (unsigned)nbatch(pbase << CH_LOGF(c)); if ((tt) < cb0 + nbc_) break; \
            cb0 += nbc_; kt0 += 1 << CH_LOGF(c); ++c; \
            if (c == NCH1) { kt0 = 0; row0 = U1 * 64; pbase = PBASE_S2; nx0 = PBASE_S2; } } } while (0)
#define PIECE(tt, u, soff, doff, w) do { const int lg_ = CH_LOGF(c), ptot_ = pbase << lg_, nx_ = nx0 << lg_; const int i_ = (int)((tt) - cb0) * PB + (u); \
            soff = 0xffffff00u; doff = 0xffffff00u; w = false; \
            if (i_ < nx_) { const unsigned o_ = (unsigned)i_ * 1024u + (unsigned)lane * 16u; \
                const unsigned row_ = (unsigned)row0 + (o_ >> (7 + lg_)), col_ = o_ & ((128u << lg_) - 1u); soff = doff = row_ * XPITCH + (unsigned)kt0 * 128u + col_; } \
            else if (i_ < ptot_) { const unsigned k_ = (unsigned)kt0 * 64u + (unsigned)(i_ - nx_) * 2u + (unsigned)(lane >> 5); \
                soff = k_ * WPITCH + (unsigned)(lane & 31) * 16u; doff = k_ * QPITCH + (unsigned)(lane & 31) * 16u; w = true; } } while (0)
#define PLOAD(w, soff) ((w) ? __builtin_amdgcn_raw_buffer_load_b128(rws, (soff), 0, 0) : __builtin_amdgcn_raw_buffer_load_b128(rxs, (soff), 0, 0))
#define FETCH() (__hip_atomic_fetch_add(tick, 1u, __ATOMIC_RELAXED, __HIP_MEMORY_SCOPE_AGENT) - tbasew)
    u32x4 v[U], nv[U]; unsigned so[U], nso[U]; bool isw[U], nisw[U];
    int cc, pc = -1;
    unsigned tn = 0;
    int nstamp = 0; (void)nstamp;
    {
        LOCATE(t); cc = c;
        if (lane == 0) tn = FETCH();
#pragma unroll
        for (int u = 0; u < U; ++u) { unsigned sf; PIECE(t, u, sf, so[u], isw[u]); v[u] = PLOAD(isw[u], sf); }
        tn = __builtin_amdgcn_readfirstlane(tn);
    }
    for (;;) {
        const bool nlive = tn < nbt;
        unsigned tnn = 0; int nc = 0;
        if (nlive) { LOCATE(tn); nc = c; }
#pragma unroll
        for (int u = 0; u < U; ++u) {
            if (nlive) {
                if (u == 0 && lane == 0) tnn = FETCH();
                unsigned sf; PIECE(tn, u, sf, nso[u], nisw[u]); nv[u] = PLOAD(nisw[u], sf);
                WAIT_V(CFG_PA_CNT);
            } else { WAIT_V(CFG_PA_TAIL); }
            if (u == CFG_PA - 1 && pc >= 0) {
                if (lane == 0) __hip_atomic_fetch_add(flag + pc, 1u, __ATOMIC_RELAXED, __HIP_MEMORY_SCOPE_SYSTEM);
                pc = -1;
            }
            if (isw[u]) __builtin_amdgcn_raw_buffer_store_b128(v[u], rwd, so[u], 0, 17);
            else __builtin_amdgcn_raw_buffer_store_b128(v[u], rxd, so[u], 0, 17);
        }
        pc = cc;
        if (!nlive) break;
#pragma unroll
        for (int u = 0; u < U; ++u) { v[u] = nv[u]; so[u] = nso[u]; isw[u] = nisw[u]; }
        cc = nc; tn = __builtin_amdgcn_readfirstlane(tnn);
    }
    WAIT_V(0);
    if (lane == 0) __hip_atomic_fetch_add(flag + pc, 1u, __ATOMIC_RELAXED, __HIP_MEMORY_SCOPE_SYSTEM);
#undef LOCATE
#undef PIECE
#undef PLOAD
#undef FETCH
    (void)nch;
}

__device__ __forceinline__ void signal_done(const Params& P, int crank) {
    WAIT_V(0);
    BAR();
    if (threadIdx.x == 0) __hip_atomic_fetch_add((unsigned*)(P.ws[crank] + WS_DONE), 1u, __ATOMIC_RELAXED, __HIP_MEMORY_SCOPE_SYSTEM);
}

__global__ __launch_bounds__(512, 2) void ag_kernel(Params P) {
    extern __shared__ __attribute__((aligned(16))) unsigned char shm[];
    LAS unsigned char* lds = (LAS unsigned char*)shm;
    const int tid = threadIdx.x, lane = tid & 63;
    const int rank = P.rank, cls = rank >> 2, idx = rank & 3;
    char* myws = P.ws[rank];
    const int b = blockIdx.x;
    volatile LAS unsigned* cw = (volatile LAS unsigned*)(lds + LDS_CTL + 16);
    unsigned call;
    {
        if (tid == 0) *cw = __hip_atomic_fetch_add((unsigned*)(myws + WS_CTR) + b * 16, 1u, __ATOMIC_RELAXED, __HIP_MEMORY_SCOPE_AGENT) + 1u;
        __syncthreads();
        call = __builtin_amdgcn_readfirstlane(*cw);
    }
    STAMP(0);
    const size_t par = WS_STAGE + (size_t)(call & 1u) * PAR_BYTES;
    const char* xs = myws + par;
    const unsigned* flags = (const unsigned*)(myws + WS_FLAG);
    const unsigned needv = (P.dbg & 4) ? 0u : call;
    if (b < NPUSH) {
        if (!(P.dbg & 1)) push_role(lds, P, call, b);
    } else if (b >= GEMM0 && !(P.dbg & 8)) {
        const int g = b - GEMM0;
        constexpr int NA = 4 * RBA, NB = 4 * RBB;
        const int x = g & 7, s = g >> 3;
        const int a0 = x * NA / 8, a1 = (x + 1) * NA / 8, b0 = x * NB / 8, b1 = (x + 1) * NB / 8;
        const int na = a1 - a0;
        const int type = s < na ? 0 : 1;
        int t = type == 0 ? a0 + s : b0 + (s - na);
        if (type == 0) {
            constexpr int NLOC = 4 * (16 / NU), PERL = NLOC / 8;
            if (NA % 8 == 0 && NLOC % 8 == 0) t = s < PERL ? x * PERL + s : NLOC + x * (NA / 8 - PERL) + (s - PERL);
        }
        if (type == 0 || s - na < b1 - b0) {
            const int rb = t >> 2, cb = t & 3;
            TileDesc d;
            const int crank = (1 - cls) * 4 + cb;
            d.flag = flags; d.call1 = 0u; d.pbase = 0; d.anywait = false;
#pragma unroll
            for (int u = 0; u < NU; ++u) {
                int ui = rb * NU + u;
                const int nun = type == 0 ? UA : UB;
                const bool valid = ui < nun;
                if (!valid) ui = nun - 1;
                int src, su; const char* xp;
                const int nown = type == 0 ? 16 : U16;
                if (ui < nown) { src = rank; su = ui; xp = P.x + (size_t)ui * UNITB; }
                else if (ui < nown + 3 * U1) { const int sl = (ui - nown) / U1; su = (ui - nown) % U1; src = cls * 4 + ((idx + 1 + sl) & 3); xp = xs + (size_t)(sl * U16 + su) * UNITB; }
                else { int sl = (ui - nown - 3 * U1) / V16; const int jj = (ui - nown - 3 * U1) % V16; if (type == 1 && sl >= cb) ++sl;
                       src = (1 - cls) * 4 + sl; su = U16 + jj; xp = xs + (size_t)(3 * U16 + sl * V16 + jj) * UNITB; }
                d.xa[u] = xp;
                char* ob = type == 0 ? P.out[rank] + cb * 512 : P.out[crank] + idx * 512;
                d.out[u] = valid ? ob + (size_t)(src * MSH + su * 64) * WPITCH : nullptr;
                if (src != rank) { d.anywait = true; if (lane / (64 / (NU + 1)) == u) { d.flag = flags + src * FROW; d.call1 = needv; d.pbase = (src >> 2) == cls ? PBASE_S1 : PBASE_X; } }
            }
            d.throttle = type == 1;
            if (type == 0) { d.wb = P.w + cb * 512; d.wpitch = WPITCH; d.wbytes = KDIM * WPITCH - cb * 512; }
            else { d.wb = xs + XS_BYTES + (size_t)cb * WSQ_BYTES; d.wpitch = QPITCH; d.wbytes = (unsigned)WSQ_BYTES; d.anywait = true;
                   if (lane / (64 / (NU + 1)) == NU) { d.flag = flags + crank * FROW; d.call1 = needv; d.pbase = PBASE_X; } }
            gemm_tile(lds, d, P, call);
            if (type == 1) signal_done(P, crank);
        }
    }
    STAMP(3);
    if (U2 > 0 && !(P.dbg & 8)) {
        const int x = b & 7;
        for (;;) {
            __syncthreads();
            if (tid == 0) *cw = __hip_atomic_fetch_add((unsigned*)(myws + WS_QUEUE) + x * 32, 1u, __ATOMIC_RELAXED, __HIP_MEMORY_SCOPE_AGENT) - (call - 1u) * (unsigned)(NT2X + NWG / 8);
            __syncthreads();
            const unsigned tk = __builtin_amdgcn_readfirstlane(*cw);
            if (tk >= (unsigned)NT2X) break;
            const int cbk = 2 * x + (int)(tk & 1u), un = (int)(tk >> 1);
            const int sl = un / (U2 > 0 ? U2 : 1), su = U1 + un % (U2 > 0 ? U2 : 1);
            const int src = cls * 4 + ((idx + 1 + sl) & 3);
            const char* xa = xs + (size_t)(sl * U16 + su) * UNITB;
            const unsigned* xflag = flags + src * FROW + NCH1;
            const size_t orow = (size_t)(src * MSH + su * 64) * WPITCH;
            const bool own = cbk < 8;
            const int cb = (cbk - 8) >> 1, hf = cbk & 1, crank = own ? rank : (1 - cls) * 4 + cb;
            small_tile(lds, xa, own ? P.w + cbk * 256 : xs + XS_BYTES + (size_t)cb * WSQ_BYTES + hf * 256, own ? WPITCH : QPITCH,
                       own ? KDIM * WPITCH - cbk * 256 : (unsigned)WSQ_BYTES - hf * 256, P.out[crank] + orow + (own ? cbk * 256 : idx * 512 + hf * 256),
                       xflag, needv, flags + crank * FROW, own ? 0u : needv, P, call);
            if (!own) signal_done(P, crank);
            STAMP(7);
        }
    }
    if (b == 0 && !(P.dbg & 2)) {
        if (tid < 64) {
            unsigned* tmo = (unsigned*)(myws + WS_TMO);
            const bool ok = wave_wait((const unsigned*)(myws + WS_DONE), call * (unsigned)(4 * RBB + 4 * 3 * U2 * 2), tmo, call);
            if (!ok && lane == 0) *(unsigned short*)P.out[rank] = 0x7fc0u;
        }
        STAMP(4);
    }
}

}

#define AG_DBG 0
extern "C" void kernel_launch(int rank, int nranks, ncclComm_t comm, hipStream_t stream,
                              void* const* d_in, const int* in_sizes, int n_in,
                              void* d_out, int out_size, const int* in_dtypes, int out_dtype,
                              void* const* d_out_all, void* const* d_ws_all, size_t ws_bytes) {
    using namespace agk;
    (void)comm; (void)in_sizes; (void)n_in; (void)out_size; (void)in_dtypes; (void)out_dtype; (void)d_out;
    if (nranks != NR || ws_bytes < WS_NEED || rank < 0 || rank >= NR) { fprintf(stderr, "unsupported configuration\n"); return; }
    static bool init[64] = {};
    int dev = 0; HIP_CHECK(hipGetDevice(&dev));
    if (!init[dev & 63]) {
        HIP_CHECK(hipFuncSetAttribute((const void*)ag_kernel, hipFuncAttributeMaxDynamicSharedMemorySize, LDS_BYTES));
        init[dev & 63] = true;
    }
    Params P;
    P.x = (const char*)d_in[0]; P.w = (const char*)d_in[1];
    for (int r = 0; r < NR; ++r) { P.out[r] = (char*)d_out_all[r]; P.ws[r] = (char*)d_ws_all[r]; }
    P.rank = rank; P.dbg = AG_DBG;
    hipLaunchKernelGGL(ag_kernel, dim3(NWG), dim3(512), LDS_BYTES, stream, P);
    HIP_CHECK(hipGetLastError());
}
```

```cpp
#include <hip/hip_runtime.h>
#include <rccl/rccl.h>
#include <cstdio>
#include <cstdint>

#define HIP_CHECK(expr) do { hipError_t _e = (expr); if (_e != hipSuccess) { \
    fprintf(stderr, "HIP error %s at %s:%d\n", hipGetErrorString(_e), __FILE__, __LINE__); \
    return; } } while (0)

#define DBG(bit) 0

namespace agk {

#define LAS __attribute__((address_space(3)))
typedef short bf16x8 __attribute__((ext_vector_type(8)));
typedef short s16x4 __attribute__((ext_vector_type(4)));
typedef float f32x4 __attribute__((ext_vector_type(4)));
typedef unsigned u32x4 __attribute__((ext_vector_type(4)));
typedef unsigned u32x2 __attribute__((ext_vector_type(2)));
typedef __amdgpu_buffer_rsrc_t rsrc_t;

constexpr int NR = 8, KDIM = 8192, MSH = 1024, NSH = 1024;
constexpr unsigned XPITCH = KDIM * 2;
constexpr unsigned WPITCH = NSH * 2;
constexpr unsigned QPITCH = 512;
constexpr unsigned UNITB = 64u * XPITCH;

#define CFG_U16 11
#define CFG_U2 2
#define CFG_NU 4
#define CFG_NJ 16
#define CFG_EPI_INFLIGHT 0
#define CFG_PU 2
constexpr int U16 = CFG_U16, V16 = 16 - U16;
constexpr int U2 = CFG_U2, U1 = U16 - U2;
constexpr int NU = CFG_NU;
#define CFG_NJS CFG_NJ
constexpr int NJX = CFG_NJ, NJS = CFG_NJS;
constexpr int NJM = NJX < NJS ? NJX : NJS;
constexpr int NPUSH = 3 * NJS + 4 * NJX;
constexpr int UA = 16 + 3 * U1 + 4 * V16;
constexpr int UB = U16 + 3 * U1 + 3 * V16;
constexpr int RBA = (UA + NU - 1) / NU, RBB = (UB + NU - 1) / NU;
constexpr int NWG = 256;
constexpr int NGWG = 8 * ((4 * RBA + 7) / 8 + (4 * RBB + 7) / 8);
static_assert(NPUSH + NGWG <= NWG, "grid");
constexpr int GEMM0 = NWG - NGWG;
constexpr int NT2X = 3 * U2 * 2;

#define CFG_PA 1
#define CFG_PA_CNT 1
#define CFG_PA_TAIL 0
#define CFG_POLL_SLEEP 2
#define CFG_EPI_W2 0
#define CFG_ASYNC 0
#define CFG_PMODE 0
#define WAIT_VU2() WAIT_V(4)
#define WAIT_VU1() WAIT_V(3)
#define CFG_LG1 2
#define CFG_LG2 2
constexpr int NT = KDIM / 64;
constexpr int LG1 = CFG_LG1, LG2 = CFG_LG2;
__host__ __device__ constexpr int ch_body(int lg) { return (NT >> lg) - 1; }
__host__ __device__ constexpr int ch_count(int lg) { return ch_body(lg) + lg + 1; }
__host__ __device__ constexpr int ch_lg(int lg, int c) { return c < ch_body(lg) ? lg : (c - ch_body(lg) < lg ? lg - 1 - (c - ch_body(lg)) : 0); }
constexpr int NCH1 = ch_count(LG1), NCH2 = ch_count(LG2), NCH = NCH1 + NCH2;
__host__ __device__ constexpr int CH_LOGF(int c) { return c < NCH1 ? ch_lg(LG1, c) : ch_lg(LG2, c - NCH1); }
constexpr int ch_sum(int a, int b) { int s = 0; for (int i = a; i < b; ++i) s += 1 << CH_LOGF(i); return s; }
static_assert(ch_sum(0, NCH1) == NT && ch_sum(NCH1, NCH) == NT, "chunks");
constexpr unsigned long long ch_mask(int gen, int half) { unsigned long long m = 0; int s = 0;
    for (int i = gen ? NCH1 : 0; i < (gen ? NCH : NCH1); ++i) { if (s / 64 == half) m |= 1ull << (s % 64); s += 1 << CH_LOGF(i); } return m; }
constexpr unsigned long long CHM0 = ch_mask(0, 0), CHM1 = ch_mask(0, 1), CH2M0 = ch_mask(1, 0), CH2M1 = ch_mask(1, 1);
constexpr int FROW = 256;
static_assert(NCH <= FROW, "flags");

#define CFG_WAVETK 1
constexpr int PB = (CFG_WAVETK ? 1 : 8) * CFG_PU;
constexpr int PBASE_S1 = U1 * 8, PBASE_S2 = U2 * 8, PBASE_X = V16 * 8 + 32;
__host__ __device__ constexpr int nbatch(int pieces) { return (pieces + PB - 1) / PB; }
constexpr int nb_total(bool same) { int s = 0; for (int c = 0; c < NCH; ++c) { const int pb = c < NCH1 ? (same ? PBASE_S1 : PBASE_X) : (same ? PBASE_S2 : 0); s += nbatch(pb << CH_LOGF(c)); } return s; }
constexpr int NB_S = nb_total(true), NB_X = nb_total(false);

constexpr size_t WS_CTR = 0;
constexpr size_t WS_TMO = 16384;
constexpr size_t WS_DONE = 16384 + 256;
constexpr size_t WS_QUEUE = 16384 + 512;
constexpr size_t WS_TICK = 16384 + 2048;
constexpr size_t WS_FLAG = 32768;
constexpr size_t WS_DBG = 65536;
constexpr size_t WS_STAGE = 1u << 20;
constexpr size_t XS_BYTES = (size_t)(3 * U16 + 4 * V16) * UNITB;
constexpr size_t WSQ_BYTES = (size_t)KDIM * QPITCH;
constexpr size_t PAR_BYTES = XS_BYTES + 4 * WSQ_BYTES;
constexpr size_t WS_NEED = WS_STAGE + 2 * PAR_BYTES;
constexpr unsigned long long DEADLINE = 100000000ull;

constexpr int LDS_STAGE = 2 * NU * 8192 + 65536;
constexpr int LDS_EPI = NU * 64 * 528;
constexpr int LDS_SMALL = 6 * 24576;
constexpr int cmax(int a, int b) { return a > b ? a : b; }
constexpr int LDS_CTL = cmax(cmax(LDS_STAGE, LDS_EPI), LDS_SMALL);
constexpr int LDS_BYTES = LDS_CTL + 256;

struct Params {
    const char* x; const char* w;
    char* out[NR]; char* ws[NR];
    int rank; int dbg;
};

__device__ __forceinline__ unsigned cvt_pk_bf16(float lo, float hi) { unsigned r; asm("v_cvt_pk_bf16_f32 %0, %1, %2" : "=v"(r) : "v"(lo), "v"(hi)); return r; }
__device__ __forceinline__ float gelu1(float y) {
    const float c2 = -2.0f * 0.7978845608028654f * 1.4426950408889634f;
    const float a = y * __builtin_fmaf(y * y, 0.044715f * c2, c2);
    return y * __builtin_amdgcn_rcpf(1.0f + __builtin_amdgcn_exp2f(a));
}
__device__ __forceinline__ rsrc_t mk_rsrc(const void* p, unsigned bytes) { return __builtin_amdgcn_make_buffer_rsrc((void*)p, 0, bytes, 0x00020000); }

#define WAIT_V_(n) asm volatile("s_waitcnt vmcnt(" #n ")" ::: "memory")
#define WAIT_V(n) WAIT_V_(n)
#define WAIT_L(n) asm volatile("s_waitcnt lgkmcnt(" #n ")" ::: "memory")
#define BAR() __builtin_amdgcn_s_barrier()
#define SCHED() __builtin_amdgcn_sched_barrier(0)
#define SYS_LOAD(p) __hip_atomic_load((p), __ATOMIC_RELAXED, __HIP_MEMORY_SCOPE_SYSTEM)
#define AGT_LOAD(p) __hip_atomic_load((p), __ATOMIC_RELAXED, __HIP_MEMORY_SCOPE_AGENT)
#define CFG_DMA_AUX 17
#define DMA(rs, ldsp, voff, soff) __builtin_amdgcn_raw_ptr_buffer_load_lds((rs), (LAS void*)(ldsp), 16, (voff), (soff), 0, CFG_DMA_AUX)
#define TRR(dst, base, off) asm volatile("ds_read_b64_tr_b16 %0, %1 offset:%2" : "=v"(dst) : "v"(base), "i"(off) : "memory")

#define STAMP(slot) do {} while (0)

__device__ __forceinline__ bool wave_wait(const unsigned* p, unsigned need, unsigned* tmo, unsigned call) {
    const unsigned long long t0 = __builtin_amdgcn_s_memrealtime();
    for (;;) {
        const unsigned v = SYS_LOAD(p);
        if (__all((int)(v - need) >= 0 || need == 0u)) return true;
        if (__builtin_amdgcn_readfirstlane(AGT_LOAD(tmo)) == call || __builtin_amdgcn_s_memrealtime() - t0 > DEADLINE) {
            __hip_atomic_store(tmo, call, __ATOMIC_RELAXED, __HIP_MEMORY_SCOPE_AGENT);
            return false;
        }
        __builtin_amdgcn_s_sleep(4);
    }
}

template <int WN, int NS>
__device__ __forceinline__ int window_wait(const unsigned* p, unsigned need, unsigned* tmo, unsigned call) {
    const unsigned long long t0 = __builtin_amdgcn_s_memrealtime();
    for (;;) {
        const unsigned v = SYS_LOAD(p);
        unsigned long long m = __ballot((int)(v - need) >= 0 || need == 0u), a = m;
#pragma unroll
        for (int s = 1; s < NS; ++s) a &= m >> (s * WN);
        a &= (1ull << WN) - 1ull;
        if (a & 1ull) return (int)__builtin_ctzll(~a);
        if (__builtin_amdgcn_readfirstlane(AGT_LOAD(tmo)) == call || __builtin_amdgcn_s_memrealtime() - t0 > DEADLINE) {
            __hip_atomic_store(tmo, call, __ATOMIC_RELAXED, __HIP_MEMORY_SCOPE_AGENT);
            return WN;
        }
        __builtin_amdgcn_s_sleep(CFG_POLL_SLEEP);
    }
}

__device__ __forceinline__ void store_out(char* o, u32x4 v) { asm volatile("global_store_dwordx4 %0, %1, off sc0 sc1\n\ts_nop 1" :: "v"(o), "v"(v) : "memory"); }

struct TileDesc {
    const char* xa[NU];
    const char* wb; unsigned wpitch, wbytes;
    char* out[NU];
    const unsigned* flag;
    unsigned call1;
    int pbase;
    bool anywait, throttle;
};

__device__ __forceinline__ void gemm_tile(LAS unsigned char* lds, const TileDesc& d, const Params& P, unsigned call) {
    constexpr int MT = NU * 2;
    constexpr int ABUF = NU * 8192;
    constexpr int BOFF = 2 * ABUF;
    const int tid = threadIdx.x, wid = __builtin_amdgcn_readfirstlane(tid >> 6), lane = tid & 63;
    const int wr = wid >> 2, wc = wid & 3, fr = lane & 15, fq = lane >> 4;
    unsigned* tmo = (unsigned*)(P.ws[P.rank] + WS_TMO);
    volatile LAS unsigned* ready = (volatile LAS unsigned*)(lds + LDS_CTL);
    const unsigned voffA = (unsigned)(8 * wid + (lane >> 3)) * XPITCH + (unsigned)(((lane & 7) ^ (((wid & 1) << 2) | (lane >> 4))) * 16);
    const unsigned fB = (unsigned)((lane >> 4) | (((wid >> 1) & 1) << 2));
    const unsigned voffB = (unsigned)(4 * wid + (lane >> 4)) * d.wpitch + ((((unsigned)(lane & 15) >> 1) ^ fB) * 32u + (unsigned)(lane & 1) * 16u);
    const unsigned ldsw = (unsigned)wid * 1024u;
    const unsigned aoff0 = (unsigned)(wr * (NU * 32) + fr) * 128u + (unsigned)((fq ^ (fr >> 1)) * 16);
    const unsigned aoff1 = aoff0 ^ 64u;
    const unsigned q = (unsigned)(fr >> 2), p = (unsigned)(fr & 3);
    const unsigned boff0 = (unsigned)(size_t)lds + BOFF + (unsigned)(8 * fq + q) * 256u + (((unsigned)(2 * wc) ^ (q | ((unsigned)(fq & 1) << 2))) * 32u) + p * 8u;
    const unsigned boff1 = boff0 ^ 32u;
    rsrc_t ra[NU];
#pragma unroll
    for (int u = 0; u < NU; ++u) ra[u] = mk_rsrc(d.xa[u], UNITB);
    const rsrc_t rb = mk_rsrc(d.wb, d.wbytes);
    const unsigned wstep = __builtin_amdgcn_readfirstlane(64u * d.wpitch), whalf = wstep >> 1;

    f32x4 acc[2][MT][2];
#pragma unroll
    for (int b = 0; b < 2; ++b)
#pragma unroll
        for (int m = 0; m < MT; ++m)
#pragma unroll
            for (int n = 0; n < 2; ++n) acc[b][m][n] = (f32x4){0.f, 0.f, 0.f, 0.f};
    bf16x8 af[MT][2], bfr[2][2]; s16x4 tb[8];

#define STAGE_A(buf, t) do { _Pragma("unroll") for (int _i = 0; _i < NU; ++_i) DMA(ra[_i], lds + (buf) * ABUF + _i * 8192 + ldsw, voffA, (unsigned)(t) * 128u); } while (0)
#define STAGE_B(buf, h, t) do { _Pragma("unroll") for (int _i = 0; _i < 2; ++_i) \
        DMA(rb, lds + BOFF + ((buf) * 2 + (h)) * 16384 + _i * 8192 + ldsw, voffB, (unsigned)(t) * wstep + _i * whalf + (h) * 256u); } while (0)
#define LDA(buf) do { _Pragma("unroll") for (int m = 0; m < MT; ++m) { \
        af[m][0] = *(const LAS bf16x8*)(lds + (buf) * ABUF + aoff0 + m * 2048); \
        af[m][1] = *(const LAS bf16x8*)(lds + (buf) * ABUF + aoff1 + m * 2048); } } while (0)
#define LDB(buf, h) do { \
        TRR(tb[0], boff0, ((buf) * 2 + (h)) * 16384); TRR(tb[1], boff0, ((buf) * 2 + (h)) * 16384 + 1024); \
        TRR(tb[2], boff1, ((buf) * 2 + (h)) * 16384); TRR(tb[3], boff1, ((buf) * 2 + (h)) * 16384 + 1024); \
        TRR(tb[4], boff0, ((buf) * 2 + (h)) * 16384 + 8192); TRR(tb[5], boff0, ((buf) * 2 + (h)) * 16384 + 8192 + 1024); \
        TRR(tb[6], boff1, ((buf) * 2 + (h)) * 16384 + 8192); TRR(tb[7], boff1, ((buf) * 2 + (h)) * 16384 + 8192 + 1024); } while (0)
#define WAITB() do { asm volatile("s_waitcnt lgkmcnt(0)" : "+v"(tb[0]), "+v"(tb[1]), "+v"(tb[2]), "+v"(tb[3]), "+v"(tb[4]), "+v"(tb[5]), "+v"(tb[6]), "+v"(tb[7]) :: "memory"); \
        bfr[0][0] = __builtin_shufflevector(tb[0], tb[1], 0, 1, 2, 3, 4, 5, 6, 7); bfr[1][0] = __builtin_shufflevector(tb[2], tb[3], 0, 1, 2, 3, 4, 5, 6, 7); \
        bfr[0][1] = __builtin_shufflevector(tb[4], tb[5], 0, 1, 2, 3, 4, 5, 6, 7); bfr[1][1] = __builtin_shufflevector(tb[6], tb[7], 0, 1, 2, 3, 4, 5, 6, 7); } while (0)
#define MMA(bj) do { __builtin_amdgcn_s_setprio(1); \
        _Pragma("unroll") for (int kk = 0; kk < 2; ++kk) _Pragma("unroll") for (int m = 0; m < MT; ++m) _Pragma("unroll") for (int n = 0; n < 2; ++n) \
            acc[bj][m][n] = __builtin_amdgcn_mfma_f32_16x16x32_bf16(bfr[n][kk], af[m][kk], acc[bj][m][n], 0, 0, 0); \
        __builtin_amdgcn_s_setprio(0); } while (0)
#define WAITN() do { if constexpr (NU == 3) { WAIT_V(7); } else if constexpr (NU == 2) { WAIT_V(6); } else { WAIT_V(8); } } while (0)
    constexpr int NS = NU + 1, WN = 64 / NS;
    int cnext = 0, known = 0;
#define POLL() do { const int ck_ = cnext + lane % WN; const bool w_ = ck_ < NCH1 && lane < NS * WN; const int cc_ = w_ ? ck_ : 0; \
        known = cnext + window_wait<WN, NS>(d.flag + cc_, w_ ? d.call1 * (unsigned)nbatch(d.pbase << CH_LOGF(cc_)) : 0u, tmo, call); } while (0)
#define GATE() do { \
        if (known <= cnext) { \
            if (wid == 0) { POLL(); if (lane == 0) *ready = (unsigned)known; } \
            else { unsigned r_; while ((r_ = *ready) <= (unsigned)cnext) __builtin_amdgcn_s_sleep(1); known = (int)__builtin_amdgcn_readfirstlane(r_); } \
        } \
        if ((cnext & 3) == 0) STAMP(8 + (cnext >> 2)); \
        ++cnext; } while (0)
#define CHECK(tt) do { if (d.anywait && (tt) < NT && ((((tt) < 64 ? CHM0 : CHM1) >> ((tt) & 63)) & 1ull)) GATE(); } while (0)

    if (d.anywait) {
        if (wid == 0) { POLL(); if (lane == 0) *ready = (unsigned)known; }
        WAIT_L(0); BAR();
        known = (int)__builtin_amdgcn_readfirstlane(*ready);
        cnext = 1;
    }
    static_assert(LG1 >= 1, "prologue loads K tiles 0, 1");
    STAMP(1);
    STAGE_A(0, 0); STAGE_B(0, 0, 0); STAGE_B(0, 1, 0);
    STAGE_A(1, 1); STAGE_B(1, 0, 1);
    if (wr == 1) BAR();
    if constexpr (NU == 3) { WAIT_V(5); } else if constexpr (NU == 2) { WAIT_V(4); } else { WAIT_V(6); }
    BAR(); BAR();
#define KTILE(b, t) do { \
        LDB(b, 0); LDA(b); \
        STAGE_B((b) ^ 1, 1, min((t) + 1, NT - 1)); \
        WAITN(); WAITB(); BAR(); SCHED(); MMA(0); SCHED(); BAR(); \
        CHECK((t) + 2); \
        LDB(b, 1); \
        STAGE_A(b, min((t) + 2, NT - 1)); STAGE_B(b, 0, min((t) + 2, NT - 1)); \
        WAITN(); WAITB(); BAR(); SCHED(); MMA(1); SCHED(); BAR(); } while (0)
    for (int t = 0; t < NT; t += 2) {
        KTILE(0, t);
        KTILE(1, t + 1);
    }
    WAIT_V(0);
    if (wr == 0) BAR();
    BAR();
    STAMP(2);
    constexpr int EP = 528;
    const bool bad = AGT_LOAD(tmo) == call;
#pragma unroll
    for (int bj = 0; bj < 2; ++bj)
#pragma unroll
        for (int m = 0; m < MT; ++m)
#pragma unroll
            for (int n = 0; n < 2; ++n) {
                const f32x4 v = acc[bj][m][n];
                u32x2 w; w.x = cvt_pk_bf16(gelu1(v[0]), gelu1(v[1])); w.y = cvt_pk_bf16(gelu1(v[2]), gelu1(v[3]));
                if (bad) { w.x = 0x7fc07fc0u; w.y = 0x7fc07fc0u; }
                *(LAS u32x2*)(lds + (wr * (NU * 32) + m * 16 + fr) * EP + (bj * 128 + wc * 32 + n * 16 + fq * 4) * 2) = w;
            }
    WAIT_L(0); BAR();
#pragma unroll
    for (int u = 0; u < NU; ++u) {
        if (d.out[u] != nullptr) {
#pragma unroll
            for (int i = 0; i < 4; ++i) {
                const int r = (i * 8 + wid) * 2 + (lane >> 5);
                const u32x4 v = *(const LAS u32x4*)(lds + (u * 64 + r) * EP + (lane & 31) * 16);
                store_out(d.out[u] + (size_t)r * WPITCH + (lane & 31) * 16, v);
                if (d.throttle) { if (wid < CFG_EPI_W2) WAIT_V(1); else WAIT_V(0); }
            }
        }
    }
#undef STAGE_A
#undef STAGE_B
#undef LDA
#undef LDB
#undef MMA
#undef KTILE
#undef WAITB
#undef WAITN
#undef GATE
#undef CHECK
#undef POLL
}

__device__ __forceinline__ void small_tile(LAS unsigned char* lds, const char* xa, const char* wb, unsigned wpitch, unsigned wbytes, char* out,
                                           const unsigned* xflag, unsigned xcall, const unsigned* wflag, unsigned wcall, const Params& P, unsigned call) {
    const int tid = threadIdx.x, wid = __builtin_amdgcn_readfirstlane(tid >> 6), lane = tid & 63;
    const int g = wid >> 2, wq = wid & 3, wm = wq >> 1, wn = wq & 1, fr = lane & 15, fq = lane >> 4;
    unsigned* tmo = (unsigned*)(P.ws[P.rank] + WS_TMO);
    volatile LAS unsigned* ready = (volatile LAS unsigned*)(lds + LDS_CTL + 32) + g;
    const unsigned voffA = (unsigned)(8 * wq + (lane >> 3)) * XPITCH + (unsigned)(((lane & 7) ^ (((wq & 1) << 2) | (lane >> 4))) * 16);
    const unsigned fB = (unsigned)((lane >> 4) | (((wq >> 1) & 1) << 2));
    const unsigned voffB = (unsigned)(4 * wq + (lane >> 4)) * wpitch + ((((unsigned)(lane & 15) >> 1) ^ fB) * 32u + (unsigned)(lane & 1) * 16u);
    const unsigned gbase = (unsigned)g * (3u * 24576u);
    const unsigned ldsw = gbase + (unsigned)wq * 1024u;
    const unsigned aoff0 = gbase + (unsigned)(wm * 32 + fr) * 128u + (unsigned)((fq ^ (fr >> 1)) * 16);
    const unsigned aoff1 = aoff0 ^ 64u;
    const unsigned q = (unsigned)(fr >> 2), p = (unsigned)(fr & 3);
    const unsigned boff = (unsigned)(size_t)lds + gbase + 8192u + (unsigned)(8 * fq + q) * 256u + (((unsigned)(4 * wn) ^ (q | ((unsigned)(fq & 1) << 2))) * 32u) + p * 8u;
    const unsigned bo0 = boff, bo1 = boff ^ 32u, bo2 = boff ^ 64u, bo3 = boff ^ 96u;
    const rsrc_t ra = mk_rsrc(xa, UNITB), rb = mk_rsrc(wb, wbytes);
    const unsigned wstep = __builtin_amdgcn_readfirstlane(64u * wpitch), wq16 = wstep >> 2;
    f32x4 acc[2][4];
#pragma unroll
    for (int m = 0; m < 2; ++m)
#pragma unroll
        for (int n = 0; n < 4; ++n) acc[m][n] = (f32x4){0.f, 0.f, 0.f, 0.f};
#define STAGE(s, t) do { DMA(ra, lds + (s) * 24576 + ldsw, voffA, (unsigned)(t) * 128u); DMA(ra, lds + (s) * 24576 + 4096 + ldsw, voffA, (unsigned)(t) * 128u + 32u * XPITCH); \
        _Pragma("unroll") for (int _j = 0; _j < 4; ++_j) DMA(rb, lds + (s) * 24576 + 8192 + _j * 4096 + ldsw, voffB, (unsigned)(t) * wstep + _j * wq16); } while (0)
    int known = 0;
#define CIDX(tt) ((tt) < 64 ? __builtin_popcountll(CH2M0 & ((2ull << (tt)) - 1ull)) - 1 : __builtin_popcountll(CH2M0) + __builtin_popcountll(CH2M1 & ((2ull << ((tt) - 64)) - 1ull)) - 1)
    static_assert(NCH2 <= 64 && NCH2 < 64, "one flag per lane");
    const unsigned* faddr = xflag + (lane < NCH2 ? lane : 0);
    const unsigned fneed = lane < NCH2 ? xcall * (unsigned)nbatch(PBASE_S2 << CH_LOGF(NCH1 + (lane < NCH2 ? lane : 0))) : 0u;
    const unsigned fmask = wq == 0 ? 1u : 0u;
    unsigned pv = 0;
#define KNOWN_OF(v) ((int)__builtin_ctzll(~__ballot((int)((v) - fneed) >= 0 && lane < NCH2)))
#define ASYNC_F() do { unsigned long long ex_; asm volatile("v_cmp_ne_u32 vcc, 0, %3\n\ts_and_saveexec_b64 %1, vcc\n\tglobal_load_dword %0, %2, off sc0 sc1\n\ts_mov_b64 exec, %1" \
        : "+v"(pv), "=&s"(ex_) : "v"(faddr), "v"(fmask) : "memory", "vcc", "scc"); } while (0)
#define POLL(ci) do { const unsigned long long t0_ = __builtin_amdgcn_s_memrealtime(); \
        for (;;) { const unsigned v_ = SYS_LOAD(faddr); known = KNOWN_OF(v_); if (known > (ci)) break; \
            if (__builtin_amdgcn_readfirstlane(AGT_LOAD(tmo)) == call || __builtin_amdgcn_s_memrealtime() - t0_ > DEADLINE) { \
                __hip_atomic_store(tmo, call, __ATOMIC_RELAXED, __HIP_MEMORY_SCOPE_AGENT); known = NCH2; break; } \
            __builtin_amdgcn_s_sleep(1); } } while (0)
#define CHECK(tt) do { \
        if (CFG_ASYNC == 2 && wq == 0) { asm volatile("" : "+v"(pv)); const int k_ = KNOWN_OF(pv); if (k_ > known) { known = k_; if (lane == 0) *ready = (unsigned)known; } } \
        if ((tt) < NT) { const int ci_ = CIDX(tt); if (known <= ci_) { \
            if (wq == 0) { POLL(ci_); if (lane == 0) *ready = (unsigned)known; } \
            else { unsigned r_; while ((r_ = *ready) <= (unsigned)ci_) __builtin_amdgcn_s_sleep(1); known = (int)__builtin_amdgcn_readfirstlane(r_); } } } \
        if (CFG_ASYNC >= 1) ASYNC_F(); } while (0)
    BAR();
    if (wid == 0) {
        for (int c0 = 0; c0 < NCH1; c0 += 64) { const int lc = c0 + lane < NCH1 ? c0 + lane : 0;
            wave_wait(wflag + lc, c0 + lane < NCH1 ? wcall * (unsigned)nbatch(PBASE_X << CH_LOGF(lc)) : 0u, tmo, call); }
        POLL(0); if (lane == 0) { ready[0] = (unsigned)known; ready[1] = (unsigned)known; }
    }
    WAIT_L(0); BAR();
    known = (int)__builtin_amdgcn_readfirstlane(*ready);
    STAMP(5);
    static_assert(LG2 >= 2, "small tile prologue loads K tiles 0..3");
    ASYNC_F();
    STAGE(0, g); STAGE(1, g + 2);
    WAIT_V(6); BAR();
    if (g == 1) BAR();
    s16x4 tb[16]; bf16x8 af[2][2];
#define STEP(s, t) do { \
        CHECK((t) + 4); \
        TRR(tb[0], bo0, (s) * 24576); TRR(tb[1], bo0, (s) * 24576 + 1024); TRR(tb[2], bo1, (s) * 24576); TRR(tb[3], bo1, (s) * 24576 + 1024); \
        TRR(tb[4], bo2, (s) * 24576); TRR(tb[5], bo2, (s) * 24576 + 1024); TRR(tb[6], bo3, (s) * 24576); TRR(tb[7], bo3, (s) * 24576 + 1024); \
        TRR(tb[8], bo0, (s) * 24576 + 8192); TRR(tb[9], bo0, (s) * 24576 + 9216); TRR(tb[10], bo1, (s) * 24576 + 8192); TRR(tb[11], bo1, (s) * 24576 + 9216); \
        TRR(tb[12], bo2, (s) * 24576 + 8192); TRR(tb[13], bo2, (s) * 24576 + 9216); TRR(tb[14], bo3, (s) * 24576 + 8192); TRR(tb[15], bo3, (s) * 24576 + 9216); \
        _Pragma("unroll") for (int m = 0; m < 2; ++m) { af[m][0] = *(const LAS bf16x8*)(lds + (s) * 24576 + aoff0 + m * 2048); af[m][1] = *(const LAS bf16x8*)(lds + (s) * 24576 + aoff1 + m * 2048); } \
        STAGE(((s) + 2) % 3, min((t) + 4, NT - 2 + g)); \
        asm volatile("s_waitcnt lgkmcnt(0)" : "+v"(tb[0]), "+v"(tb[1]), "+v"(tb[2]), "+v"(tb[3]), "+v"(tb[4]), "+v"(tb[5]), "+v"(tb[6]), "+v"(tb[7]), \
                     "+v"(tb[8]), "+v"(tb[9]), "+v"(tb[10]), "+v"(tb[11]), "+v"(tb[12]), "+v"(tb[13]), "+v"(tb[14]), "+v"(tb[15]) :: "memory"); \
        BAR(); SCHED(); \
        __builtin_amdgcn_s_setprio(1); \
        _Pragma("unroll") for (int kk = 0; kk < 2; ++kk) _Pragma("unroll") for (int n = 0; n < 4; ++n) { \
            const bf16x8 b_ = __builtin_shufflevector(tb[kk * 8 + n * 2], tb[kk * 8 + n * 2 + 1], 0, 1, 2, 3, 4, 5, 6, 7); \
            _Pragma("unroll") for (int m = 0; m < 2; ++m) acc[m][n] = __builtin_amdgcn_mfma_f32_16x16x32_bf16(b_, af[m][kk], acc[m][n], 0, 0, 0); } \
        __builtin_amdgcn_s_setprio(0); \
        SCHED(); WAIT_V(6); BAR(); } while (0)
    for (int i = 0; i < 63; i += 3) {
        STEP(0, g + 2 * i); STEP(1, g + 2 * i + 2); STEP(2, g + 2 * i + 4);
    }
    STEP(0, g + 126);
    WAIT_V(0);
    asm volatile("" :: "v"(pv));
    if (g == 0) BAR();
    BAR();
    STAMP(6);
    constexpr int EP = 272;
    const bool bad = AGT_LOAD(tmo) == call;
    if (g == 1) {
#pragma unroll
        for (int m = 0; m < 2; ++m)
#pragma unroll
            for (int n = 0; n < 4; ++n) *(LAS f32x4*)(lds + 32768 + ((wq * 8 + m * 4 + n) * 64 + lane) * 16) = acc[m][n];
    }
    WAIT_L(0); BAR();
    if (g == 0) {
#pragma unroll
        for (int m = 0; m < 2; ++m)
#pragma unroll
            for (int n = 0; n < 4; ++n) {
                const f32x4 v = acc[m][n] + *(const LAS f32x4*)(lds + 32768 + ((wq * 8 + m * 4 + n) * 64 + lane) * 16);
                u32x2 w; w.x = cvt_pk_bf16(gelu1(v[0]), gelu1(v[1])); w.y = cvt_pk_bf16(gelu1(v[2]), gelu1(v[3]));
                if (bad) { w.x = 0x7fc07fc0u; w.y = 0x7fc07fc0u; }
                *(LAS u32x2*)(lds + (wm * 32 + m * 16 + fr) * EP + (wn * 64 + n * 16 + fq * 4) * 2) = w;
            }
    }
    WAIT_L(0); BAR();
#pragma unroll
    for (int i = 0; i < 2; ++i) {
        const int r = (i * 8 + wid) * 4 + (lane >> 4);
        const u32x4 v = *(const LAS u32x4*)(lds + r * EP + (lane & 15) * 16);
        store_out(out + (size_t)r * WPITCH + (lane & 15) * 16, v);
    }
#undef STAGE
#undef CIDX
#undef CHECK
#undef POLL
#undef STEP
#undef ASYNC_F
#undef KNOWN_OF
}

__device__ __forceinline__ void push_role(LAS unsigned char* lds, const Params& P, unsigned call, int pw) {
    const int tid = threadIdx.x, wid = __builtin_amdgcn_readfirstlane(tid >> 6), lane = tid & 63;
    const int rank = P.rank, cls = rank >> 2, idx = rank & 3;
    int di, j;
    if (pw < 7 * NJM) { di = pw % 7; j = pw / 7; }
    else if (NJS > NJX) { const int r = pw - 7 * NJM; di = r % 3; j = NJM + r / 3; }
    else { const int r = pw - 7 * NJM; di = 3 + r % 4; j = NJM + r / 4; }
    const bool same = di < 3;
    const int q = same ? (cls * 4 + ((idx + 1 + di) & 3)) : ((1 - cls) * 4 + (di - 3));
    const int qidx = q & 3;
    const size_t par = WS_STAGE + (size_t)(call & 1u) * PAR_BYTES;
    const int slot = same ? ((idx - qidx - 1) & 3) : idx;
    const size_t xdst_off = par + (same ? (size_t)slot * U16 * UNITB : (size_t)(3 * U16 + slot * V16) * UNITB);
    const size_t wdst_off = par + XS_BYTES + (size_t)slot * WSQ_BYTES;
    const unsigned xbytes = (unsigned)(same ? U16 : V16) * UNITB;
    const char* xsrc = P.x + (same ? 0 : (size_t)U16 * UNITB);
    const char* wsrc = P.w + qidx * 512;
    char* wsq = P.ws[q];
    const rsrc_t rxs = mk_rsrc(xsrc, xbytes), rxd = mk_rsrc(wsq + xdst_off, xbytes);
    const rsrc_t rws = mk_rsrc(wsrc, (unsigned)KDIM * WPITCH - (unsigned)qidx * 512u), rwd = mk_rsrc(wsq + wdst_off, (unsigned)WSQ_BYTES);
    unsigned* flag = (unsigned*)(wsq + WS_FLAG) + rank * FROW;
    constexpr int U = CFG_PU;
    const int nch = same ? (U2 > 0 ? NCH : NCH1) : NCH1;
    const unsigned nbt = same ? NB_S : NB_X;
    unsigned* tick = (unsigned*)(P.ws[rank] + WS_TICK) + di * 32;
    const unsigned tbase = 0;
    const unsigned tbasew = (call - 1u) * nbt - (unsigned)(same ? NJS : NJX) * 8u;
    (void)tbase; (void)lds;
    unsigned t = (unsigned)(j * 8 + wid);
    int c = 0, kt0 = 0, row0 = 0, pbase = same ? PBASE_S1 : PBASE_X, nx0 = same ? PBASE_S1 : V16 * 8;
    unsigned cb0 = 0;
#define LOCATE(tt) do { for (;;) { const unsigned nbc_ = (unsigned)nbatch(pbase << CH_LOGF(c)); if ((tt) < cb0 + nbc_) break; \
            cb0 += nbc_; kt0 += 1 << CH_LOGF(c); ++c; \
            if (c == NCH1) { kt0 = 0; row0 = U1 * 64; pbase = PBASE_S2; nx0 = PBASE_S2; } } } while (0)
#define PIECE(tt, u, soff, doff, w) do { const int lg_ = CH_LOGF(c), ptot_ = pbase << lg_, nx_ = nx0 << lg_; const int i_ = (int)((tt) - cb0) * PB + (u); \
            soff = 0xffffff00u; doff = 0xffffff00u; w = false; \
            if (i_ < nx_) { const unsigned o_ = (unsigned)i_ * 1024u + (unsigned)lane * 16u; \
                const unsigned row_ = (unsigned)row0 + (o_ >> (7 + lg_)), col_ = o_ & ((128u << lg_) - 1u); soff = doff = row_ * XPITCH + (unsigned)kt0 * 128u + col_; } \
            else if (i_ < ptot_) { const unsigned k_ = (unsigned)kt0 * 64u + (unsigned)(i_ - nx_) * 2u + (unsigned)(lane >> 5); \
                soff = k_ * WPITCH + (unsigned)(lane & 31) * 16u; doff = k_ * QPITCH + (unsigned)(lane & 31) * 16u; w = true; } } while (0)
#define PLOAD(w, soff) ((w) ? __builtin_amdgcn_raw_buffer_load_b128(rws, (soff), 0, 0) : __builtin_amdgcn_raw_buffer_load_b128(rxs, (soff), 0, 0))
#define FETCH_ASYNC(dst) do { unsigned long long ex_; asm volatile("s_mov_b64 %1, exec\n\ts_mov_b64 exec, 1\n\tglobal_atomic_add %0, %2, %3, %4 sc0\n\ts_mov_b64 exec, %1" \
        : "=&v"(dst), "=&s"(ex_) : "v"(0u), "v"(1u), "s"(tick) : "memory"); } while (0)
#define FETCH_ASYNC2(dst, inc) do { unsigned long long ex_; asm volatile("s_mov_b64 %1, exec\n\ts_mov_b64 exec, 1\n\tglobal_atomic_add %0, %2, %3, %4 sc0\n\ts_mov_b64 exec, %1" \
        : "=&v"(dst), "=&s"(ex_) : "v"(0u), "v"(inc), "s"(tick) : "memory"); } while (0)
#define FETCH() (__hip_atomic_fetch_add(tick, 1u, __ATOMIC_RELAXED, __HIP_MEMORY_SCOPE_AGENT) - tbasew)
    auto desc = [](const void* p, unsigned bytes) { const unsigned long long a = (unsigned long long)p; u32x4 r;
        r.x = __builtin_amdgcn_readfirstlane((unsigned)a); r.y = __builtin_amdgcn_readfirstlane((unsigned)(a >> 32) & 0xffffu); r.z = __builtin_amdgcn_readfirstlane(bytes); r.w = 0x00020000u; return r; };
    const u32x4 dxs = desc(xsrc, xbytes), dxd = desc(wsq + xdst_off, xbytes), dws = desc(wsrc, (unsigned)KDIM * WPITCH - (unsigned)qidx * 512u), dwd = desc(wsq + wdst_off, (unsigned)WSQ_BYTES);
#define LD(dst, w, off) asm volatile("buffer_load_dwordx4 %0, %1, %2, 0 offen" : "=v"(dst) : "v"(off), "s"((w) ? dws : dxs) : "memory")
#define ST(src, w, off) asm volatile("buffer_store_dwordx4 %0, %1, %2, 0 offen sc0 sc1\n\ts_nop 1" :: "v"(src), "v"(off), "s"((w) ? dwd : dxd) : "memory")
#define FLAG_ADD(idx, val) do { unsigned long long ex_; asm volatile("s_mov_b64 %0, exec\n\ts_mov_b64 exec, 1\n\tglobal_atomic_add %1, %2, %3 sc1\n\ts_mov_b64 exec, %0" \
        : "=&s"(ex_) : "v"((unsigned)(idx) * 4u), "v"((unsigned)(val)), "s"(flag) : "memory"); } while (0)
    static_assert(U >= 2, "batch");
    u32x4 va[U], vb[U]; unsigned oa[U], ob[U]; bool wa[U], wb[U];
    int cc, pc = -1;
    unsigned tn = 0;
    int nstamp = 0; (void)nstamp;
    {
        LOCATE(t); cc = c;
        FETCH_ASYNC(tn);
#pragma unroll
        for (int u = 0; u < U; ++u) { unsigned sf; PIECE(t, u, sf, oa[u], wa[u]); LD(va[u], wa[u], sf); }
        WAIT_V(0);
        asm volatile("" : "+v"(tn));
        tn = __builtin_amdgcn_readfirstlane(tn) - tbasew;
    }
#define BATCH(cv, co, cw, nv, no, nw) \
        const bool nlive = tn < nbt; \
        unsigned tnn; int nc = 0; \
        if (nlive) { LOCATE(tn); nc = c; } \
        unsigned sf[U]; \
        _Pragma("unroll") for (int u = 0; u < U; ++u) { PIECE(tn, u, sf[u], no[u], nw[u]); if (!nlive) sf[u] = 0xffffff00u; } \
        if constexpr (CFG_PMODE == 0) { \
        _Pragma("unroll") for (int u = 0; u < U; ++u) { \
            LD(nv[u], nw[u], sf[u]); \
            if (u == 0) { FETCH_ASYNC2(tnn, nlive ? 1u : 0u); WAIT_V(2); } else if (u == 1) { WAIT_V(2); } else { WAIT_V(1); } \
            ST(cv[u], cw[u], co[u]); \
            if (u == 0) FLAG_ADD(pc >= 0 ? pc : cc, pc >= 0 ? 1u : 0u);       \
        } } else { \
              \
            _Pragma("unroll") for (int u = 0; u < U; ++u) LD(nv[u], nw[u], sf[u]); \
            FETCH_ASYNC2(tnn, nlive ? 1u : 0u); \
            WAIT_VU2(); \
            _Pragma("unroll") for (int u = 0; u < U; ++u) ST(cv[u], cw[u], co[u]); \
            FLAG_ADD(pc >= 0 ? pc : cc, pc >= 0 ? 1u : 0u); \
        } \
        pc = cc; \
        if constexpr (CFG_PMODE == 1) WAIT_VU1(); \
        asm volatile("" : "+v"(tnn));        \
        if (!nlive) break; \
        cc = nc; tn = __builtin_amdgcn_readfirstlane(tnn) - tbasew;
    for (;;) {
        { BATCH(va, oa, wa, vb, ob, wb) }
        { BATCH(vb, ob, wb, va, oa, wa) }
    }
#undef BATCH
#undef LD
#undef ST
    WAIT_V(0);
    FLAG_ADD(pc, 1u);
#undef FLAG_ADD
#undef LOCATE
#undef PIECE
#undef PLOAD
#undef FETCH
#undef FETCH_ASYNC
#undef FETCH_ASYNC2
    (void)nch;
}

__device__ __forceinline__ void signal_done(const Params& P, int crank) {
    WAIT_V(0);
    BAR();
    if (threadIdx.x == 0) __hip_atomic_fetch_add((unsigned*)(P.ws[crank] + WS_DONE), 1u, __ATOMIC_RELAXED, __HIP_MEMORY_SCOPE_SYSTEM);
}

__global__ __launch_bounds__(512, 2) void ag_kernel(Params P) {
    extern __shared__ __attribute__((aligned(16))) unsigned char shm[];
    LAS unsigned char* lds = (LAS unsigned char*)shm;
    const int tid = threadIdx.x, lane = tid & 63;
    const int rank = P.rank, cls = rank >> 2, idx = rank & 3;
    char* myws = P.ws[rank];
    const int b = blockIdx.x;
    volatile LAS unsigned* cw = (volatile LAS unsigned*)(lds + LDS_CTL + 16);
    unsigned call;
    {
        if (tid == 0) *cw = __hip_atomic_fetch_add((unsigned*)(myws + WS_CTR) + b * 16, 1u, __ATOMIC_RELAXED, __HIP_MEMORY_SCOPE_AGENT) + 1u;
        __syncthreads();
        call = __builtin_amdgcn_readfirstlane(*cw);
    }
    STAMP(0);
    const size_t par = WS_STAGE + (size_t)(call & 1u) * PAR_BYTES;
    const char* xs = myws + par;
    const unsigned* flags = (const unsigned*)(myws + WS_FLAG);
    const unsigned needv = DBG(4) ? 0u : call;
    if (b < NPUSH) {
        if (!DBG(1)) push_role(lds, P, call, b);
    } else if (b >= GEMM0 && !DBG(8)) {
        const int g = b - GEMM0;
        constexpr int NA = 4 * RBA, NB = 4 * RBB;
        const int x = g & 7, s = g >> 3;
        const int a0 = x * NA / 8, a1 = (x + 1) * NA / 8, b0 = x * NB / 8, b1 = (x + 1) * NB / 8;
        const int na = a1 - a0;
        const int type = s < na ? 0 : 1;
        int t = type == 0 ? a0 + s : b0 + (s - na);
        if (type == 0) {
            constexpr int NLOC = 4 * (16 / NU), PERL = NLOC / 8;
            if (NA % 8 == 0 && NLOC % 8 == 0) t = s < PERL ? x * PERL + s : NLOC + x * (NA / 8 - PERL) + (s - PERL);
        }
        if (type == 0 || s - na < b1 - b0) {
            const int rb = t >> 2, cb = t & 3;
            TileDesc d;
            const int crank = (1 - cls) * 4 + cb;
            d.flag = flags; d.call1 = 0u; d.pbase = 0; d.anywait = false;
#pragma unroll
            for (int u = 0; u < NU; ++u) {
                int ui = rb * NU + u;
                const int nun = type == 0 ? UA : UB;
                const bool valid = ui < nun;
                if (!valid) ui = nun - 1;
                int src, su; const char* xp;
                const int nown = type == 0 ? 16 : U16;
                if (ui < nown) { src = rank; su = ui; xp = P.x + (size_t)ui * UNITB; }
                else if (ui < nown + 3 * U1) { const int sl = (ui - nown) / U1; su = (ui - nown) % U1; src = cls * 4 + ((idx + 1 + sl) & 3); xp = xs + (size_t)(sl * U16 + su) * UNITB; }
                else { int sl = (ui - nown - 3 * U1) / V16; const int jj = (ui - nown - 3 * U1) % V16; if (type == 1 && sl >= cb) ++sl;
                       src = (1 - cls) * 4 + sl; su = U16 + jj; xp = xs + (size_t)(3 * U16 + sl * V16 + jj) * UNITB; }
                d.xa[u] = xp;
                char* ob = type == 0 ? P.out[rank] + cb * 512 : P.out[crank] + idx * 512;
                d.out[u] = valid ? ob + (size_t)(src * MSH + su * 64) * WPITCH : nullptr;
                if (src != rank) { d.anywait = true; if (lane / (64 / (NU + 1)) == u) { d.flag = flags + src * FROW; d.call1 = needv; d.pbase = (src >> 2) == cls ? PBASE_S1 : PBASE_X; } }
            }
            d.throttle = type == 1;
            if (type == 0) { d.wb = P.w + cb * 512; d.wpitch = WPITCH; d.wbytes = KDIM * WPITCH - cb * 512; }
            else { d.wb = xs + XS_BYTES + (size_t)cb * WSQ_BYTES; d.wpitch = QPITCH; d.wbytes = (unsigned)WSQ_BYTES; d.anywait = true;
                   if (lane / (64 / (NU + 1)) == NU) { d.flag = flags + crank * FROW; d.call1 = needv; d.pbase = PBASE_X; } }
            gemm_tile(lds, d, P, call);
            if (type == 1) signal_done(P, crank);
        }
    }
    STAMP(3);
    if (U2 > 0 && !DBG(8)) {
        const int x = b & 7;
        for (;;) {
            __syncthreads();
            if (tid == 0) *cw = __hip_atomic_fetch_add((unsigned*)(myws + WS_QUEUE) + x * 32, 1u, __ATOMIC_RELAXED, __HIP_MEMORY_SCOPE_AGENT) - (call - 1u) * (unsigned)(NT2X + NWG / 8);
            __syncthreads();
            const unsigned tk = __builtin_amdgcn_readfirstlane(*cw);
            if (tk >= (unsigned)NT2X) break;
            const int cbk = 2 * x + (int)(tk & 1u), un = (int)(tk >> 1);
            const int sl = un / (U2 > 0 ? U2 : 1), su = U1 + un % (U2 > 0 ? U2 : 1);
            const int src = cls * 4 + ((idx + 1 + sl) & 3);
            const char* xa = xs + (size_t)(sl * U16 + su) * UNITB;
            const unsigned* xflag = flags + src * FROW + NCH1;
            const size_t orow = (size_t)(src * MSH + su * 64) * WPITCH;
            const bool own = cbk < 8;
            const int cb = (cbk - 8) >> 1, hf = cbk & 1, crank = own ? rank : (1 - cls) * 4 + cb;
            small_tile(lds, xa, own ? P.w + cbk * 256 : xs + XS_BYTES + (size_t)cb * WSQ_BYTES + hf * 256, own ? WPITCH : QPITCH,
                       own ? KDIM * WPITCH - cbk * 256 : (unsigned)WSQ_BYTES - hf * 256, P.out[crank] + orow + (own ? cbk * 256 : idx * 512 + hf * 256),
                       xflag, needv, flags + crank * FROW, own ? 0u : needv, P, call);
            if (!own) signal_done(P, crank);
            STAMP(7);
        }
    }
    if (b == 0 && !DBG(2)) {
        if (tid < 64) {
            unsigned* tmo = (unsigned*)(myws + WS_TMO);
            const bool ok = wave_wait((const unsigned*)(myws + WS_DONE), call * (unsigned)(4 * RBB + 4 * 3 * U2 * 2), tmo, call);
            if (!ok && lane == 0) *(unsigned short*)(P.out[rank] + (size_t)rank * MSH * WPITCH) = 0x7fc0u;
        }
        STAMP(4);
    }
}

}

#define AG_DBG 0
extern "C" void kernel_launch(int rank, int nranks, ncclComm_t comm, hipStream_t stream,
                              void* const* d_in, const int* in_sizes, int n_in,
                              void* d_out, int out_size, const int* in_dtypes, int out_dtype,
                              void* const* d_out_all, void* const* d_ws_all, size_t ws_bytes) {
    using namespace agk;
    (void)comm; (void)in_sizes; (void)n_in; (void)out_size; (void)in_dtypes; (void)out_dtype; (void)d_out;
    if (nranks != NR || ws_bytes < WS_NEED || rank < 0 || rank >= NR) { fprintf(stderr, "unsupported configuration\n"); return; }
    static bool init[64] = {};
    int dev = 0; HIP_CHECK(hipGetDevice(&dev));
    if (!init[dev & 63]) {
        HIP_CHECK(hipFuncSetAttribute((const void*)ag_kernel, hipFuncAttributeMaxDynamicSharedMemorySize, LDS_BYTES));
        init[dev & 63] = true;
    }
    Params P;
    P.x = (const char*)d_in[0]; P.w = (const char*)d_in[1];
    for (int r = 0; r < NR; ++r) { P.out[r] = (char*)d_out_all[r]; P.ws[r] = (char*)d_ws_all[r]; }
    P.rank = rank; P.dbg = AG_DBG;
    hipLaunchKernelGGL(ag_kernel, dim3(NWG), dim3(512), LDS_BYTES, stream, P);
    HIP_CHECK(hipGetLastError());
}
```

```cpp
#include <hip/hip_runtime.h>
#include <rccl/rccl.h>
#include <cstdio>
#include <cstdint>
#include <type_traits>

#define HIP_CHECK(expr) do { hipError_t _e = (expr); if (_e != hipSuccess) { \
    fprintf(stderr, "HIP error %s at %s:%d\n", hipGetErrorString(_e), __FILE__, __LINE__); \
    return; } } while (0)

#define DBG(bit) 0

namespace agk {

#define LAS __attribute__((address_space(3)))
typedef short bf16x8 __attribute__((ext_vector_type(8)));
typedef short s16x4 __attribute__((ext_vector_type(4)));
typedef float f32x4 __attribute__((ext_vector_type(4)));
typedef unsigned u32x4 __attribute__((ext_vector_type(4)));
typedef unsigned u32x2 __attribute__((ext_vector_type(2)));
typedef __amdgpu_buffer_rsrc_t rsrc_t;

constexpr int NR = 8, KDIM = 8192, MSH = 1024, NSH = 1024;
constexpr unsigned XPITCH = KDIM * 2;
constexpr unsigned WPITCH = NSH * 2;
constexpr unsigned QPITCH = 512;
constexpr unsigned UNITB = 64u * XPITCH;

#define CFG_U16 11
#define CFG_U2 2
#define CFG_NU 4
#define CFG_NJ 16
#define CFG_EPI_INFLIGHT 0
#define CFG_PU 2
constexpr int U16 = CFG_U16, V16 = 16 - U16;
constexpr int U2 = CFG_U2, U1 = U16 - U2;
constexpr int NU = CFG_NU;
#define CFG_NJS CFG_NJ
constexpr int NJX = CFG_NJ, NJS = CFG_NJS;
constexpr int NJM = NJX < NJS ? NJX : NJS;
constexpr int NPUSH = 3 * NJS + 4 * NJX;
constexpr int UA = 16 + 3 * U1 + 4 * V16;
constexpr int UB = U16 + 3 * U1 + 3 * V16;
constexpr int RBA = (UA + NU - 1) / NU, RBB = (UB + NU - 1) / NU;
constexpr int NWG = 256;
constexpr int NGWG = 8 * ((4 * RBA + 7) / 8 + (4 * RBB + 7) / 8);
static_assert(NPUSH + NGWG <= NWG, "grid");
constexpr int GEMM0 = NWG - NGWG;
constexpr int NT2X = 3 * U2 * 2;

#define CFG_PA 1
#define CFG_PA_CNT 1
#define CFG_PA_TAIL 0
#define CFG_POLL_SLEEP 1
#define CFG_EPI_W2 0
#define CFG_ASYNC 0
#define CFG_PMODE 0
#define WAIT_VU2() WAIT_V(4)
#define WAIT_VU1() WAIT_V(3)
#define CFG_LG1 0
#define CFG_LG2 1
constexpr int NT = KDIM / 64;
constexpr int LG1 = CFG_LG1, LG2 = CFG_LG2;
__host__ __device__ constexpr int ch_body(int lg) { return (NT >> lg) - 1; }
__host__ __device__ constexpr int ch_count(int lg) { return ch_body(lg) + lg + 1; }
__host__ __device__ constexpr int ch_lg(int lg, int c) { return c < ch_body(lg) ? lg : (c - ch_body(lg) < lg ? lg - 1 - (c - ch_body(lg)) : 0); }
constexpr int NCH1 = ch_count(LG1), NCH2 = ch_count(LG2), NCH = NCH1 + NCH2;
__host__ __device__ constexpr int CH_LOGF(int c) { return c < NCH1 ? ch_lg(LG1, c) : ch_lg(LG2, c - NCH1); }
constexpr int ch_sum(int a, int b) { int s = 0; for (int i = a; i < b; ++i) s += 1 << CH_LOGF(i); return s; }
static_assert(ch_sum(0, NCH1) == NT && ch_sum(NCH1, NCH) == NT, "chunks");
constexpr unsigned long long ch_mask(int gen, int half) { unsigned long long m = 0; int s = 0;
    for (int i = gen ? NCH1 : 0; i < (gen ? NCH : NCH1); ++i) { if (s / 64 == half) m |= 1ull << (s % 64); s += 1 << CH_LOGF(i); } return m; }
constexpr unsigned long long CHM0 = ch_mask(0, 0), CHM1 = ch_mask(0, 1), CH2M0 = ch_mask(1, 0), CH2M1 = ch_mask(1, 1);
constexpr int FROW = 256;
static_assert(NCH <= FROW, "flags");

#define CFG_WAVETK 1
constexpr int PB = 8;
constexpr int PBASE_S1 = U1 * 8, PBASE_S2 = U2 * 8, PBASE_X = V16 * 8 + 32;
__host__ __device__ constexpr int nbatch(int pieces) { return (pieces + PB - 1) / PB; }
constexpr int nb_total(bool same) { int s = 0; for (int c = 0; c < NCH; ++c) { const int pb = c < NCH1 ? (same ? PBASE_S1 : PBASE_X) : (same ? PBASE_S2 : 0); s += nbatch(pb << CH_LOGF(c)); } return s; }
constexpr int NB_S = nb_total(true), NB_X = nb_total(false);

constexpr size_t WS_CTR = 0;
constexpr size_t WS_TMO = 16384;
constexpr size_t WS_DONE = 16384 + 256;
constexpr size_t WS_QUEUE = 16384 + 512;
constexpr size_t WS_TICK = 16384 + 2048;
constexpr size_t WS_FLAG = 32768;
constexpr size_t WS_DBG = 65536;
constexpr size_t WS_STAGE = 1u << 20;
constexpr size_t XS_BYTES = (size_t)(3 * U16 + 4 * V16) * UNITB;
constexpr size_t WSQ_BYTES = (size_t)KDIM * QPITCH;
constexpr size_t PAR_BYTES = XS_BYTES + 4 * WSQ_BYTES;
constexpr size_t WS_NEED = WS_STAGE + 2 * PAR_BYTES;
#define CFG_DEADLINE 100000000ull
constexpr unsigned long long DEADLINE = CFG_DEADLINE;

constexpr int LDS_STAGE = 2 * NU * 8192 + 65536;
constexpr int LDS_EPI = NU * 64 * 528;
constexpr int LDS_SMALL = 6 * 24576;
constexpr int cmax(int a, int b) { return a > b ? a : b; }
constexpr int LDS_CTL = cmax(cmax(LDS_STAGE, LDS_EPI), LDS_SMALL);
constexpr int LDS_BYTES = LDS_CTL + 256;

struct Params {
    const char* x; const char* w;
    char* out[NR]; char* ws[NR];
    int rank; int dbg;
};

__device__ __forceinline__ unsigned cvt_pk_bf16(float lo, float hi) { unsigned r; asm("v_cvt_pk_bf16_f32 %0, %1, %2" : "=v"(r) : "v"(lo), "v"(hi)); return r; }
__device__ __forceinline__ float gelu1(float y) {
    const float c2 = -2.0f * 0.7978845608028654f * 1.4426950408889634f;
    const float a = y * __builtin_fmaf(y * y, 0.044715f * c2, c2);
    return y * __builtin_amdgcn_rcpf(1.0f + __builtin_amdgcn_exp2f(a));
}
__device__ __forceinline__ rsrc_t mk_rsrc(const void* p, unsigned bytes) { return __builtin_amdgcn_make_buffer_rsrc((void*)p, 0, bytes, 0x00020000); }

#define WAIT_V_(n) asm volatile("s_waitcnt vmcnt(" #n ")" ::: "memory")
#define WAIT_V(n) WAIT_V_(n)
#define WAIT_L(n) asm volatile("s_waitcnt lgkmcnt(" #n ")" ::: "memory")
#define BAR() __builtin_amdgcn_s_barrier()
#define SCHED() __builtin_amdgcn_sched_barrier(0)
#define SYS_LOAD(p) __hip_atomic_load((p), __ATOMIC_RELAXED, __HIP_MEMORY_SCOPE_SYSTEM)
#define AGT_LOAD(p) __hip_atomic_load((p), __ATOMIC_RELAXED, __HIP_MEMORY_SCOPE_AGENT)
#define CFG_DMA_AUX 17
#define DMA(rs, ldsp, voff, soff) __builtin_amdgcn_raw_ptr_buffer_load_lds((rs), (LAS void*)(ldsp), 16, (voff), (soff), 0, CFG_DMA_AUX)
#define TRR(dst, base, off) asm volatile("ds_read_b64_tr_b16 %0, %1 offset:%2" : "=v"(dst) : "v"(base), "i"(off) : "memory")

#define STAMP(slot) do {} while (0)

__device__ __forceinline__ bool wave_wait(const unsigned* p, unsigned need, unsigned* tmo, unsigned call) {
    const unsigned long long t0 = __builtin_amdgcn_s_memrealtime();
    for (unsigned n = 1;; ++n) {
        const unsigned v = SYS_LOAD(p);
        if (__all((int)(v - need) >= 0 || need == 0u)) return true;
        if ((n & 15u) == 0u && (__builtin_amdgcn_readfirstlane(AGT_LOAD(tmo)) == call || __builtin_amdgcn_s_memrealtime() - t0 > DEADLINE)) {
            __hip_atomic_store(tmo, call, __ATOMIC_RELAXED, __HIP_MEMORY_SCOPE_AGENT);
            return false;
        }
        __builtin_amdgcn_s_sleep(CFG_POLL_SLEEP);
    }
}

template <int WN, int NS>
__device__ __forceinline__ int window_wait(const unsigned* p, unsigned need, unsigned* tmo, unsigned call) {
    const unsigned long long t0 = __builtin_amdgcn_s_memrealtime();
    for (unsigned n = 1;; ++n) {
        const unsigned v = SYS_LOAD(p);
        unsigned long long m = __ballot((int)(v - need) >= 0 || need == 0u), a = m;
#pragma unroll
        for (int s = 1; s < NS; ++s) a &= m >> (s * WN);
        a &= (1ull << WN) - 1ull;
        if (a & 1ull) return (int)__builtin_ctzll(~a);
        if ((n & 15u) == 0u && (__builtin_amdgcn_readfirstlane(AGT_LOAD(tmo)) == call || __builtin_amdgcn_s_memrealtime() - t0 > DEADLINE)) {
            __hip_atomic_store(tmo, call, __ATOMIC_RELAXED, __HIP_MEMORY_SCOPE_AGENT);
            return WN;
        }
        __builtin_amdgcn_s_sleep(CFG_POLL_SLEEP);
    }
}

__device__ __forceinline__ void store_out(char* o, u32x4 v) { asm volatile("global_store_dwordx4 %0, %1, off sc0 sc1\n\ts_nop 1" :: "v"(o), "v"(v) : "memory"); }

struct TileDesc {
    const char* xa[NU];
    const char* wb; unsigned wpitch, wbytes;
    char* out[NU];
    const unsigned* flag;
    unsigned call1;
    int pbase;
    bool anywait, throttle;
};

__device__ __forceinline__ void gemm_tile(LAS unsigned char* lds, const TileDesc& d, const Params& P, unsigned call) {
    constexpr int MT = NU * 2;
    constexpr int ABUF = NU * 8192;
    constexpr int BOFF = 2 * ABUF;
    const int tid = threadIdx.x, wid = __builtin_amdgcn_readfirstlane(tid >> 6), lane = tid & 63;
    const int wr = wid >> 2, wc = wid & 3, fr = lane & 15, fq = lane >> 4;
    unsigned* tmo = (unsigned*)(P.ws[P.rank] + WS_TMO);
    volatile LAS unsigned* ready = (volatile LAS unsigned*)(lds + LDS_CTL);
    const unsigned voffA = (unsigned)(8 * wid + (lane >> 3)) * XPITCH + (unsigned)(((lane & 7) ^ (((wid & 1) << 2) | (lane >> 4))) * 16);
    const unsigned fB = (unsigned)((lane >> 4) | (((wid >> 1) & 1) << 2));
    const unsigned voffB = (unsigned)(4 * wid + (lane >> 4)) * d.wpitch + ((((unsigned)(lane & 15) >> 1) ^ fB) * 32u + (unsigned)(lane & 1) * 16u);
    const unsigned ldsw = (unsigned)wid * 1024u;
    const unsigned aoff0 = (unsigned)(wr * (NU * 32) + fr) * 128u + (unsigned)((fq ^ (fr >> 1)) * 16);
    const unsigned aoff1 = aoff0 ^ 64u;
    const unsigned q = (unsigned)(fr >> 2), p = (unsigned)(fr & 3);
    const unsigned boff0 = (unsigned)(size_t)lds + BOFF + (unsigned)(8 * fq + q) * 256u + (((unsigned)(2 * wc) ^ (q | ((unsigned)(fq & 1) << 2))) * 32u) + p * 8u;
    const unsigned boff1 = boff0 ^ 32u;
    rsrc_t ra[NU];
#pragma unroll
    for (int u = 0; u < NU; ++u) ra[u] = mk_rsrc(d.xa[u], UNITB);
    const rsrc_t rb = mk_rsrc(d.wb, d.wbytes);
    const unsigned wstep = __builtin_amdgcn_readfirstlane(64u * d.wpitch), whalf = wstep >> 1;

    f32x4 acc[2][MT][2];
#pragma unroll
    for (int b = 0; b < 2; ++b)
#pragma unroll
        for (int m = 0; m < MT; ++m)
#pragma unroll
            for (int n = 0; n < 2; ++n) acc[b][m][n] = (f32x4){0.f, 0.f, 0.f, 0.f};
    bf16x8 af[MT][2], bfr[2][2]; s16x4 tb[8];

#define STAGE_A(buf, t) do { _Pragma("unroll") for (int _i = 0; _i < NU; ++_i) DMA(ra[_i], lds + (buf) * ABUF + _i * 8192 + ldsw, voffA, (unsigned)(t) * 128u); } while (0)
#define STAGE_B(buf, h, t) do { _Pragma("unroll") for (int _i = 0; _i < 2; ++_i) \
        DMA(rb, lds + BOFF + ((buf) * 2 + (h)) * 16384 + _i * 8192 + ldsw, voffB, (unsigned)(t) * wstep + _i * whalf + (h) * 256u); } while (0)
#define LDA(buf) do { _Pragma("unroll") for (int m = 0; m < MT; ++m) { \
        af[m][0] = *(const LAS bf16x8*)(lds + (buf) * ABUF + aoff0 + m * 2048); \
        af[m][1] = *(const LAS bf16x8*)(lds + (buf) * ABUF + aoff1 + m * 2048); } } while (0)
#define LDB(buf, h) do { \
        TRR(tb[0], boff0, ((buf) * 2 + (h)) * 16384); TRR(tb[1], boff0, ((buf) * 2 + (h)) * 16384 + 1024); \
        TRR(tb[2], boff1, ((buf) * 2 + (h)) * 16384); TRR(tb[3], boff1, ((buf) * 2 + (h)) * 16384 + 1024); \
        TRR(tb[4], boff0, ((buf) * 2 + (h)) * 16384 + 8192); TRR(tb[5], boff0, ((buf) * 2 + (h)) * 16384 + 8192 + 1024); \
        TRR(tb[6], boff1, ((buf) * 2 + (h)) * 16384 + 8192); TRR(tb[7], boff1, ((buf) * 2 + (h)) * 16384 + 8192 + 1024); } while (0)
#define WAITB() do { asm volatile("s_waitcnt lgkmcnt(0)" : "+v"(tb[0]), "+v"(tb[1]), "+v"(tb[2]), "+v"(tb[3]), "+v"(tb[4]), "+v"(tb[5]), "+v"(tb[6]), "+v"(tb[7]) :: "memory"); \
        bfr[0][0] = __builtin_shufflevector(tb[0], tb[1], 0, 1, 2, 3, 4, 5, 6, 7); bfr[1][0] = __builtin_shufflevector(tb[2], tb[3], 0, 1, 2, 3, 4, 5, 6, 7); \
        bfr[0][1] = __builtin_shufflevector(tb[4], tb[5], 0, 1, 2, 3, 4, 5, 6, 7); bfr[1][1] = __builtin_shufflevector(tb[6], tb[7], 0, 1, 2, 3, 4, 5, 6, 7); } while (0)
#define MMA(bj) do { __builtin_amdgcn_s_setprio(1); \
        _Pragma("unroll") for (int kk = 0; kk < 2; ++kk) _Pragma("unroll") for (int m = 0; m < MT; ++m) _Pragma("unroll") for (int n = 0; n < 2; ++n) \
            acc[bj][m][n] = __builtin_amdgcn_mfma_f32_16x16x32_bf16(bfr[n][kk], af[m][kk], acc[bj][m][n], 0, 0, 0); \
        __builtin_amdgcn_s_setprio(0); } while (0)
#define WAITN() do { if constexpr (NU == 3) { WAIT_V(7); } else if constexpr (NU == 2) { WAIT_V(6); } else { WAIT_V(8); } } while (0)
    constexpr int NS = NU + 1, WN = 64 / NS;
    int cnext = 0, known = 0;
#define POLL() do { const int ck_ = cnext + lane % WN; const bool w_ = ck_ < NCH1 && lane < NS * WN; const int cc_ = w_ ? ck_ : 0; \
        known = cnext + window_wait<WN, NS>(d.flag + cc_, w_ ? d.call1 * (unsigned)nbatch(d.pbase << CH_LOGF(cc_)) : 0u, tmo, call); } while (0)
#define GATE() do { \
        if (known <= cnext) { \
            if (wid == 0) { POLL(); if (lane == 0) *ready = (unsigned)known; } \
            else { unsigned r_; while ((r_ = *ready) <= (unsigned)cnext) __builtin_amdgcn_s_sleep(1); known = (int)__builtin_amdgcn_readfirstlane(r_); } \
        } \
        if ((cnext & 3) == 0) STAMP(8 + (cnext >> 2)); \
        ++cnext; } while (0)
#define CHECK(tt) do { if (d.anywait && (tt) < NT && ((((tt) < 64 ? CHM0 : CHM1) >> ((tt) & 63)) & 1ull)) GATE(); } while (0)

    if (d.anywait) {
        constexpr int C01 = LG1 == 0 ? 2 : 1;
        cnext = C01 - 1;
        if (wid == 0) { if (C01 == 2) { cnext = 0; POLL(); cnext = 1; } if (known <= cnext) POLL(); if (lane == 0) *ready = (unsigned)known; }
        WAIT_L(0); BAR();
        known = (int)__builtin_amdgcn_readfirstlane(*ready);
        cnext = C01;
    }
    STAMP(1);
    STAGE_A(0, 0); STAGE_B(0, 0, 0); STAGE_B(0, 1, 0);
    STAGE_A(1, 1); STAGE_B(1, 0, 1);
    if (wr == 1) BAR();
    if constexpr (NU == 3) { WAIT_V(5); } else if constexpr (NU == 2) { WAIT_V(4); } else { WAIT_V(6); }
    BAR(); BAR();
#define KTILE(b, t) do { \
        LDB(b, 0); LDA(b); \
        STAGE_B((b) ^ 1, 1, min((t) + 1, NT - 1)); \
        WAITN(); WAITB(); BAR(); SCHED(); MMA(0); SCHED(); BAR(); \
        CHECK((t) + 2); \
        LDB(b, 1); \
        STAGE_A(b, min((t) + 2, NT - 1)); STAGE_B(b, 0, min((t) + 2, NT - 1)); \
        WAITN(); WAITB(); BAR(); SCHED(); MMA(1); SCHED(); BAR(); } while (0)
    for (int t = 0; t < NT; t += 2) {
        KTILE(0, t);
        KTILE(1, t + 1);
    }
    WAIT_V(0);
    if (wr == 0) BAR();
    BAR();
    STAMP(2);
    constexpr int EP = 528;
    const bool bad = AGT_LOAD(tmo) == call;
#pragma unroll
    for (int bj = 0; bj < 2; ++bj)
#pragma unroll
        for (int m = 0; m < MT; ++m)
#pragma unroll
            for (int n = 0; n < 2; ++n) {
                const f32x4 v = acc[bj][m][n];
                u32x2 w; w.x = cvt_pk_bf16(gelu1(v[0]), gelu1(v[1])); w.y = cvt_pk_bf16(gelu1(v[2]), gelu1(v[3]));
                *(LAS u32x2*)(lds + (wr * (NU * 32) + m * 16 + fr) * EP + (bj * 128 + wc * 32 + n * 16 + fq * 4) * 2) = w;
            }
    WAIT_L(0); BAR();
#pragma unroll
    for (int u = 0; u < NU; ++u) {
        if (d.out[u] != nullptr) {
#pragma unroll
            for (int i = 0; i < 4; ++i) {
                const int r = (i * 8 + wid) * 2 + (lane >> 5);
                u32x4 v = *(const LAS u32x4*)(lds + (u * 64 + r) * EP + (lane & 31) * 16);
                if (bad) v = (u32x4){0x7fc07fc0u, 0x7fc07fc0u, 0x7fc07fc0u, 0x7fc07fc0u};
                store_out(d.out[u] + (size_t)r * WPITCH + (lane & 31) * 16, v);
                if (d.throttle) { if (wid < CFG_EPI_W2) WAIT_V(1); else WAIT_V(0); }
            }
        }
    }
#undef STAGE_A
#undef STAGE_B
#undef LDA
#undef LDB
#undef MMA
#undef KTILE
#undef WAITB
#undef WAITN
#undef GATE
#undef CHECK
#undef POLL
}

__device__ __forceinline__ void small_tile(LAS unsigned char* lds, const char* xa, const char* wb, unsigned wpitch, unsigned wbytes, char* out,
                                           const unsigned* xflag, unsigned xcall, const unsigned* wflag, unsigned wcall, const Params& P, unsigned call) {
    const int tid = threadIdx.x, wid = __builtin_amdgcn_readfirstlane(tid >> 6), lane = tid & 63;
    const int g = wid >> 2, wq = wid & 3, wm = wq >> 1, wn = wq & 1, fr = lane & 15, fq = lane >> 4;
    unsigned* tmo = (unsigned*)(P.ws[P.rank] + WS_TMO);
    volatile LAS unsigned* ready = (volatile LAS unsigned*)(lds + LDS_CTL + 32) + g;
    const unsigned voffA = (unsigned)(8 * wq + (lane >> 3)) * XPITCH + (unsigned)(((lane & 7) ^ (((wq & 1) << 2) | (lane >> 4))) * 16);
    const unsigned fB = (unsigned)((lane >> 4) | (((wq >> 1) & 1) << 2));
    const unsigned voffB = (unsigned)(4 * wq + (lane >> 4)) * wpitch + ((((unsigned)(lane & 15) >> 1) ^ fB) * 32u + (unsigned)(lane & 1) * 16u);
    const unsigned gbase = (unsigned)g * (3u * 24576u);
    const unsigned ldsw = gbase + (unsigned)wq * 1024u;
    const unsigned aoff0 = gbase + (unsigned)(wm * 32 + fr) * 128u + (unsigned)((fq ^ (fr >> 1)) * 16);
    const unsigned aoff1 = aoff0 ^ 64u;
    const unsigned q = (unsigned)(fr >> 2), p = (unsigned)(fr & 3);
    const unsigned boff = (unsigned)(size_t)lds + gbase + 8192u + (unsigned)(8 * fq + q) * 256u + (((unsigned)(4 * wn) ^ (q | ((unsigned)(fq & 1) << 2))) * 32u) + p * 8u;
    const unsigned bo0 = boff, bo1 = boff ^ 32u, bo2 = boff ^ 64u, bo3 = boff ^ 96u;
    const rsrc_t ra = mk_rsrc(xa, UNITB), rb = mk_rsrc(wb, wbytes);
    const unsigned wstep = __builtin_amdgcn_readfirstlane(64u * wpitch), wq16 = wstep >> 2;
    f32x4 acc[2][4];
#pragma unroll
    for (int m = 0; m < 2; ++m)
#pragma unroll
        for (int n = 0; n < 4; ++n) acc[m][n] = (f32x4){0.f, 0.f, 0.f, 0.f};
#define STAGE(s, t) do { DMA(ra, lds + (s) * 24576 + ldsw, voffA, (unsigned)(t) * 128u); DMA(ra, lds + (s) * 24576 + 4096 + ldsw, voffA, (unsigned)(t) * 128u + 32u * XPITCH); \
        _Pragma("unroll") for (int _j = 0; _j < 4; ++_j) DMA(rb, lds + (s) * 24576 + 8192 + _j * 4096 + ldsw, voffB, (unsigned)(t) * wstep + _j * wq16); } while (0)
    int known = 0;
#define CIDX(tt) ((tt) < 64 ? __builtin_popcountll(CH2M0 & ((2ull << (tt)) - 1ull)) - 1 : __builtin_popcountll(CH2M0) + __builtin_popcountll(CH2M1 & ((2ull << ((tt) - 64)) - 1ull)) - 1)
#define POLL(ci) do { const int ck_ = (ci) + lane; const bool w_ = ck_ < NCH2 && lane < 32; const int cc_ = w_ ? ck_ : 0; \
        known = (ci) + window_wait<32, 1>(xflag + cc_, w_ ? xcall * (unsigned)nbatch(PBASE_S2 << CH_LOGF(NCH1 + cc_)) : 0u, tmo, call); } while (0)
#define CHECK(tt) do { if ((tt) < NT) { const int ci_ = CIDX(tt); if (known <= ci_) { \
            if (wq == 0) { POLL(ci_); if (lane == 0) *ready = (unsigned)known; } \
            else { unsigned r_; while ((r_ = *ready) <= (unsigned)ci_) __builtin_amdgcn_s_sleep(1); known = (int)__builtin_amdgcn_readfirstlane(r_); } } } } while (0)
    BAR();
    if (wid == 0) {
        for (int c0 = 0; c0 < NCH1; c0 += 64) { const int lc = c0 + lane < NCH1 ? c0 + lane : 0;
            wave_wait(wflag + lc, c0 + lane < NCH1 ? wcall * (unsigned)nbatch(PBASE_X << CH_LOGF(lc)) : 0u, tmo, call); }
        known = 0; while (known <= CIDX(3)) POLL(known);
        if (lane == 0) { ready[0] = (unsigned)known; ready[1] = (unsigned)known; }
    }
    WAIT_L(0); BAR();
    known = (int)__builtin_amdgcn_readfirstlane(*ready);
    STAMP(5);
    STAGE(0, g); STAGE(1, g + 2);
    WAIT_V(6); BAR();
    if (g == 1) BAR();
    s16x4 tb[16]; bf16x8 af[2][2];
#define STEP(s, t) do { \
        CHECK((t) + 4); \
        TRR(tb[0], bo0, (s) * 24576); TRR(tb[1], bo0, (s) * 24576 + 1024); TRR(tb[2], bo1, (s) * 24576); TRR(tb[3], bo1, (s) * 24576 + 1024); \
        TRR(tb[4], bo2, (s) * 24576); TRR(tb[5], bo2, (s) * 24576 + 1024); TRR(tb[6], bo3, (s) * 24576); TRR(tb[7], bo3, (s) * 24576 + 1024); \
        TRR(tb[8], bo0, (s) * 24576 + 8192); TRR(tb[9], bo0, (s) * 24576 + 9216); TRR(tb[10], bo1, (s) * 24576 + 8192); TRR(tb[11], bo1, (s) * 24576 + 9216); \
        TRR(tb[12], bo2, (s) * 24576 + 8192); TRR(tb[13], bo2, (s) * 24576 + 9216); TRR(tb[14], bo3, (s) * 24576 + 8192); TRR(tb[15], bo3, (s) * 24576 + 9216); \
        _Pragma("unroll") for (int m = 0; m < 2; ++m) { af[m][0] = *(const LAS bf16x8*)(lds + (s) * 24576 + aoff0 + m * 2048); af[m][1] = *(const LAS bf16x8*)(lds + (s) * 24576 + aoff1 + m * 2048); } \
        STAGE(((s) + 2) % 3, min((t) + 4, NT - 2 + g)); \
        asm volatile("s_waitcnt lgkmcnt(0)" : "+v"(tb[0]), "+v"(tb[1]), "+v"(tb[2]), "+v"(tb[3]), "+v"(tb[4]), "+v"(tb[5]), "+v"(tb[6]), "+v"(tb[7]), \
                     "+v"(tb[8]), "+v"(tb[9]), "+v"(tb[10]), "+v"(tb[11]), "+v"(tb[12]), "+v"(tb[13]), "+v"(tb[14]), "+v"(tb[15]) :: "memory"); \
        BAR(); SCHED(); \
        __builtin_amdgcn_s_setprio(1); \
        _Pragma("unroll") for (int kk = 0; kk < 2; ++kk) _Pragma("unroll") for (int n = 0; n < 4; ++n) { \
            const bf16x8 b_ = __builtin_shufflevector(tb[kk * 8 + n * 2], tb[kk * 8 + n * 2 + 1], 0, 1, 2, 3, 4, 5, 6, 7); \
            _Pragma("unroll") for (int m = 0; m < 2; ++m) acc[m][n] = __builtin_amdgcn_mfma_f32_16x16x32_bf16(b_, af[m][kk], acc[m][n], 0, 0, 0); } \
        __builtin_amdgcn_s_setprio(0); \
        SCHED(); WAIT_V(6); BAR(); } while (0)
    for (int i = 0; i < 63; i += 3) {
        STEP(0, g + 2 * i); STEP(1, g + 2 * i + 2); STEP(2, g + 2 * i + 4);
    }
    STEP(0, g + 126);
    WAIT_V(0);
    if (g == 0) BAR();
    BAR();
    STAMP(6);
    constexpr int EP = 272;
    const bool bad = AGT_LOAD(tmo) == call;
    if (g == 1) {
#pragma unroll
        for (int m = 0; m < 2; ++m)
#pragma unroll
            for (int n = 0; n < 4; ++n) *(LAS f32x4*)(lds + 32768 + ((wq * 8 + m * 4 + n) * 64 + lane) * 16) = acc[m][n];
    }
    WAIT_L(0); BAR();
    if (g == 0) {
#pragma unroll
        for (int m = 0; m < 2; ++m)
#pragma unroll
            for (int n = 0; n < 4; ++n) {
                const f32x4 v = acc[m][n] + *(const LAS f32x4*)(lds + 32768 + ((wq * 8 + m * 4 + n) * 64 + lane) * 16);
                u32x2 w; w.x = cvt_pk_bf16(gelu1(v[0]), gelu1(v[1])); w.y = cvt_pk_bf16(gelu1(v[2]), gelu1(v[3]));
                *(LAS u32x2*)(lds + (wm * 32 + m * 16 + fr) * EP + (wn * 64 + n * 16 + fq * 4) * 2) = w;
            }
    }
    WAIT_L(0); BAR();
#pragma unroll
    for (int i = 0; i < 2; ++i) {
        const int r = (i * 8 + wid) * 4 + (lane >> 4);
        u32x4 v = *(const LAS u32x4*)(lds + r * EP + (lane & 15) * 16);
        if (bad) v = (u32x4){0x7fc07fc0u, 0x7fc07fc0u, 0x7fc07fc0u, 0x7fc07fc0u};
        store_out(out + (size_t)r * WPITCH + (lane & 15) * 16, v);
    }
#undef STAGE
#undef CIDX
#undef CHECK
#undef POLL
#undef STEP
}

__device__ __forceinline__ void push_role(LAS unsigned char* lds, const Params& P, unsigned call, int pw) {
    const int tid = threadIdx.x, wid = __builtin_amdgcn_readfirstlane(tid >> 6), lane = tid & 63;
    const int rank = P.rank, cls = rank >> 2, idx = rank & 3;
    int di, j;
    if (pw < 7 * NJM) { di = pw % 7; j = pw / 7; }
    else if (NJS > NJX) { const int r = pw - 7 * NJM; di = r % 3; j = NJM + r / 3; }
    else { const int r = pw - 7 * NJM; di = 3 + r % 4; j = NJM + r / 4; }
    const bool same = di < 3;
    const int q = same ? (cls * 4 + ((idx + 1 + di) & 3)) : ((1 - cls) * 4 + (di - 3));
    const int qidx = q & 3;
    const size_t par = WS_STAGE + (size_t)(call & 1u) * PAR_BYTES;
    const int slot = same ? ((idx - qidx - 1) & 3) : idx;
    const size_t xdst_off = par + (same ? (size_t)slot * U16 * UNITB : (size_t)(3 * U16 + slot * V16) * UNITB);
    const size_t wdst_off = par + XS_BYTES + (size_t)slot * WSQ_BYTES;
    const unsigned xbytes = (unsigned)(same ? U16 : V16) * UNITB;
    const char* xsrc = P.x + (same ? 0 : (size_t)U16 * UNITB);
    const char* wsrc = P.w + qidx * 512;
    char* wsq = P.ws[q];
    unsigned* flag = (unsigned*)(wsq + WS_FLAG) + rank * FROW;
    const unsigned nbt = same ? NB_S : NB_X;
    const unsigned njd = same ? NJS : NJX;
    unsigned* tick = (unsigned*)(P.ws[rank] + WS_TICK) + di * 32;
    const unsigned tbase = (call - 1u) * nbt - 3u * njd;
    volatile LAS unsigned* tk = (volatile LAS unsigned*)lds;
    LAS unsigned* cnt = (LAS unsigned*)(lds + 32);
    volatile LAS unsigned* done = (volatile LAS unsigned*)(lds + 64);
    if (tid < 8) { cnt[tid] = 0u; tk[tid] = tid < 3 ? (((unsigned)tid + 1u) << 20 | ((unsigned)j + (unsigned)tid * njd)) : 0u; }
    if (tid == 8) *done = 0u;
    __syncthreads();
    int c = 0, kt0 = 0, row0 = 0, pbase = same ? PBASE_S1 : PBASE_X, nx0 = same ? PBASE_S1 : V16 * 8;
    unsigned cb0 = 0;
#define LOCATE(tt) do { for (;;) { const unsigned nbc_ = (unsigned)nbatch(pbase << CH_LOGF(c)); if ((tt) < cb0 + nbc_) break; \
            cb0 += nbc_; kt0 += 1 << CH_LOGF(c); ++c; \
            if (c == NCH1) { kt0 = 0; row0 = U1 * 64; pbase = PBASE_S2; nx0 = PBASE_S2; } } } while (0)
#define PIECE(tt, soff, doff, w) do { const int lg_ = CH_LOGF(c), ptot_ = pbase << lg_, nx_ = nx0 << lg_; const int i_ = (int)((tt) - cb0) * PB + wid; \
            soff = 0xffffff00u; doff = 0xffffff00u; w = false; \
            if (i_ < nx_) { const unsigned o_ = (unsigned)i_ * 1024u + (unsigned)lane * 16u; \
                const unsigned row_ = (unsigned)row0 + (o_ >> (7 + lg_)), col_ = o_ & ((128u << lg_) - 1u); soff = doff = row_ * XPITCH + (unsigned)kt0 * 128u + col_; } \
            else if (i_ < ptot_) { const unsigned k_ = (unsigned)kt0 * 64u + (unsigned)(i_ - nx_) * 2u + (unsigned)(lane >> 5); \
                soff = k_ * WPITCH + (unsigned)(lane & 31) * 16u; doff = k_ * QPITCH + (unsigned)(lane & 31) * 16u; w = true; } } while (0)
    auto desc = [](const void* p, unsigned bytes) { const unsigned long long a = (unsigned long long)p; u32x4 r;
        r.x = __builtin_amdgcn_readfirstlane((unsigned)a); r.y = __builtin_amdgcn_readfirstlane((unsigned)(a >> 32) & 0xffffu); r.z = __builtin_amdgcn_readfirstlane(bytes); r.w = 0x00020000u; return r; };
    const u32x4 dxs = desc(xsrc, xbytes), dxd = desc(wsq + xdst_off, xbytes), dws = desc(wsrc, (unsigned)KDIM * WPITCH - (unsigned)qidx * 512u), dwd = desc(wsq + wdst_off, (unsigned)WSQ_BYTES);
#define LD(dst, w, off) asm volatile("buffer_load_dwordx4 %0, %1, %2, 0 offen" : "=v"(dst) : "v"(off), "s"((w) ? dws : dxs) : "memory")
#define ST(src, w, off) asm volatile("buffer_store_dwordx4 %0, %1, %2, 0 offen sc0 sc1\n\ts_nop 1" :: "v"(src), "v"(off), "s"((w) ? dwd : dxd) : "memory")
#define FETCH_ASYNC(dst) do { unsigned long long ex_; asm volatile("s_mov_b64 %1, exec\n\ts_mov_b64 exec, 1\n\tglobal_atomic_add %0, %2, %3, %4 sc0\n\ts_mov_b64 exec, %1" \
        : "=&v"(dst), "=&s"(ex_) : "v"(0u), "v"(1u), "s"(tick) : "memory"); } while (0)
#define FLAG_ADD(idx) do { unsigned long long ex_; asm volatile("s_mov_b64 %0, exec\n\ts_mov_b64 exec, 1\n\tglobal_atomic_add %1, %2, %3 sc1\n\ts_mov_b64 exec, %0" \
        : "=&s"(ex_) : "v"((unsigned)(idx) * 4u), "v"(1u), "s"(flag) : "memory"); } while (0)
#define COMPLETE(k, ch) do { unsigned old_ = 0; if (lane == 0) old_ = __hip_atomic_fetch_add(cnt + ((k) & 7u), 1u, __ATOMIC_RELAXED, __HIP_MEMORY_SCOPE_WORKGROUP); \
        if (__builtin_amdgcn_readfirstlane(old_) == 7u) { FLAG_ADD(ch); if (lane == 0) { cnt[(k) & 7u] = 0u; *done = (k) + 1u; } } } while (0)
    auto loop = [&](auto leader) {
        constexpr bool LEADER = decltype(leader)::value;
        u32x4 va, vb; unsigned oa, ob; bool wa, wb;
        unsigned tfa = 0, tfb = 0;
        unsigned it = 0;
        int cc, pc = 0;
        { LOCATE((unsigned)j); cc = c; unsigned sf; PIECE((unsigned)j, sf, oa, wa); LD(va, wa, sf); }
#define ITER(cv, co, cw, nv, no, nw, tfnew, tfold) \
        unsigned w_; while (((w_ = tk[(it + 1u) & 7u]) >> 20) != it + 2u) __builtin_amdgcn_s_sleep(1); \
        const unsigned tn = __builtin_amdgcn_readfirstlane(w_) & 0xfffffu; \
        const bool nlive = tn < nbt; \
        int nc = 0; \
        if (nlive) { LOCATE(tn); nc = c; } \
        unsigned sf; PIECE(tn, sf, no, nw); if (!nlive) sf = 0xffffff00u; \
        LD(nv, nw, sf); \
        if constexpr (LEADER) { WAIT_V(1); asm volatile("" : "+v"(tfold)); FETCH_ASYNC(tfnew); \
            if (it >= 1u) { while (*done + 3u < it) __builtin_amdgcn_s_sleep(1); \
                const unsigned tv_ = __builtin_amdgcn_readfirstlane(tfold) - tbase; \
                if (lane == 0) tk[(it + 2u) & 7u] = (it + 3u) << 20 | (tv_ < 0xfffffu ? tv_ : 0xfffffu); } \
        } else { WAIT_V(1); } \
        if (it >= 1u) COMPLETE(it - 1u, pc); \
        ST(cv, cw, co); \
        pc = cc; cc = nc; ++it; \
        if (!nlive) break;
        for (;;) {
            { ITER(va, oa, wa, vb, ob, wb, tfa, tfb) }
            { ITER(vb, ob, wb, va, oa, wa, tfb, tfa) }
        }
#undef ITER
        WAIT_V(0);
        asm volatile("" :: "v"(va), "v"(vb), "v"(tfa), "v"(tfb));
        COMPLETE(it - 1u, pc);
    };
    if (wid == 0) loop(std::true_type{}); else loop(std::false_type{});
#undef COMPLETE
#undef FLAG_ADD
#undef FETCH_ASYNC
#undef LD
#undef ST
#undef LOCATE
#undef PIECE
}

__device__ __forceinline__ void signal_done(const Params& P, int crank) {
    WAIT_V(0);
    BAR();
    if (threadIdx.x == 0) __hip_atomic_fetch_add((unsigned*)(P.ws[crank] + WS_DONE), 1u, __ATOMIC_RELAXED, __HIP_MEMORY_SCOPE_SYSTEM);
}

__global__ __launch_bounds__(512, 2) void ag_kernel(Params P) {
    extern __shared__ __attribute__((aligned(16))) unsigned char shm[];
    LAS unsigned char* lds = (LAS unsigned char*)shm;
    const int tid = threadIdx.x, lane = tid & 63;
    const int rank = P.rank, cls = rank >> 2, idx = rank & 3;
    char* myws = P.ws[rank];
    const int b = blockIdx.x;
    volatile LAS unsigned* cw = (volatile LAS unsigned*)(lds + LDS_CTL + 16);
    unsigned call;
    {
        if (tid == 0) *cw = __hip_atomic_fetch_add((unsigned*)(myws + WS_CTR) + b * 16, 1u, __ATOMIC_RELAXED, __HIP_MEMORY_SCOPE_AGENT) + 1u;
        __syncthreads();
        call = __builtin_amdgcn_readfirstlane(*cw);
    }
    STAMP(0);
    const size_t par = WS_STAGE + (size_t)(call & 1u) * PAR_BYTES;
    const char* xs = myws + par;
    const unsigned* flags = (const unsigned*)(myws + WS_FLAG);
    const unsigned needv = DBG(4) ? 0u : call;
    if (b < NPUSH) {
        if (!DBG(1)) push_role(lds, P, call, b);
    } else if (b >= GEMM0 && !DBG(8)) {
        const int g = b - GEMM0;
        constexpr int NA = 4 * RBA, NB = 4 * RBB;
        const int x = g & 7, s = g >> 3;
        const int a0 = x * NA / 8, a1 = (x + 1) * NA / 8, b0 = x * NB / 8, b1 = (x + 1) * NB / 8;
        const int na = a1 - a0;
        const int type = s < na ? 0 : 1;
        int t = type == 0 ? a0 + s : b0 + (s - na);
        if (type == 0) {
            constexpr int NLOC = 4 * (16 / NU), PERL = NLOC / 8;
            if (NA % 8 == 0 && NLOC % 8 == 0) t = s < PERL ? x * PERL + s : NLOC + x * (NA / 8 - PERL) + (s - PERL);
        }
        if (type == 0 || s - na < b1 - b0) {
            const int rb = t >> 2, cb = t & 3;
            TileDesc d;
            const int crank = (1 - cls) * 4 + cb;
            d.flag = flags; d.call1 = 0u; d.pbase = 0; d.anywait = false;
#pragma unroll
            for (int u = 0; u < NU; ++u) {
                int ui = rb * NU + u;
                const int nun = type == 0 ? UA : UB;
                const bool valid = ui < nun;
                if (!valid) ui = nun - 1;
                int src, su; const char* xp;
                const int nown = type == 0 ? 16 : U16;
                if (ui < nown) { src = rank; su = ui; xp = P.x + (size_t)ui * UNITB; }
                else if (ui < nown + 3 * U1) { const int sl = (ui - nown) / U1; su = (ui - nown) % U1; src = cls * 4 + ((idx + 1 + sl) & 3); xp = xs + (size_t)(sl * U16 + su) * UNITB; }
                else { int sl = (ui - nown - 3 * U1) / V16; const int jj = (ui - nown - 3 * U1) % V16; if (type == 1 && sl >= cb) ++sl;
                       src = (1 - cls) * 4 + sl; su = U16 + jj; xp = xs + (size_t)(3 * U16 + sl * V16 + jj) * UNITB; }
                d.xa[u] = xp;
                char* ob = type == 0 ? P.out[rank] + cb * 512 : P.out[crank] + idx * 512;
                d.out[u] = valid ? ob + (size_t)(src * MSH + su * 64) * WPITCH : nullptr;
                if (src != rank) { d.anywait = true; if (lane / (64 / (NU + 1)) == u) { d.flag = flags + src * FROW; d.call1 = needv; d.pbase = (src >> 2) == cls ? PBASE_S1 : PBASE_X; } }
            }
            d.throttle = type == 1;
            if (type == 0) { d.wb = P.w + cb * 512; d.wpitch = WPITCH; d.wbytes = KDIM * WPITCH - cb * 512; }
            else { d.wb = xs + XS_BYTES + (size_t)cb * WSQ_BYTES; d.wpitch = QPITCH; d.wbytes = (unsigned)WSQ_BYTES; d.anywait = true;
                   if (lane / (64 / (NU + 1)) == NU) { d.flag = flags + crank * FROW; d.call1 = needv; d.pbase = PBASE_X; } }
            gemm_tile(lds, d, P, call);
            if (type == 1) signal_done(P, crank);
        }
    }
    STAMP(3);
    if (U2 > 0 && !DBG(8)) {
        const int x = b & 7;
        for (;;) {
            __syncthreads();
            if (tid == 0) *cw = __hip_atomic_fetch_add((unsigned*)(myws + WS_QUEUE) + x * 32, 1u, __ATOMIC_RELAXED, __HIP_MEMORY_SCOPE_AGENT) - (call - 1u) * (unsigned)(NT2X + NWG / 8);
            __syncthreads();
            const unsigned tk = __builtin_amdgcn_readfirstlane(*cw);
            if (tk >= (unsigned)NT2X) break;
            const int cbk = 2 * x + (int)(tk & 1u), un = (int)(tk >> 1);
            const int sl = un / (U2 > 0 ? U2 : 1), su = U1 + un % (U2 > 0 ? U2 : 1);
            const int src = cls * 4 + ((idx + 1 + sl) & 3);
            const char* xa = xs + (size_t)(sl * U16 + su) * UNITB;
            const unsigned* xflag = flags + src * FROW + NCH1;
            const size_t orow = (size_t)(src * MSH + su * 64) * WPITCH;
            const bool own = cbk < 8;
            const int cb = (cbk - 8) >> 1, hf = cbk & 1, crank = own ? rank : (1 - cls) * 4 + cb;
            small_tile(lds, xa, own ? P.w + cbk * 256 : xs + XS_BYTES + (size_t)cb * WSQ_BYTES + hf * 256, own ? WPITCH : QPITCH,
                       own ? KDIM * WPITCH - cbk * 256 : (unsigned)WSQ_BYTES - hf * 256, P.out[crank] + orow + (own ? cbk * 256 : idx * 512 + hf * 256),
                       xflag, needv, flags + crank * FROW, own ? 0u : needv, P, call);
            if (!own) signal_done(P, crank);
            STAMP(7);
        }
    }
    if (b == 0 && !DBG(2)) {
        if (tid < 64) {
            unsigned* tmo = (unsigned*)(myws + WS_TMO);
            const bool ok = wave_wait((const unsigned*)(myws + WS_DONE), call * (unsigned)(4 * RBB + 4 * 3 * U2 * 2), tmo, call);
            if (!ok && lane == 0) *(unsigned short*)(P.out[rank] + (size_t)rank * MSH * WPITCH) = 0x7fc0u;
        }
        STAMP(4);
    }
}

}

#define AG_DBG 0
extern "C" void kernel_launch(int rank, int nranks, ncclComm_t comm, hipStream_t stream,
                              void* const* d_in, const int* in_sizes, int n_in,
                              void* d_out, int out_size, const int* in_dtypes, int out_dtype,
                              void* const* d_out_all, void* const* d_ws_all, size_t ws_bytes) {
    using namespace agk;
    (void)comm; (void)in_sizes; (void)n_in; (void)out_size; (void)in_dtypes; (void)out_dtype; (void)d_out;
    if (nranks != NR || ws_bytes < WS_NEED || rank < 0 || rank >= NR) { fprintf(stderr, "unsupported configuration\n"); return; }
    static bool init[64] = {};
    int dev = 0; HIP_CHECK(hipGetDevice(&dev));
    if (!init[dev & 63]) {
        HIP_CHECK(hipFuncSetAttribute((const void*)ag_kernel, hipFuncAttributeMaxDynamicSharedMemorySize, LDS_BYTES));
        init[dev & 63] = true;
    }
    Params P;
    P.x = (const char*)d_in[0]; P.w = (const char*)d_in[1];
    for (int r = 0; r < NR; ++r) { P.out[r] = (char*)d_out_all[r]; P.ws[r] = (char*)d_ws_all[r]; }
    P.rank = rank; P.dbg = AG_DBG;
    hipLaunchKernelGGL(ag_kernel, dim3(NWG), dim3(512), LDS_BYTES, stream, P);
    HIP_CHECK(hipGetLastError());
}
```

```cpp
#include <hip/hip_runtime.h>
#include <rccl/rccl.h>
#include <cstdio>
#include <cstdint>
#include <type_traits>

#define HIP_CHECK(expr) do { hipError_t _e = (expr); if (_e != hipSuccess) { \
    fprintf(stderr, "HIP error %s at %s:%d\n", hipGetErrorString(_e), __FILE__, __LINE__); \
    return; } } while (0)

#define DBG(bit) 0

namespace agk {

#define LAS __attribute__((address_space(3)))
typedef short bf16x8 __attribute__((ext_vector_type(8)));
typedef short s16x4 __attribute__((ext_vector_type(4)));
typedef float f32x4 __attribute__((ext_vector_type(4)));
typedef unsigned u32x4 __attribute__((ext_vector_type(4)));
typedef unsigned u32x2 __attribute__((ext_vector_type(2)));
typedef __amdgpu_buffer_rsrc_t rsrc_t;

constexpr int NR = 8, KDIM = 8192, MSH = 1024, NSH = 1024;
constexpr unsigned XPITCH = KDIM * 2;
constexpr unsigned WPITCH = NSH * 2;
constexpr unsigned QPITCH = 512;
constexpr unsigned UNITB = 64u * XPITCH;

#define CFG_U16 11
#define CFG_U2 2
#define CFG_NU 4
#define CFG_NJ 16
#define CFG_EPI_INFLIGHT 0
#define CFG_PU 2
constexpr int U16 = CFG_U16, V16 = 16 - U16;
constexpr int U2 = CFG_U2, U1 = U16 - U2;
constexpr int NU = CFG_NU;
#define CFG_NJS CFG_NJ
constexpr int NJX = CFG_NJ, NJS = CFG_NJS;
constexpr int NJM = NJX < NJS ? NJX : NJS;
constexpr int NPUSH = 3 * NJS + 4 * NJX;
constexpr int UA = 16 + 3 * U1 + 4 * V16;
constexpr int UB = U16 + 3 * U1 + 3 * V16;
constexpr int RBA = (UA + NU - 1) / NU, RBB = (UB + NU - 1) / NU;
constexpr int NWG = 256;
constexpr int NGWG = 8 * ((4 * RBA + 7) / 8 + (4 * RBB + 7) / 8);
static_assert(NPUSH + NGWG <= NWG, "grid");
constexpr int GEMM0 = NWG - NGWG;
constexpr int NT2X = 3 * U2 * 2;

#define CFG_PA 1
#define CFG_PA_CNT 1
#define CFG_PA_TAIL 0
#define CFG_POLL_SLEEP 1
#define CFG_EPI_W2 0
#define CFG_ASYNC 0
#define CFG_XSLEEP 0
#define CFG_SM_SPLIT 1
#define CFG_PMODE 0
#define WAIT_VU2() WAIT_V(4)
#define WAIT_VU1() WAIT_V(3)
#define CFG_LG1 0
#define CFG_LG2 1
constexpr int NT = KDIM / 64;
constexpr int LG1 = CFG_LG1, LG2 = CFG_LG2;
__host__ __device__ constexpr int ch_body(int lg) { return (NT >> lg) - 1; }
__host__ __device__ constexpr int ch_count(int lg) { return ch_body(lg) + lg + 1; }
__host__ __device__ constexpr int ch_lg(int lg, int c) { return c < ch_body(lg) ? lg : (c - ch_body(lg) < lg ? lg - 1 - (c - ch_body(lg)) : 0); }
constexpr int NCH1 = ch_count(LG1), NCH2 = ch_count(LG2), NCH = NCH1 + NCH2;
__host__ __device__ constexpr int CH_LOGF(int c) { return c < NCH1 ? ch_lg(LG1, c) : ch_lg(LG2, c - NCH1); }
constexpr int ch_sum(int a, int b) { int s = 0; for (int i = a; i < b; ++i) s += 1 << CH_LOGF(i); return s; }
static_assert(ch_sum(0, NCH1) == NT && ch_sum(NCH1, NCH) == NT, "chunks");
constexpr unsigned long long ch_mask(int gen, int half) { unsigned long long m = 0; int s = 0;
    for (int i = gen ? NCH1 : 0; i < (gen ? NCH : NCH1); ++i) { if (s / 64 == half) m |= 1ull << (s % 64); s += 1 << CH_LOGF(i); } return m; }
constexpr unsigned long long CHM0 = ch_mask(0, 0), CHM1 = ch_mask(0, 1), CH2M0 = ch_mask(1, 0), CH2M1 = ch_mask(1, 1);
constexpr int FROW = 256;
static_assert(NCH <= FROW, "flags");

#define CFG_WAVETK 1
constexpr int PB = 8;
constexpr int PBASE_S1 = U1 * 8, PBASE_S2 = U2 * 8, PBASE_X = V16 * 8 + 32;
__host__ __device__ constexpr int nbatch(int pieces) { return (pieces + PB - 1) / PB; }
constexpr int nb_total(bool same) { int s = 0; for (int c = 0; c < NCH; ++c) { const int pb = c < NCH1 ? (same ? PBASE_S1 : PBASE_X) : (same ? PBASE_S2 : 0); s += nbatch(pb << CH_LOGF(c)); } return s; }
constexpr int NB_S = nb_total(true), NB_X = nb_total(false);

constexpr size_t WS_CTR = 0;
constexpr size_t WS_TMO = 16384;
constexpr size_t WS_DONE = 16384 + 256;
constexpr size_t WS_QUEUE = 16384 + 512;
constexpr size_t WS_TICK = 16384 + 2048;
constexpr size_t WS_FLAG = 32768;
constexpr size_t WS_DBG = 65536;
constexpr size_t WS_STAGE = 1u << 20;
constexpr size_t XS_BYTES = (size_t)(3 * U16 + 4 * V16) * UNITB;
constexpr size_t WSQ_BYTES = (size_t)KDIM * QPITCH;
constexpr size_t PAR_BYTES = XS_BYTES + 4 * WSQ_BYTES;
constexpr size_t WS_NEED = WS_STAGE + 2 * PAR_BYTES;
#define CFG_DEADLINE 100000000ull
constexpr unsigned long long DEADLINE = CFG_DEADLINE;

constexpr int LDS_STAGE = 2 * NU * 8192 + 65536;
constexpr int LDS_EPI = NU * 64 * 528;
constexpr int LDS_SMALL = 6 * 24576;
constexpr int cmax(int a, int b) { return a > b ? a : b; }
constexpr int LDS_CTL = cmax(cmax(LDS_STAGE, LDS_EPI), LDS_SMALL);
constexpr int LDS_BYTES = LDS_CTL + 256;

struct Params {
    const char* x; const char* w;
    char* out[NR]; char* ws[NR];
    int rank; int dbg;
};

__device__ __forceinline__ unsigned cvt_pk_bf16(float lo, float hi) { unsigned r; asm("v_cvt_pk_bf16_f32 %0, %1, %2" : "=v"(r) : "v"(lo), "v"(hi)); return r; }
__device__ __forceinline__ float gelu1(float y) {
    const float c2 = -2.0f * 0.7978845608028654f * 1.4426950408889634f;
    const float a = y * __builtin_fmaf(y * y, 0.044715f * c2, c2);
    return y * __builtin_amdgcn_rcpf(1.0f + __builtin_amdgcn_exp2f(a));
}
__device__ __forceinline__ rsrc_t mk_rsrc(const void* p, unsigned bytes) { return __builtin_amdgcn_make_buffer_rsrc((void*)p, 0, bytes, 0x00020000); }

#define WAIT_V_(n) asm volatile("s_waitcnt vmcnt(" #n ")" ::: "memory")
#define WAIT_V(n) WAIT_V_(n)
#define WAIT_L(n) asm volatile("s_waitcnt lgkmcnt(" #n ")" ::: "memory")
#define BAR() __builtin_amdgcn_s_barrier()
#define SCHED() __builtin_amdgcn_sched_barrier(0)
#define SYS_LOAD(p) __hip_atomic_load((p), __ATOMIC_RELAXED, __HIP_MEMORY_SCOPE_SYSTEM)
#define AGT_LOAD(p) __hip_atomic_load((p), __ATOMIC_RELAXED, __HIP_MEMORY_SCOPE_AGENT)
#define CFG_DMA_AUX 17
#define DMA(rs, ldsp, voff, soff) __builtin_amdgcn_raw_ptr_buffer_load_lds((rs), (LAS void*)(ldsp), 16, (voff), (soff), 0, CFG_DMA_AUX)
#define TRR(dst, base, off) asm volatile("ds_read_b64_tr_b16 %0, %1 offset:%2" : "=v"(dst) : "v"(base), "i"(off) : "memory")

#define STAMP(slot) do {} while (0)

__device__ __forceinline__ bool wave_wait(const unsigned* p, unsigned need, unsigned* tmo, unsigned call) {
    const unsigned long long t0 = __builtin_amdgcn_s_memrealtime();
    for (unsigned n = 1;; ++n) {
        const unsigned v = SYS_LOAD(p);
        if (__all((int)(v - need) >= 0 || need == 0u)) return true;
        if ((n & 15u) == 0u && (__builtin_amdgcn_readfirstlane(AGT_LOAD(tmo)) == call || __builtin_amdgcn_s_memrealtime() - t0 > DEADLINE)) {
            __hip_atomic_store(tmo, call, __ATOMIC_RELAXED, __HIP_MEMORY_SCOPE_AGENT);
            return false;
        }
        __builtin_amdgcn_s_sleep(CFG_POLL_SLEEP);
    }
}

template <int WN, int NS>
__device__ __forceinline__ int window_wait(const unsigned* p, unsigned need, unsigned* tmo, unsigned call) {
    const unsigned long long t0 = __builtin_amdgcn_s_memrealtime();
    for (unsigned n = 1;; ++n) {
        const unsigned v = SYS_LOAD(p);
        unsigned long long m = __ballot((int)(v - need) >= 0 || need == 0u), a = m;
#pragma unroll
        for (int s = 1; s < NS; ++s) a &= m >> (s * WN);
        a &= (1ull << WN) - 1ull;
        if (a & 1ull) return (int)__builtin_ctzll(~a);
        if ((n & 15u) == 0u && (__builtin_amdgcn_readfirstlane(AGT_LOAD(tmo)) == call || __builtin_amdgcn_s_memrealtime() - t0 > DEADLINE)) {
            __hip_atomic_store(tmo, call, __ATOMIC_RELAXED, __HIP_MEMORY_SCOPE_AGENT);
            return WN;
        }
        __builtin_amdgcn_s_sleep(CFG_POLL_SLEEP);
    }
}

__device__ __forceinline__ void store_out(char* o, u32x4 v) { asm volatile("global_store_dwordx4 %0, %1, off sc0 sc1\n\ts_nop 1" :: "v"(o), "v"(v) : "memory"); }

struct TileDesc {
    const char* xa[NU];
    const char* wb; unsigned wpitch, wbytes;
    char* out[NU];
    const unsigned* flag;
    unsigned call1;
    int pbase;
    bool anywait, throttle;
};

__device__ __forceinline__ void gemm_tile(LAS unsigned char* lds, const TileDesc& d, const Params& P, unsigned call) {
    constexpr int MT = NU * 2;
    constexpr int ABUF = NU * 8192;
    constexpr int BOFF = 2 * ABUF;
    const int tid = threadIdx.x, wid = __builtin_amdgcn_readfirstlane(tid >> 6), lane = tid & 63;
    const int wr = wid >> 2, wc = wid & 3, fr = lane & 15, fq = lane >> 4;
    unsigned* tmo = (unsigned*)(P.ws[P.rank] + WS_TMO);
    volatile LAS unsigned* ready = (volatile LAS unsigned*)(lds + LDS_CTL);
    const unsigned voffA = (unsigned)(8 * wid + (lane >> 3)) * XPITCH + (unsigned)(((lane & 7) ^ (((wid & 1) << 2) | (lane >> 4))) * 16);
    const unsigned fB = (unsigned)((lane >> 4) | (((wid >> 1) & 1) << 2));
    const unsigned voffB = (unsigned)(4 * wid + (lane >> 4)) * d.wpitch + ((((unsigned)(lane & 15) >> 1) ^ fB) * 32u + (unsigned)(lane & 1) * 16u);
    const unsigned ldsw = (unsigned)wid * 1024u;
    const unsigned aoff0 = (unsigned)(wr * (NU * 32) + fr) * 128u + (unsigned)((fq ^ (fr >> 1)) * 16);
    const unsigned aoff1 = aoff0 ^ 64u;
    const unsigned q = (unsigned)(fr >> 2), p = (unsigned)(fr & 3);
    const unsigned boff0 = (unsigned)(size_t)lds + BOFF + (unsigned)(8 * fq + q) * 256u + (((unsigned)(2 * wc) ^ (q | ((unsigned)(fq & 1) << 2))) * 32u) + p * 8u;
    const unsigned boff1 = boff0 ^ 32u;
    rsrc_t ra[NU];
#pragma unroll
    for (int u = 0; u < NU; ++u) ra[u] = mk_rsrc(d.xa[u], UNITB);
    const rsrc_t rb = mk_rsrc(d.wb, d.wbytes);
    const unsigned wstep = __builtin_amdgcn_readfirstlane(64u * d.wpitch), whalf = wstep >> 1;

    f32x4 acc[2][MT][2];
#pragma unroll
    for (int b = 0; b < 2; ++b)
#pragma unroll
        for (int m = 0; m < MT; ++m)
#pragma unroll
            for (int n = 0; n < 2; ++n) acc[b][m][n] = (f32x4){0.f, 0.f, 0.f, 0.f};
    bf16x8 af[MT][2], bfr[2][2]; s16x4 tb[8];

#define STAGE_A(buf, t) do { _Pragma("unroll") for (int _i = 0; _i < NU; ++_i) DMA(ra[_i], lds + (buf) * ABUF + _i * 8192 + ldsw, voffA, (unsigned)(t) * 128u); } while (0)
#define STAGE_B(buf, h, t) do { _Pragma("unroll") for (int _i = 0; _i < 2; ++_i) \
        DMA(rb, lds + BOFF + ((buf) * 2 + (h)) * 16384 + _i * 8192 + ldsw, voffB, (unsigned)(t) * wstep + _i * whalf + (h) * 256u); } while (0)
#define LDA(buf) do { _Pragma("unroll") for (int m = 0; m < MT; ++m) { \
        af[m][0] = *(const LAS bf16x8*)(lds + (buf) * ABUF + aoff0 + m * 2048); \
        af[m][1] = *(const LAS bf16x8*)(lds + (buf) * ABUF + aoff1 + m * 2048); } } while (0)
#define LDB(buf, h) do { \
        TRR(tb[0], boff0, ((buf) * 2 + (h)) * 16384); TRR(tb[1], boff0, ((buf) * 2 + (h)) * 16384 + 1024); \
        TRR(tb[2], boff1, ((buf) * 2 + (h)) * 16384); TRR(tb[3], boff1, ((buf) * 2 + (h)) * 16384 + 1024); \
        TRR(tb[4], boff0, ((buf) * 2 + (h)) * 16384 + 8192); TRR(tb[5], boff0, ((buf) * 2 + (h)) * 16384 + 8192 + 1024); \
        TRR(tb[6], boff1, ((buf) * 2 + (h)) * 16384 + 8192); TRR(tb[7], boff1, ((buf) * 2 + (h)) * 16384 + 8192 + 1024); } while (0)
#define WAITB() do { asm volatile("s_waitcnt lgkmcnt(0)" : "+v"(tb[0]), "+v"(tb[1]), "+v"(tb[2]), "+v"(tb[3]), "+v"(tb[4]), "+v"(tb[5]), "+v"(tb[6]), "+v"(tb[7]) :: "memory"); \
        bfr[0][0] = __builtin_shufflevector(tb[0], tb[1], 0, 1, 2, 3, 4, 5, 6, 7); bfr[1][0] = __builtin_shufflevector(tb[2], tb[3], 0, 1, 2, 3, 4, 5, 6, 7); \
        bfr[0][1] = __builtin_shufflevector(tb[4], tb[5], 0, 1, 2, 3, 4, 5, 6, 7); bfr[1][1] = __builtin_shufflevector(tb[6], tb[7], 0, 1, 2, 3, 4, 5, 6, 7); } while (0)
#define MMA(bj) do { __builtin_amdgcn_s_setprio(1); \
        _Pragma("unroll") for (int kk = 0; kk < 2; ++kk) _Pragma("unroll") for (int m = 0; m < MT; ++m) _Pragma("unroll") for (int n = 0; n < 2; ++n) \
            acc[bj][m][n] = __builtin_amdgcn_mfma_f32_16x16x32_bf16(bfr[n][kk], af[m][kk], acc[bj][m][n], 0, 0, 0); \
        __builtin_amdgcn_s_setprio(0); } while (0)
#define WAITN() do { if constexpr (NU == 3) { WAIT_V(7); } else if constexpr (NU == 2) { WAIT_V(6); } else { WAIT_V(8); } } while (0)
    constexpr int NS = NU + 1, WN = 64 / NS;
    int cnext = 0, known = 0;
#define POLL() do { const int ck_ = cnext + lane % WN; const bool w_ = ck_ < NCH1 && lane < NS * WN; const int cc_ = w_ ? ck_ : 0; \
        known = cnext + window_wait<WN, NS>(d.flag + cc_, w_ ? d.call1 * (unsigned)nbatch(d.pbase << CH_LOGF(cc_)) : 0u, tmo, call); } while (0)
#define GATE() do { \
        if (known <= cnext) { \
            if (wid == 0) { POLL(); if (lane == 0) *ready = (unsigned)known; } \
            else { unsigned r_; while ((r_ = *ready) <= (unsigned)cnext) __builtin_amdgcn_s_sleep(1); known = (int)__builtin_amdgcn_readfirstlane(r_); } \
        } \
        if ((cnext & 3) == 0) STAMP(8 + (cnext >> 2)); \
        ++cnext; } while (0)
#define CHECK(tt) do { if (d.anywait && (tt) < NT && ((((tt) < 64 ? CHM0 : CHM1) >> ((tt) & 63)) & 1ull)) GATE(); } while (0)

    if (d.anywait) {
        constexpr int C01 = LG1 == 0 ? 2 : 1;
        cnext = C01 - 1;
        if (wid == 0) { if (C01 == 2) { cnext = 0; POLL(); cnext = 1; } if (known <= cnext) POLL(); if (lane == 0) *ready = (unsigned)known; }
        WAIT_L(0); BAR();
        known = (int)__builtin_amdgcn_readfirstlane(*ready);
        cnext = C01;
    }
    STAMP(1);
    STAGE_A(0, 0); STAGE_B(0, 0, 0); STAGE_B(0, 1, 0);
    STAGE_A(1, 1); STAGE_B(1, 0, 1);
    if (wr == 1) BAR();
    if constexpr (NU == 3) { WAIT_V(5); } else if constexpr (NU == 2) { WAIT_V(4); } else { WAIT_V(6); }
    BAR(); BAR();
#define KTILE(b, t) do { \
        LDB(b, 0); LDA(b); \
        STAGE_B((b) ^ 1, 1, min((t) + 1, NT - 1)); \
        WAITN(); WAITB(); BAR(); SCHED(); MMA(0); SCHED(); BAR(); \
        CHECK((t) + 2); \
        LDB(b, 1); \
        STAGE_A(b, min((t) + 2, NT - 1)); STAGE_B(b, 0, min((t) + 2, NT - 1)); \
        WAITN(); WAITB(); BAR(); SCHED(); MMA(1); SCHED(); BAR(); } while (0)
    for (int t = 0; t < NT; t += 2) {
        KTILE(0, t);
        KTILE(1, t + 1);
    }
    WAIT_V(0);
    if (wr == 0) BAR();
    BAR();
    STAMP(2);
    constexpr int EP = 528;
    const bool bad = AGT_LOAD(tmo) == call;
#pragma unroll
    for (int bj = 0; bj < 2; ++bj)
#pragma unroll
        for (int m = 0; m < MT; ++m)
#pragma unroll
            for (int n = 0; n < 2; ++n) {
                const f32x4 v = acc[bj][m][n];
                u32x2 w; w.x = cvt_pk_bf16(gelu1(v[0]), gelu1(v[1])); w.y = cvt_pk_bf16(gelu1(v[2]), gelu1(v[3]));
                *(LAS u32x2*)(lds + (wr * (NU * 32) + m * 16 + fr) * EP + (bj * 128 + wc * 32 + n * 16 + fq * 4) * 2) = w;
            }
    WAIT_L(0); BAR();
#pragma unroll
    for (int u = 0; u < NU; ++u) {
        if (d.out[u] != nullptr) {
#pragma unroll
            for (int i = 0; i < 4; ++i) {
                const int r = (i * 8 + wid) * 2 + (lane >> 5);
                u32x4 v = *(const LAS u32x4*)(lds + (u * 64 + r) * EP + (lane & 31) * 16);
                if (bad) v = (u32x4){0x7fc07fc0u, 0x7fc07fc0u, 0x7fc07fc0u, 0x7fc07fc0u};
                store_out(d.out[u] + (size_t)r * WPITCH + (lane & 31) * 16, v);
                if (d.throttle) { if (wid < CFG_EPI_W2) WAIT_V(1); else WAIT_V(0); }
            }
        }
    }
#undef STAGE_A
#undef STAGE_B
#undef LDA
#undef LDB
#undef MMA
#undef KTILE
#undef WAITB
#undef WAITN
#undef GATE
#undef CHECK
#undef POLL
}

__device__ __forceinline__ void small_tile(LAS unsigned char* lds, const char* xa, const char* wb, unsigned wpitch, unsigned wbytes, char* out,
                                           const unsigned* xflag, unsigned xcall, const unsigned* wflag, unsigned wcall, const Params& P, unsigned call) {
    const int tid = threadIdx.x, wid = __builtin_amdgcn_readfirstlane(tid >> 6), lane = tid & 63;
    const int g = wid >> 2, wq = wid & 3, wm = wq >> 1, wn = wq & 1, fr = lane & 15, fq = lane >> 4;
    unsigned* tmo = (unsigned*)(P.ws[P.rank] + WS_TMO);
    volatile LAS unsigned* ready = (volatile LAS unsigned*)(lds + LDS_CTL + 32) + g;
    const unsigned voffA = (unsigned)(8 * wq + (lane >> 3)) * XPITCH + (unsigned)(((lane & 7) ^ (((wq & 1) << 2) | (lane >> 4))) * 16);
    const unsigned fB = (unsigned)((lane >> 4) | (((wq >> 1) & 1) << 2));
    const unsigned voffB = (unsigned)(4 * wq + (lane >> 4)) * wpitch + ((((unsigned)(lane & 15) >> 1) ^ fB) * 32u + (unsigned)(lane & 1) * 16u);
    const unsigned gbase = (unsigned)g * (3u * 24576u);
    const unsigned ldsw = gbase + (unsigned)wq * 1024u;
    const unsigned aoff0 = gbase + (unsigned)(wm * 32 + fr) * 128u + (unsigned)((fq ^ (fr >> 1)) * 16);
    const unsigned aoff1 = aoff0 ^ 64u;
    const unsigned q = (unsigned)(fr >> 2), p = (unsigned)(fr & 3);
    const unsigned boff = (unsigned)(size_t)lds + gbase + 8192u + (unsigned)(8 * fq + q) * 256u + (((unsigned)(4 * wn) ^ (q | ((unsigned)(fq & 1) << 2))) * 32u) + p * 8u;
    const unsigned bo0 = boff, bo1 = boff ^ 32u, bo2 = boff ^ 64u, bo3 = boff ^ 96u;
    const rsrc_t ra = mk_rsrc(xa, UNITB), rb = mk_rsrc(wb, wbytes);
    const unsigned wstep = __builtin_amdgcn_readfirstlane(64u * wpitch), wq16 = wstep >> 2;
    f32x4 acc[2][4];
#pragma unroll
    for (int m = 0; m < 2; ++m)
#pragma unroll
        for (int n = 0; n < 4; ++n) acc[m][n] = (f32x4){0.f, 0.f, 0.f, 0.f};
#define STAGE(s, t) do { DMA(ra, lds + (s) * 24576 + ldsw, voffA, (unsigned)(t) * 128u); DMA(ra, lds + (s) * 24576 + 4096 + ldsw, voffA, (unsigned)(t) * 128u + 32u * XPITCH); \
        _Pragma("unroll") for (int _j = 0; _j < 4; ++_j) DMA(rb, lds + (s) * 24576 + 8192 + _j * 4096 + ldsw, voffB, (unsigned)(t) * wstep + _j * wq16); } while (0)
#define DMA_A(s, t, i) DMA(ra, lds + (s) * 24576 + (i) * 4096 + ldsw, voffA, (unsigned)(t) * 128u + (i) * 32u * XPITCH)
#define DMA_B(s, t, j) DMA(rb, lds + (s) * 24576 + 8192 + (j) * 4096 + ldsw, voffB, (unsigned)(t) * wstep + (j) * wq16)
    int known = 0;
#define CIDX(tt) ((tt) < 64 ? __builtin_popcountll(CH2M0 & ((2ull << (tt)) - 1ull)) - 1 : __builtin_popcountll(CH2M0) + __builtin_popcountll(CH2M1 & ((2ull << ((tt) - 64)) - 1ull)) - 1)
#define POLL(ci) do { const int ck_ = (ci) + lane; const bool w_ = ck_ < NCH2 && lane < 32; const int cc_ = w_ ? ck_ : 0; \
        known = (ci) + window_wait<32, 1>(xflag + cc_, w_ ? xcall * (unsigned)nbatch(PBASE_S2 << CH_LOGF(NCH1 + cc_)) : 0u, tmo, call); } while (0)
#define CHECK(tt) do { if ((tt) < NT) { const int ci_ = CIDX(tt); if (known <= ci_) { \
            if (wq == 0) { POLL(ci_); if (lane == 0) *ready = (unsigned)known; } \
            else { unsigned r_; while ((r_ = *ready) <= (unsigned)ci_) __builtin_amdgcn_s_sleep(1); known = (int)__builtin_amdgcn_readfirstlane(r_); } } } } while (0)
    BAR();
    if (wid == 0) {
        for (int c0 = 0; c0 < NCH1; c0 += 64) { const int lc = c0 + lane < NCH1 ? c0 + lane : 0;
            wave_wait(wflag + lc, c0 + lane < NCH1 ? wcall * (unsigned)nbatch(PBASE_X << CH_LOGF(lc)) : 0u, tmo, call); }
        known = 0; while (known <= CIDX(3)) POLL(known);
        if (lane == 0) { ready[0] = (unsigned)known; ready[1] = (unsigned)known; }
    }
    WAIT_L(0); BAR();
    known = (int)__builtin_amdgcn_readfirstlane(*ready);
    STAMP(5);
    STAGE(0, g); STAGE(1, g + 2);
    WAIT_V(6); BAR();
    if (g == 1) BAR();
    s16x4 tb[16]; bf16x8 af[2][2];
#define STEP(s, t) do { \
        CHECK((t) + 4); \
        TRR(tb[0], bo0, (s) * 24576); TRR(tb[1], bo0, (s) * 24576 + 1024); TRR(tb[2], bo1, (s) * 24576); TRR(tb[3], bo1, (s) * 24576 + 1024); \
        TRR(tb[4], bo2, (s) * 24576); TRR(tb[5], bo2, (s) * 24576 + 1024); TRR(tb[6], bo3, (s) * 24576); TRR(tb[7], bo3, (s) * 24576 + 1024); \
        TRR(tb[8], bo0, (s) * 24576 + 8192); TRR(tb[9], bo0, (s) * 24576 + 9216); TRR(tb[10], bo1, (s) * 24576 + 8192); TRR(tb[11], bo1, (s) * 24576 + 9216); \
        TRR(tb[12], bo2, (s) * 24576 + 8192); TRR(tb[13], bo2, (s) * 24576 + 9216); TRR(tb[14], bo3, (s) * 24576 + 8192); TRR(tb[15], bo3, (s) * 24576 + 9216); \
        _Pragma("unroll") for (int m = 0; m < 2; ++m) { af[m][0] = *(const LAS bf16x8*)(lds + (s) * 24576 + aoff0 + m * 2048); af[m][1] = *(const LAS bf16x8*)(lds + (s) * 24576 + aoff1 + m * 2048); } \
        const int tn_ = min((t) + 4, NT - 2 + g); \
        DMA_A(((s) + 2) % 3, tn_, 0); DMA_B(((s) + 2) % 3, tn_, 0); if (CFG_SM_SPLIT < 2) { DMA_B(((s) + 2) % 3, tn_, 1); } if (CFG_SM_SPLIT < 1) { DMA_A(((s) + 2) % 3, tn_, 1); DMA_B(((s) + 2) % 3, tn_, 2); DMA_B(((s) + 2) % 3, tn_, 3); } \
        asm volatile("s_waitcnt lgkmcnt(0)" : "+v"(tb[0]), "+v"(tb[1]), "+v"(tb[2]), "+v"(tb[3]), "+v"(tb[4]), "+v"(tb[5]), "+v"(tb[6]), "+v"(tb[7]), \
                     "+v"(tb[8]), "+v"(tb[9]), "+v"(tb[10]), "+v"(tb[11]), "+v"(tb[12]), "+v"(tb[13]), "+v"(tb[14]), "+v"(tb[15]) :: "memory"); \
        BAR(); SCHED(); \
        __builtin_amdgcn_s_setprio(1); \
        _Pragma("unroll") for (int kk = 0; kk < 2; ++kk) _Pragma("unroll") for (int n = 0; n < 4; ++n) { \
            const bf16x8 b_ = __builtin_shufflevector(tb[kk * 8 + n * 2], tb[kk * 8 + n * 2 + 1], 0, 1, 2, 3, 4, 5, 6, 7); \
            _Pragma("unroll") for (int m = 0; m < 2; ++m) acc[m][n] = __builtin_amdgcn_mfma_f32_16x16x32_bf16(b_, af[m][kk], acc[m][n], 0, 0, 0); \
            if (CFG_SM_SPLIT >= 1 && (n & 1) == 1) { SCHED(); \
                if (kk == 0 && n == 1) { DMA_A(((s) + 2) % 3, tn_, 1); } \
                if (kk == 0 && n == 3) { DMA_B(((s) + 2) % 3, tn_, 2); } \
                if (kk == 1 && n == 1) { DMA_B(((s) + 2) % 3, tn_, 3); } \
                if (kk == 1 && n == 3 && CFG_SM_SPLIT >= 2) { DMA_B(((s) + 2) % 3, tn_, 1); } \
                SCHED(); } } \
        __builtin_amdgcn_s_setprio(0); \
        SCHED(); WAIT_V(6); BAR(); } while (0)
    for (int i = 0; i < 63; i += 3) {
        STEP(0, g + 2 * i); STEP(1, g + 2 * i + 2); STEP(2, g + 2 * i + 4);
    }
    STEP(0, g + 126);
    WAIT_V(0);
    if (g == 0) BAR();
    BAR();
    STAMP(6);
    constexpr int EP = 272;
    const bool bad = AGT_LOAD(tmo) == call;
    if (g == 1) {
#pragma unroll
        for (int m = 0; m < 2; ++m)
#pragma unroll
            for (int n = 0; n < 4; ++n) *(LAS f32x4*)(lds + 32768 + ((wq * 8 + m * 4 + n) * 64 + lane) * 16) = acc[m][n];
    }
    WAIT_L(0); BAR();
    if (g == 0) {
#pragma unroll
        for (int m = 0; m < 2; ++m)
#pragma unroll
            for (int n = 0; n < 4; ++n) {
                const f32x4 v = acc[m][n] + *(const LAS f32x4*)(lds + 32768 + ((wq * 8 + m * 4 + n) * 64 + lane) * 16);
                u32x2 w; w.x = cvt_pk_bf16(gelu1(v[0]), gelu1(v[1])); w.y = cvt_pk_bf16(gelu1(v[2]), gelu1(v[3]));
                *(LAS u32x2*)(lds + (wm * 32 + m * 16 + fr) * EP + (wn * 64 + n * 16 + fq * 4) * 2) = w;
            }
    }
    WAIT_L(0); BAR();
#pragma unroll
    for (int i = 0; i < 2; ++i) {
        const int r = (i * 8 + wid) * 4 + (lane >> 4);
        u32x4 v = *(const LAS u32x4*)(lds + r * EP + (lane & 15) * 16);
        if (bad) v = (u32x4){0x7fc07fc0u, 0x7fc07fc0u, 0x7fc07fc0u, 0x7fc07fc0u};
        store_out(out + (size_t)r * WPITCH + (lane & 15) * 16, v);
    }
#undef STAGE
#undef DMA_A
#undef DMA_B
#undef CIDX
#undef CHECK
#undef POLL
#undef STEP
}

__device__ __forceinline__ void push_role(LAS unsigned char* lds, const Params& P, unsigned call, int pw) {
    const int tid = threadIdx.x, wid = __builtin_amdgcn_readfirstlane(tid >> 6), lane = tid & 63;
    const int rank = P.rank, cls = rank >> 2, idx = rank & 3;
    int di, j;
    if (pw < 7 * NJM) { di = pw % 7; j = pw / 7; }
    else if (NJS > NJX) { const int r = pw - 7 * NJM; di = r % 3; j = NJM + r / 3; }
    else { const int r = pw - 7 * NJM; di = 3 + r % 4; j = NJM + r / 4; }
    const bool same = di < 3;
    const int q = same ? (cls * 4 + ((idx + 1 + di) & 3)) : ((1 - cls) * 4 + (di - 3));
    const int qidx = q & 3;
    const size_t par = WS_STAGE + (size_t)(call & 1u) * PAR_BYTES;
    const int slot = same ? ((idx - qidx - 1) & 3) : idx;
    const size_t xdst_off = par + (same ? (size_t)slot * U16 * UNITB : (size_t)(3 * U16 + slot * V16) * UNITB);
    const size_t wdst_off = par + XS_BYTES + (size_t)slot * WSQ_BYTES;
    const unsigned xbytes = (unsigned)(same ? U16 : V16) * UNITB;
    const char* xsrc = P.x + (same ? 0 : (size_t)U16 * UNITB);
    const char* wsrc = P.w + qidx * 512;
    char* wsq = P.ws[q];
    unsigned* flag = (unsigned*)(wsq + WS_FLAG) + rank * FROW;
    const unsigned nbt = same ? NB_S : NB_X;
    const unsigned njd = same ? NJS : NJX;
    unsigned* tick = (unsigned*)(P.ws[rank] + WS_TICK) + di * 32;
    const unsigned tbase = (call - 1u) * nbt - 3u * njd;
    volatile LAS unsigned* tk = (volatile LAS unsigned*)lds;
    LAS unsigned* cnt = (LAS unsigned*)(lds + 32);
    volatile LAS unsigned* done = (volatile LAS unsigned*)(lds + 64);
    if (tid < 8) { cnt[tid] = 0u; tk[tid] = tid < 3 ? (((unsigned)tid + 1u) << 20 | ((unsigned)j + (unsigned)tid * njd)) : 0u; }
    if (tid == 8) *done = 0u;
    __syncthreads();
    int c = 0, kt0 = 0, row0 = 0, pbase = same ? PBASE_S1 : PBASE_X, nx0 = same ? PBASE_S1 : V16 * 8;
    unsigned cb0 = 0;
#define LOCATE(tt) do { for (;;) { const unsigned nbc_ = (unsigned)nbatch(pbase << CH_LOGF(c)); if ((tt) < cb0 + nbc_) break; \
            cb0 += nbc_; kt0 += 1 << CH_LOGF(c); ++c; \
            if (c == NCH1) { kt0 = 0; row0 = U1 * 64; pbase = PBASE_S2; nx0 = PBASE_S2; } } } while (0)
#define PIECE(tt, soff, doff, w) do { const int lg_ = CH_LOGF(c), ptot_ = pbase << lg_, nx_ = nx0 << lg_; const int i_ = (int)((tt) - cb0) * PB + wid; \
            soff = 0xffffff00u; doff = 0xffffff00u; w = false; \
            if (i_ < nx_) { const unsigned o_ = (unsigned)i_ * 1024u + (unsigned)lane * 16u; \
                const unsigned row_ = (unsigned)row0 + (o_ >> (7 + lg_)), col_ = o_ & ((128u << lg_) - 1u); soff = doff = row_ * XPITCH + (unsigned)kt0 * 128u + col_; } \
            else if (i_ < ptot_) { const unsigned k_ = (unsigned)kt0 * 64u + (unsigned)(i_ - nx_) * 2u + (unsigned)(lane >> 5); \
                soff = k_ * WPITCH + (unsigned)(lane & 31) * 16u; doff = k_ * QPITCH + (unsigned)(lane & 31) * 16u; w = true; } } while (0)
    auto desc = [](const void* p, unsigned bytes) { const unsigned long long a = (unsigned long long)p; u32x4 r;
        r.x = __builtin_amdgcn_readfirstlane((unsigned)a); r.y = __builtin_amdgcn_readfirstlane((unsigned)(a >> 32) & 0xffffu); r.z = __builtin_amdgcn_readfirstlane(bytes); r.w = 0x00020000u; return r; };
    const u32x4 dxs = desc(xsrc, xbytes), dxd = desc(wsq + xdst_off, xbytes), dws = desc(wsrc, (unsigned)KDIM * WPITCH - (unsigned)qidx * 512u), dwd = desc(wsq + wdst_off, (unsigned)WSQ_BYTES);
#define LD(dst, w, off) asm volatile("buffer_load_dwordx4 %0, %1, %2, 0 offen" : "=v"(dst) : "v"(off), "s"((w) ? dws : dxs) : "memory")
#define ST(src, w, off) asm volatile("buffer_store_dwordx4 %0, %1, %2, 0 offen sc0 sc1\n\ts_nop 1" :: "v"(src), "v"(off), "s"((w) ? dwd : dxd) : "memory")
#define FETCH_ASYNC(dst) do { unsigned long long ex_; asm volatile("s_mov_b64 %1, exec\n\ts_mov_b64 exec, 1\n\tglobal_atomic_add %0, %2, %3, %4 sc0\n\ts_mov_b64 exec, %1" \
        : "=&v"(dst), "=&s"(ex_) : "v"(0u), "v"(1u), "s"(tick) : "memory"); } while (0)
#define FLAG_ADD(idx) do { unsigned long long ex_; asm volatile("s_mov_b64 %0, exec\n\ts_mov_b64 exec, 1\n\tglobal_atomic_add %1, %2, %3 sc1\n\ts_mov_b64 exec, %0" \
        : "=&s"(ex_) : "v"((unsigned)(idx) * 4u), "v"(1u), "s"(flag) : "memory"); } while (0)
#define COMPLETE(k, ch) do { unsigned old_ = 0; if (lane == 0) old_ = __hip_atomic_fetch_add(cnt + ((k) & 7u), 1u, __ATOMIC_RELAXED, __HIP_MEMORY_SCOPE_WORKGROUP); \
        if (__builtin_amdgcn_readfirstlane(old_) == 7u) { FLAG_ADD(ch); if (lane == 0) { cnt[(k) & 7u] = 0u; *done = (k) + 1u; } } } while (0)
    auto loop = [&](auto leader) {
        constexpr bool LEADER = decltype(leader)::value;
        u32x4 va, vb; unsigned oa, ob; bool wa, wb;
        unsigned tfa = 0, tfb = 0;
        unsigned it = 0;
        int cc, pc = 0;
        { LOCATE((unsigned)j); cc = c; unsigned sf; PIECE((unsigned)j, sf, oa, wa); LD(va, wa, sf); }
#define ITER(cv, co, cw, nv, no, nw, tfnew, tfold) \
        unsigned w_; while (((w_ = tk[(it + 1u) & 7u]) >> 20) != it + 2u) __builtin_amdgcn_s_sleep(1); \
        const unsigned tn = __builtin_amdgcn_readfirstlane(w_) & 0xfffffu; \
        const bool nlive = tn < nbt; \
        int nc = 0; \
        if (nlive) { LOCATE(tn); nc = c; } \
        unsigned sf; PIECE(tn, sf, no, nw); if (!nlive) sf = 0xffffff00u; \
        LD(nv, nw, sf); \
        if constexpr (LEADER) { WAIT_V(1); asm volatile("" : "+v"(tfold)); FETCH_ASYNC(tfnew); \
            if (it >= 1u) { while (*done + 3u < it) __builtin_amdgcn_s_sleep(1); \
                const unsigned tv_ = __builtin_amdgcn_readfirstlane(tfold) - tbase; \
                if (lane == 0) tk[(it + 2u) & 7u] = (it + 3u) << 20 | (tv_ < 0xfffffu ? tv_ : 0xfffffu); } \
        } else { WAIT_V(1); } \
        if (it >= 1u) COMPLETE(it - 1u, pc); \
        if (CFG_XSLEEP > 0 && !same) __builtin_amdgcn_s_sleep(CFG_XSLEEP); \
        ST(cv, cw, co); \
        pc = cc; cc = nc; ++it; \
        if (!nlive) break;
        for (;;) {
            { ITER(va, oa, wa, vb, ob, wb, tfa, tfb) }
            { ITER(vb, ob, wb, va, oa, wa, tfb, tfa) }
        }
#undef ITER
        WAIT_V(0);
        asm volatile("" :: "v"(va), "v"(vb), "v"(tfa), "v"(tfb));
        COMPLETE(it - 1u, pc);
    };
    if (wid == 0) loop(std::true_type{}); else loop(std::false_type{});
#undef COMPLETE
#undef FLAG_ADD
#undef FETCH_ASYNC
#undef LD
#undef ST
#undef LOCATE
#undef PIECE
}

__device__ __forceinline__ void signal_done(const Params& P, int crank) {
    WAIT_V(0);
    BAR();
    if (threadIdx.x == 0) __hip_atomic_fetch_add((unsigned*)(P.ws[crank] + WS_DONE), 1u, __ATOMIC_RELAXED, __HIP_MEMORY_SCOPE_SYSTEM);
}

__global__ __launch_bounds__(512, 2) void ag_kernel(Params P) {
    extern __shared__ __attribute__((aligned(16))) unsigned char shm[];
    LAS unsigned char* lds = (LAS unsigned char*)shm;
    const int tid = threadIdx.x, lane = tid & 63;
    const int rank = P.rank, cls = rank >> 2, idx = rank & 3;
    char* myws = P.ws[rank];
    const int b = blockIdx.x;
    volatile LAS unsigned* cw = (volatile LAS unsigned*)(lds + LDS_CTL + 16);
    unsigned call;
    {
        if (tid == 0) *cw = __hip_atomic_fetch_add((unsigned*)(myws + WS_CTR) + b * 16, 1u, __ATOMIC_RELAXED, __HIP_MEMORY_SCOPE_AGENT) + 1u;
        __syncthreads();
        call = __builtin_amdgcn_readfirstlane(*cw);
    }
    STAMP(0);
    const size_t par = WS_STAGE + (size_t)(call & 1u) * PAR_BYTES;
    const char* xs = myws + par;
    const unsigned* flags = (const unsigned*)(myws + WS_FLAG);
    const unsigned needv = DBG(4) ? 0u : call;
    if (b < NPUSH) {
        if (!DBG(1)) push_role(lds, P, call, b);
    } else if (b >= GEMM0 && !DBG(8)) {
        const int g = b - GEMM0;
        constexpr int NA = 4 * RBA, NB = 4 * RBB;
        const int x = g & 7, s = g >> 3;
        const int a0 = x * NA / 8, a1 = (x + 1) * NA / 8, b0 = x * NB / 8, b1 = (x + 1) * NB / 8;
        const int na = a1 - a0;
        const int type = s < na ? 0 : 1;
        int t = type == 0 ? a0 + s : b0 + (s - na);
        if (type == 0) {
            constexpr int NLOC = 4 * (16 / NU), PERL = NLOC / 8;
            if (NA % 8 == 0 && NLOC % 8 == 0) t = s < PERL ? x * PERL + s : NLOC + x * (NA / 8 - PERL) + (s - PERL);
        }
        if (type == 0 || s - na < b1 - b0) {
            const int rb = t >> 2, cb = t & 3;
            TileDesc d;
            const int crank = (1 - cls) * 4 + cb;
            d.flag = flags; d.call1 = 0u; d.pbase = 0; d.anywait = false;
#pragma unroll
            for (int u = 0; u < NU; ++u) {
                int ui = rb * NU + u;
                const int nun = type == 0 ? UA : UB;
                const bool valid = ui < nun;
                if (!valid) ui = nun - 1;
                int src, su; const char* xp;
                const int nown = type == 0 ? 16 : U16;
                if (ui < nown) { src = rank; su = ui; xp = P.x + (size_t)ui * UNITB; }
                else if (ui < nown + 3 * U1) { const int sl = (ui - nown) / U1; su = (ui - nown) % U1; src = cls * 4 + ((idx + 1 + sl) & 3); xp = xs + (size_t)(sl * U16 + su) * UNITB; }
                else { int sl = (ui - nown - 3 * U1) / V16; const int jj = (ui - nown - 3 * U1) % V16; if (type == 1 && sl >= cb) ++sl;
                       src = (1 - cls) * 4 + sl; su = U16 + jj; xp = xs + (size_t)(3 * U16 + sl * V16 + jj) * UNITB; }
                d.xa[u] = xp;
                char* ob = type == 0 ? P.out[rank] + cb * 512 : P.out[crank] + idx * 512;
                d.out[u] = valid ? ob + (size_t)(src * MSH + su * 64) * WPITCH : nullptr;
                if (src != rank) { d.anywait = true; if (lane / (64 / (NU + 1)) == u) { d.flag = flags + src * FROW; d.call1 = needv; d.pbase = (src >> 2) == cls ? PBASE_S1 : PBASE_X; } }
            }
            d.throttle = type == 1;
            if (type == 0) { d.wb = P.w + cb * 512; d.wpitch = WPITCH; d.wbytes = KDIM * WPITCH - cb * 512; }
            else { d.wb = xs + XS_BYTES + (size_t)cb * WSQ_BYTES; d.wpitch = QPITCH; d.wbytes = (unsigned)WSQ_BYTES; d.anywait = true;
                   if (lane / (64 / (NU + 1)) == NU) { d.flag = flags + crank * FROW; d.call1 = needv; d.pbase = PBASE_X; } }
            gemm_tile(lds, d, P, call);
            if (type == 1) signal_done(P, crank);
        }
    }
    STAMP(3);
    if (U2 > 0 && !DBG(8)) {
        const int x = b & 7;
        for (;;) {
            __syncthreads();
            if (tid == 0) *cw = __hip_atomic_fetch_add((unsigned*)(myws + WS_QUEUE) + x * 32, 1u, __ATOMIC_RELAXED, __HIP_MEMORY_SCOPE_AGENT) - (call - 1u) * (unsigned)(NT2X + NWG / 8);
            __syncthreads();
            const unsigned tk = __builtin_amdgcn_readfirstlane(*cw);
            if (tk >= (unsigned)NT2X) break;
            const int cbk = 2 * x + (int)(tk & 1u), un = (int)(tk >> 1);
            const int sl = un / (U2 > 0 ? U2 : 1), su = U1 + un % (U2 > 0 ? U2 : 1);
            const int src = cls * 4 + ((idx + 1 + sl) & 3);
            const char* xa = xs + (size_t)(sl * U16 + su) * UNITB;
            const unsigned* xflag = flags + src * FROW + NCH1;
            const size_t orow = (size_t)(src * MSH + su * 64) * WPITCH;
            const bool own = cbk < 8;
            const int cb = (cbk - 8) >> 1, hf = cbk & 1, crank = own ? rank : (1 - cls) * 4 + cb;
            small_tile(lds, xa, own ? P.w + cbk * 256 : xs + XS_BYTES + (size_t)cb * WSQ_BYTES + hf * 256, own ? WPITCH : QPITCH,
                       own ? KDIM * WPITCH - cbk * 256 : (unsigned)WSQ_BYTES - hf * 256, P.out[crank] + orow + (own ? cbk * 256 : idx * 512 + hf * 256),
                       xflag, needv, flags + crank * FROW, own ? 0u : needv, P, call);
            if (!own) signal_done(P, crank);
            STAMP(7);
        }
    }
    if (b == 0 && !DBG(2)) {
        if (tid < 64) {
            unsigned* tmo = (unsigned*)(myws + WS_TMO);
            const bool ok = wave_wait((const unsigned*)(myws + WS_DONE), call * (unsigned)(4 * RBB + 4 * 3 * U2 * 2), tmo, call);
            if (!ok && lane == 0) *(unsigned short*)(P.out[rank] + (size_t)rank * MSH * WPITCH) = 0x7fc0u;
        }
        STAMP(4);
    }
}

}

#define AG_DBG 0
extern "C" void kernel_launch(int rank, int nranks, ncclComm_t comm, hipStream_t stream,
                              void* const* d_in, const int* in_sizes, int n_in,
                              void* d_out, int out_size, const int* in_dtypes, int out_dtype,
                              void* const* d_out_all, void* const* d_ws_all, size_t ws_bytes) {
    using namespace agk;
    (void)comm; (void)in_sizes; (void)n_in; (void)out_size; (void)in_dtypes; (void)out_dtype; (void)d_out;
    if (nranks != NR || ws_bytes < WS_NEED || rank < 0 || rank >= NR) { fprintf(stderr, "unsupported configuration\n"); return; }
    static bool init[64] = {};
    int dev = 0; HIP_CHECK(hipGetDevice(&dev));
    if (!init[dev & 63]) {
        HIP_CHECK(hipFuncSetAttribute((const void*)ag_kernel, hipFuncAttributeMaxDynamicSharedMemorySize, LDS_BYTES));
        init[dev & 63] = true;
    }
    Params P;
    P.x = (const char*)d_in[0]; P.w = (const char*)d_in[1];
    for (int r = 0; r < NR; ++r) { P.out[r] = (char*)d_out_all[r]; P.ws[r] = (char*)d_ws_all[r]; }
    P.rank = rank; P.dbg = AG_DBG;
    hipLaunchKernelGGL(ag_kernel, dim3(NWG), dim3(512), LDS_BYTES, stream, P);
    HIP_CHECK(hipGetLastError());
}
```

```cpp
#include <hip/hip_runtime.h>
#include <rccl/rccl.h>
#include <cstdio>
#include <cstdint>
#include <type_traits>

#define HIP_CHECK(expr) do { hipError_t _e = (expr); if (_e != hipSuccess) { \
    fprintf(stderr, "HIP error %s at %s:%d\n", hipGetErrorString(_e), __FILE__, __LINE__); \
    return; } } while (0)

namespace agk {

#define LAS __attribute__((address_space(3)))
typedef short bf16x8 __attribute__((ext_vector_type(8)));
typedef short s16x4 __attribute__((ext_vector_type(4)));
typedef float f32x4 __attribute__((ext_vector_type(4)));
typedef unsigned u32x4 __attribute__((ext_vector_type(4)));
typedef unsigned u32x2 __attribute__((ext_vector_type(2)));
typedef __amdgpu_buffer_rsrc_t rsrc_t;

constexpr int NR = 8, KDIM = 8192, MSH = 1024, NSH = 1024;
constexpr unsigned XPITCH = KDIM * 2;
constexpr unsigned WPITCH = NSH * 2;
constexpr unsigned QPITCH = 512;
constexpr unsigned UNITB = 64u * XPITCH;

constexpr int U16 = 11, V16 = 16 - U16;
constexpr int U2 = 2, U1 = U16 - U2;
constexpr int NU = 4;
constexpr int NJ = 16;
constexpr int NPUSH = 7 * NJ;
constexpr int UA = 16 + 3 * U1 + 4 * V16;
constexpr int UB = U16 + 3 * U1 + 3 * V16;
constexpr int RBA = (UA + NU - 1) / NU, RBB = (UB + NU - 1) / NU;
constexpr int NWG = 256;
constexpr int NGWG = 8 * ((4 * RBA + 7) / 8 + (4 * RBB + 7) / 8);
static_assert(NPUSH + NGWG <= NWG, "grid");
constexpr int GEMM0 = NWG - NGWG;
constexpr int NT2X = 3 * U2 * 2;

constexpr int NT = KDIM / 64;
constexpr int LG1 = 0, LG2 = 1;
__host__ __device__ constexpr int ch_body(int lg) { return (NT >> lg) - 1; }
__host__ __device__ constexpr int ch_count(int lg) { return ch_body(lg) + lg + 1; }
__host__ __device__ constexpr int ch_lg(int lg, int c) { return c < ch_body(lg) ? lg : (c - ch_body(lg) < lg ? lg - 1 - (c - ch_body(lg)) : 0); }
constexpr int NCH1 = ch_count(LG1), NCH2 = ch_count(LG2), NCH = NCH1 + NCH2;
__host__ __device__ constexpr int CH_LOGF(int c) { return c < NCH1 ? ch_lg(LG1, c) : ch_lg(LG2, c - NCH1); }
constexpr int ch_sum(int a, int b) { int s = 0; for (int i = a; i < b; ++i) s += 1 << CH_LOGF(i); return s; }
static_assert(ch_sum(0, NCH1) == NT && ch_sum(NCH1, NCH) == NT, "chunks");
constexpr unsigned long long ch_mask(int gen, int half) { unsigned long long m = 0; int s = 0;
    for (int i = gen ? NCH1 : 0; i < (gen ? NCH : NCH1); ++i) { if (s / 64 == half) m |= 1ull << (s % 64); s += 1 << CH_LOGF(i); } return m; }
constexpr unsigned long long CHM0 = ch_mask(0, 0), CHM1 = ch_mask(0, 1), CH2M0 = ch_mask(1, 0), CH2M1 = ch_mask(1, 1);
constexpr int FROW = 256;
static_assert(NCH <= FROW, "flags");

constexpr int PB = 8;
constexpr int PBASE_S1 = U1 * 8, PBASE_S2 = U2 * 8, PBASE_X = V16 * 8 + 32;
__host__ __device__ constexpr int nbatch(int pieces) { return (pieces + PB - 1) / PB; }
constexpr int nb_total(bool same) { int s = 0; for (int c = 0; c < NCH; ++c) { const int pb = c < NCH1 ? (same ? PBASE_S1 : PBASE_X) : (same ? PBASE_S2 : 0); s += nbatch(pb << CH_LOGF(c)); } return s; }
constexpr int NB_S = nb_total(true), NB_X = nb_total(false);

constexpr size_t WS_CTR = 0;
constexpr size_t WS_TMO = 16384;
constexpr size_t WS_DONE = 16384 + 256;
constexpr size_t WS_QUEUE = 16384 + 512;
constexpr size_t WS_TICK = 16384 + 2048;
constexpr size_t WS_FLAG = 32768;
constexpr size_t WS_STAGE = 1u << 20;
constexpr size_t XS_BYTES = (size_t)(3 * U16 + 4 * V16) * UNITB;
constexpr size_t WSQ_BYTES = (size_t)KDIM * QPITCH;
constexpr size_t PAR_BYTES = XS_BYTES + 4 * WSQ_BYTES;
constexpr size_t WS_NEED = WS_STAGE + 2 * PAR_BYTES;
constexpr unsigned long long DEADLINE = 100000000ull;

constexpr int LDS_STAGE = 2 * NU * 8192 + 65536;
constexpr int LDS_EPI = NU * 64 * 528;
constexpr int LDS_SMALL = 6 * 24576;
constexpr int cmax(int a, int b) { return a > b ? a : b; }
constexpr int LDS_CTL = cmax(cmax(LDS_STAGE, LDS_EPI), LDS_SMALL);
constexpr int LDS_BYTES = LDS_CTL + 256;

struct Params {
    const char* x; const char* w;
    char* out[NR]; char* ws[NR];
    int rank;
};

__device__ __forceinline__ unsigned cvt_pk_bf16(float lo, float hi) { unsigned r; asm("v_cvt_pk_bf16_f32 %0, %1, %2" : "=v"(r) : "v"(lo), "v"(hi)); return r; }
__device__ __forceinline__ float gelu1(float y) {
    const float c2 = -2.0f * 0.7978845608028654f * 1.4426950408889634f;
    const float a = y * __builtin_fmaf(y * y, 0.044715f * c2, c2);
    return y * __builtin_amdgcn_rcpf(1.0f + __builtin_amdgcn_exp2f(a));
}
__device__ __forceinline__ rsrc_t mk_rsrc(const void* p, unsigned bytes) { return __builtin_amdgcn_make_buffer_rsrc((void*)p, 0, bytes, 0x00020000); }

#define WAIT_V(n) asm volatile("s_waitcnt vmcnt(" #n ")" ::: "memory")
#define WAIT_L(n) asm volatile("s_waitcnt lgkmcnt(" #n ")" ::: "memory")
#define BAR() __builtin_amdgcn_s_barrier()
#define SCHED() __builtin_amdgcn_sched_barrier(0)
#define SYS_LOAD(p) __hip_atomic_load((p), __ATOMIC_RELAXED, __HIP_MEMORY_SCOPE_SYSTEM)
#define AGT_LOAD(p) __hip_atomic_load((p), __ATOMIC_RELAXED, __HIP_MEMORY_SCOPE_AGENT)
#define DMA(rs, ldsp, voff, soff) __builtin_amdgcn_raw_ptr_buffer_load_lds((rs), (LAS void*)(ldsp), 16, (voff), (soff), 0, 17)
#define TRR(dst, base, off) asm volatile("ds_read_b64_tr_b16 %0, %1 offset:%2" : "=v"(dst) : "v"(base), "i"(off) : "memory")

__device__ __forceinline__ bool wave_wait(const unsigned* p, unsigned need, unsigned* tmo, unsigned call) {
    const unsigned long long t0 = __builtin_amdgcn_s_memrealtime();
    for (unsigned n = 1;; ++n) {
        const unsigned v = SYS_LOAD(p);
        if (__all((int)(v - need) >= 0 || need == 0u)) return true;
        if ((n & 15u) == 0u && (__builtin_amdgcn_readfirstlane(AGT_LOAD(tmo)) == call || __builtin_amdgcn_s_memrealtime() - t0 > DEADLINE)) {
            __hip_atomic_store(tmo, call, __ATOMIC_RELAXED, __HIP_MEMORY_SCOPE_AGENT);
            return false;
        }
        __builtin_amdgcn_s_sleep(1);
    }
}

template <int WN, int NS>
__device__ __forceinline__ int window_wait(const unsigned* p, unsigned need, unsigned* tmo, unsigned call) {
    const unsigned long long t0 = __builtin_amdgcn_s_memrealtime();
    for (unsigned n = 1;; ++n) {
        const unsigned v = SYS_LOAD(p);
        unsigned long long m = __ballot((int)(v - need) >= 0 || need == 0u), a = m;
#pragma unroll
        for (int s = 1; s < NS; ++s) a &= m >> (s * WN);
        a &= (1ull << WN) - 1ull;
        if (a & 1ull) return (int)__builtin_ctzll(~a);
        if ((n & 15u) == 0u && (__builtin_amdgcn_readfirstlane(AGT_LOAD(tmo)) == call || __builtin_amdgcn_s_memrealtime() - t0 > DEADLINE)) {
            __hip_atomic_store(tmo, call, __ATOMIC_RELAXED, __HIP_MEMORY_SCOPE_AGENT);
            return WN;
        }
        __builtin_amdgcn_s_sleep(1);
    }
}

__device__ __forceinline__ void store_out(char* o, u32x4 v) { asm volatile("global_store_dwordx4 %0, %1, off sc0 sc1\n\ts_nop 1" :: "v"(o), "v"(v) : "memory"); }

struct TileDesc {
    const char* xa[NU];
    const char* wb; unsigned wpitch, wbytes;
    char* out[NU];
    const unsigned* flag;
    unsigned call1;
    int pbase;
    bool anywait, throttle;
};

__device__ __forceinline__ void gemm_tile(LAS unsigned char* lds, const TileDesc& d, const Params& P, unsigned call) {
    constexpr int MT = NU * 2;
    constexpr int ABUF = NU * 8192;
    constexpr int BOFF = 2 * ABUF;
    const int tid = threadIdx.x, wid = __builtin_amdgcn_readfirstlane(tid >> 6), lane = tid & 63;
    const int wr = wid >> 2, wc = wid & 3, fr = lane & 15, fq = lane >> 4;
    unsigned* tmo = (unsigned*)(P.ws[P.rank] + WS_TMO);
    volatile LAS unsigned* ready = (volatile LAS unsigned*)(lds + LDS_CTL);
    const unsigned voffA = (unsigned)(8 * wid + (lane >> 3)) * XPITCH + (unsigned)(((lane & 7) ^ (((wid & 1) << 2) | (lane >> 4))) * 16);
    const unsigned fB = (unsigned)((lane >> 4) | (((wid >> 1) & 1) << 2));
    const unsigned voffB = (unsigned)(4 * wid + (lane >> 4)) * d.wpitch + ((((unsigned)(lane & 15) >> 1) ^ fB) * 32u + (unsigned)(lane & 1) * 16u);
    const unsigned ldsw = (unsigned)wid * 1024u;
    const unsigned aoff0 = (unsigned)(wr * (NU * 32) + fr) * 128u + (unsigned)((fq ^ (fr >> 1)) * 16);
    const unsigned aoff1 = aoff0 ^ 64u;
    const unsigned q = (unsigned)(fr >> 2), p = (unsigned)(fr & 3);
    const unsigned boff0 = (unsigned)(size_t)lds + BOFF + (unsigned)(8 * fq + q) * 256u + (((unsigned)(2 * wc) ^ (q | ((unsigned)(fq & 1) << 2))) * 32u) + p * 8u;
    const unsigned boff1 = boff0 ^ 32u;
    rsrc_t ra[NU];
#pragma unroll
    for (int u = 0; u < NU; ++u) ra[u] = mk_rsrc(d.xa[u], UNITB);
    const rsrc_t rb = mk_rsrc(d.wb, d.wbytes);
    const unsigned wstep = __builtin_amdgcn_readfirstlane(64u * d.wpitch), whalf = wstep >> 1;

    f32x4 acc[2][MT][2];
#pragma unroll
    for (int b = 0; b < 2; ++b)
#pragma unroll
        for (int m = 0; m < MT; ++m)
#pragma unroll
            for (int n = 0; n < 2; ++n) acc[b][m][n] = (f32x4){0.f, 0.f, 0.f, 0.f};
    bf16x8 af[MT][2], bfr[2][2]; s16x4 tb[8];

#define STAGE_A(buf, t) do { _Pragma("unroll") for (int _i = 0; _i < NU; ++_i) DMA(ra[_i], lds + (buf) * ABUF + _i * 8192 + ldsw, voffA, (unsigned)(t) * 128u); } while (0)
#define STAGE_B(buf, h, t) do { _Pragma("unroll") for (int _i = 0; _i < 2; ++_i) \
        DMA(rb, lds + BOFF + ((buf) * 2 + (h)) * 16384 + _i * 8192 + ldsw, voffB, (unsigned)(t) * wstep + _i * whalf + (h) * 256u); } while (0)
#define LDA(buf) do { _Pragma("unroll") for (int m = 0; m < MT; ++m) { \
        af[m][0] = *(const LAS bf16x8*)(lds + (buf) * ABUF + aoff0 + m * 2048); \
        af[m][1] = *(const LAS bf16x8*)(lds + (buf) * ABUF + aoff1 + m * 2048); } } while (0)
#define LDB(buf, h) do { \
        TRR(tb[0], boff0, ((buf) * 2 + (h)) * 16384); TRR(tb[1], boff0, ((buf) * 2 + (h)) * 16384 + 1024); \
        TRR(tb[2], boff1, ((buf) * 2 + (h)) * 16384); TRR(tb[3], boff1, ((buf) * 2 + (h)) * 16384 + 1024); \
        TRR(tb[4], boff0, ((buf) * 2 + (h)) * 16384 + 8192); TRR(tb[5], boff0, ((buf) * 2 + (h)) * 16384 + 8192 + 1024); \
        TRR(tb[6], boff1, ((buf) * 2 + (h)) * 16384 + 8192); TRR(tb[7], boff1, ((buf) * 2 + (h)) * 16384 + 8192 + 1024); } while (0)
#define WAITB() do { asm volatile("s_waitcnt lgkmcnt(0)" : "+v"(tb[0]), "+v"(tb[1]), "+v"(tb[2]), "+v"(tb[3]), "+v"(tb[4]), "+v"(tb[5]), "+v"(tb[6]), "+v"(tb[7]) :: "memory"); \
        bfr[0][0] = __builtin_shufflevector(tb[0], tb[1], 0, 1, 2, 3, 4, 5, 6, 7); bfr[1][0] = __builtin_shufflevector(tb[2], tb[3], 0, 1, 2, 3, 4, 5, 6, 7); \
        bfr[0][1] = __builtin_shufflevector(tb[4], tb[5], 0, 1, 2, 3, 4, 5, 6, 7); bfr[1][1] = __builtin_shufflevector(tb[6], tb[7], 0, 1, 2, 3, 4, 5, 6, 7); } while (0)
#define MMA(bj) do { __builtin_amdgcn_s_setprio(1); \
        _Pragma("unroll") for (int kk = 0; kk < 2; ++kk) _Pragma("unroll") for (int m = 0; m < MT; ++m) _Pragma("unroll") for (int n = 0; n < 2; ++n) \
            acc[bj][m][n] = __builtin_amdgcn_mfma_f32_16x16x32_bf16(bfr[n][kk], af[m][kk], acc[bj][m][n], 0, 0, 0); \
        __builtin_amdgcn_s_setprio(0); } while (0)
#define WAITN() do { if constexpr (NU == 3) { WAIT_V(7); } else if constexpr (NU == 2) { WAIT_V(6); } else { WAIT_V(8); } } while (0)
    constexpr int NS = NU + 1, WN = 64 / NS;
    int cnext = 0, known = 0;
#define POLL() do { const int ck_ = cnext + lane % WN; const bool w_ = ck_ < NCH1 && lane < NS * WN; const int cc_ = w_ ? ck_ : 0; \
        known = cnext + window_wait<WN, NS>(d.flag + cc_, w_ ? d.call1 * (unsigned)nbatch(d.pbase << CH_LOGF(cc_)) : 0u, tmo, call); } while (0)
#define GATE() do { \
        if (known <= cnext) { \
            if (wid == 0) { POLL(); if (lane == 0) *ready = (unsigned)known; } \
            else { unsigned r_; while ((r_ = *ready) <= (unsigned)cnext) __builtin_amdgcn_s_sleep(1); known = (int)__builtin_amdgcn_readfirstlane(r_); } \
        } \
        ++cnext; } while (0)
#define CHECK(tt) do { if (d.anywait && (tt) < NT && ((((tt) < 64 ? CHM0 : CHM1) >> ((tt) & 63)) & 1ull)) GATE(); } while (0)

    if (d.anywait) {
        constexpr int C01 = LG1 == 0 ? 2 : 1;
        cnext = C01 - 1;
        if (wid == 0) { if (C01 == 2) { cnext = 0; POLL(); cnext = 1; } if (known <= cnext) POLL(); if (lane == 0) *ready = (unsigned)known; }
        WAIT_L(0); BAR();
        known = (int)__builtin_amdgcn_readfirstlane(*ready);
        cnext = C01;
    }
    STAGE_A(0, 0); STAGE_B(0, 0, 0); STAGE_B(0, 1, 0);
    STAGE_A(1, 1); STAGE_B(1, 0, 1);
    if (wr == 1) BAR();
    if constexpr (NU == 3) { WAIT_V(5); } else if constexpr (NU == 2) { WAIT_V(4); } else { WAIT_V(6); }
    BAR(); BAR();
#define KTILE(b, t) do { \
        LDB(b, 0); LDA(b); \
        STAGE_B((b) ^ 1, 1, min((t) + 1, NT - 1)); \
        WAITN(); WAITB(); BAR(); SCHED(); MMA(0); SCHED(); BAR(); \
        CHECK((t) + 2); \
        LDB(b, 1); \
        STAGE_A(b, min((t) + 2, NT - 1)); STAGE_B(b, 0, min((t) + 2, NT - 1)); \
        WAITN(); WAITB(); BAR(); SCHED(); MMA(1); SCHED(); BAR(); } while (0)
    for (int t = 0; t < NT; t += 2) {
        KTILE(0, t);
        KTILE(1, t + 1);
    }
    WAIT_V(0);
    if (wr == 0) BAR();
    BAR();
    constexpr int EP = 528;
    static_assert(NU == 4, "epilogue slices");
    const bool bad = AGT_LOAD(tmo) == call;
#define EPI_SLICE(m0) do { _Pragma("unroll") for (int bj = 0; bj < 2; ++bj) _Pragma("unroll") for (int m = (m0); m < (m0) + MT / 2; ++m) _Pragma("unroll") for (int n = 0; n < 2; ++n) { \
                const f32x4 v = acc[bj][m][n]; \
                u32x2 w; w.x = cvt_pk_bf16(gelu1(v[0]), gelu1(v[1])); w.y = cvt_pk_bf16(gelu1(v[2]), gelu1(v[3])); \
                *(LAS u32x2*)(lds + (wr * (NU * 32) + m * 16 + fr) * EP + (bj * 128 + wc * 32 + n * 16 + fq * 4) * 2) = w; } \
        WAIT_L(0); BAR(); } while (0)
#define EPI_STORE(u, i) do { if (d.out[u] != nullptr) { \
                const int r = ((i) * 8 + wid) * 2 + (lane >> 5); \
                u32x4 v = *(const LAS u32x4*)(lds + ((u) * 64 + r) * EP + (lane & 31) * 16); \
                if (bad) v = (u32x4){0x7fc07fc0u, 0x7fc07fc0u, 0x7fc07fc0u, 0x7fc07fc0u}; \
                if (d.throttle) WAIT_V(0);        \
                store_out(d.out[u] + (size_t)r * WPITCH + (lane & 31) * 16, v); } } while (0)
    EPI_SLICE(0);
    EPI_STORE(0, 0);
    EPI_SLICE(MT / 2);
#pragma unroll
    for (int i = 1; i < 4; ++i) EPI_STORE(0, i);
#pragma unroll
    for (int i = 0; i < 4; ++i) EPI_STORE(2, i);
#pragma unroll
    for (int i = 0; i < 4; ++i) EPI_STORE(1, i);
#pragma unroll
    for (int i = 0; i < 4; ++i) EPI_STORE(3, i);
#undef EPI_SLICE
#undef EPI_STORE
#undef STAGE_A
#undef STAGE_B
#undef LDA
#undef LDB
#undef MMA
#undef KTILE
#undef WAITB
#undef WAITN
#undef GATE
#undef CHECK
#undef POLL
}

__device__ __forceinline__ void small_tile(LAS unsigned char* lds, const char* xa, const char* wb, unsigned wpitch, unsigned wbytes, char* out,
                                           const unsigned* xflag, unsigned xcall, const unsigned* wflag, unsigned wcall, const Params& P, unsigned call) {
    const int tid = threadIdx.x, wid = __builtin_amdgcn_readfirstlane(tid >> 6), lane = tid & 63;
    const int g = wid >> 2, wq = wid & 3, wm = wq >> 1, wn = wq & 1, fr = lane & 15, fq = lane >> 4;
    unsigned* tmo = (unsigned*)(P.ws[P.rank] + WS_TMO);
    volatile LAS unsigned* ready = (volatile LAS unsigned*)(lds + LDS_CTL + 32) + g;
    const unsigned voffA = (unsigned)(8 * wq + (lane >> 3)) * XPITCH + (unsigned)(((lane & 7) ^ (((wq & 1) << 2) | (lane >> 4))) * 16);
    const unsigned fB = (unsigned)((lane >> 4) | (((wq >> 1) & 1) << 2));
    const unsigned voffB = (unsigned)(4 * wq + (lane >> 4)) * wpitch + ((((unsigned)(lane & 15) >> 1) ^ fB) * 32u + (unsigned)(lane & 1) * 16u);
    const unsigned gbase = (unsigned)g * (3u * 24576u);
    const unsigned ldsw = gbase + (unsigned)wq * 1024u;
    const unsigned aoff0 = gbase + (unsigned)(wm * 32 + fr) * 128u + (unsigned)((fq ^ (fr >> 1)) * 16);
    const unsigned aoff1 = aoff0 ^ 64u;
    const unsigned q = (unsigned)(fr >> 2), p = (unsigned)(fr & 3);
    const unsigned boff = (unsigned)(size_t)lds + gbase + 8192u + (unsigned)(8 * fq + q) * 256u + (((unsigned)(4 * wn) ^ (q | ((unsigned)(fq & 1) << 2))) * 32u) + p * 8u;
    const unsigned bo0 = boff, bo1 = boff ^ 32u, bo2 = boff ^ 64u, bo3 = boff ^ 96u;
    const rsrc_t ra = mk_rsrc(xa, UNITB), rb = mk_rsrc(wb, wbytes);
    const unsigned wstep = __builtin_amdgcn_readfirstlane(64u * wpitch), wq16 = wstep >> 2;
    f32x4 acc[2][4];
#pragma unroll
    for (int m = 0; m < 2; ++m)
#pragma unroll
        for (int n = 0; n < 4; ++n) acc[m][n] = (f32x4){0.f, 0.f, 0.f, 0.f};
#define STAGE(s, t) do { DMA(ra, lds + (s) * 24576 + ldsw, voffA, (unsigned)(t) * 128u); DMA(ra, lds + (s) * 24576 + 4096 + ldsw, voffA, (unsigned)(t) * 128u + 32u * XPITCH); \
        _Pragma("unroll") for (int _j = 0; _j < 4; ++_j) DMA(rb, lds + (s) * 24576 + 8192 + _j * 4096 + ldsw, voffB, (unsigned)(t) * wstep + _j * wq16); } while (0)
#define DMA_A(s, t, i) DMA(ra, lds + (s) * 24576 + (i) * 4096 + ldsw, voffA, (unsigned)(t) * 128u + (i) * 32u * XPITCH)
#define DMA_B(s, t, j) DMA(rb, lds + (s) * 24576 + 8192 + (j) * 4096 + ldsw, voffB, (unsigned)(t) * wstep + (j) * wq16)
    int known = 0;
#define CIDX(tt) ((tt) < 64 ? __builtin_popcountll(CH2M0 & ((2ull << (tt)) - 1ull)) - 1 : __builtin_popcountll(CH2M0) + __builtin_popcountll(CH2M1 & ((2ull << ((tt) - 64)) - 1ull)) - 1)
#define POLL(ci) do { const int ck_ = (ci) + lane; const bool w_ = ck_ < NCH2 && lane < 32; const int cc_ = w_ ? ck_ : 0; \
        known = (ci) + window_wait<32, 1>(xflag + cc_, w_ ? xcall * (unsigned)nbatch(PBASE_S2 << CH_LOGF(NCH1 + cc_)) : 0u, tmo, call); } while (0)
#define CHECK(tt) do { if ((tt) < NT) { const int ci_ = CIDX(tt); if (known <= ci_) { \
            if (wq == 0) { POLL(ci_); if (lane == 0) *ready = (unsigned)known; } \
            else { unsigned r_; while ((r_ = *ready) <= (unsigned)ci_) __builtin_amdgcn_s_sleep(1); known = (int)__builtin_amdgcn_readfirstlane(r_); } } } } while (0)
    BAR();
    if (wid == 0) {
        for (int c0 = 0; c0 < NCH1; c0 += 64) { const int lc = c0 + lane < NCH1 ? c0 + lane : 0;
            wave_wait(wflag + lc, c0 + lane < NCH1 ? wcall * (unsigned)nbatch(PBASE_X << CH_LOGF(lc)) : 0u, tmo, call); }
        known = 0; while (known <= CIDX(3)) POLL(known);
        if (lane == 0) { ready[0] = (unsigned)known; ready[1] = (unsigned)known; }
    }
    WAIT_L(0); BAR();
    known = (int)__builtin_amdgcn_readfirstlane(*ready);
    STAGE(0, g); STAGE(1, g + 2);
    WAIT_V(6); BAR();
    if (g == 1) BAR();
    s16x4 tb[16]; bf16x8 af[2][2];
#define STEP(s, t) do { \
        CHECK((t) + 4); \
        TRR(tb[0], bo0, (s) * 24576); TRR(tb[1], bo0, (s) * 24576 + 1024); TRR(tb[2], bo1, (s) * 24576); TRR(tb[3], bo1, (s) * 24576 + 1024); \
        TRR(tb[4], bo2, (s) * 24576); TRR(tb[5], bo2, (s) * 24576 + 1024); TRR(tb[6], bo3, (s) * 24576); TRR(tb[7], bo3, (s) * 24576 + 1024); \
        TRR(tb[8], bo0, (s) * 24576 + 8192); TRR(tb[9], bo0, (s) * 24576 + 9216); TRR(tb[10], bo1, (s) * 24576 + 8192); TRR(tb[11], bo1, (s) * 24576 + 9216); \
        TRR(tb[12], bo2, (s) * 24576 + 8192); TRR(tb[13], bo2, (s) * 24576 + 9216); TRR(tb[14], bo3, (s) * 24576 + 8192); TRR(tb[15], bo3, (s) * 24576 + 9216); \
        _Pragma("unroll") for (int m = 0; m < 2; ++m) { af[m][0] = *(const LAS bf16x8*)(lds + (s) * 24576 + aoff0 + m * 2048); af[m][1] = *(const LAS bf16x8*)(lds + (s) * 24576 + aoff1 + m * 2048); } \
        const int tn_ = min((t) + 4, NT - 2 + g); \
        DMA_A(((s) + 2) % 3, tn_, 0); DMA_B(((s) + 2) % 3, tn_, 0); DMA_B(((s) + 2) % 3, tn_, 1);       \
        asm volatile("s_waitcnt lgkmcnt(0)" : "+v"(tb[0]), "+v"(tb[1]), "+v"(tb[2]), "+v"(tb[3]), "+v"(tb[4]), "+v"(tb[5]), "+v"(tb[6]), "+v"(tb[7]), \
                     "+v"(tb[8]), "+v"(tb[9]), "+v"(tb[10]), "+v"(tb[11]), "+v"(tb[12]), "+v"(tb[13]), "+v"(tb[14]), "+v"(tb[15]) :: "memory"); \
        BAR(); SCHED(); \
        __builtin_amdgcn_s_setprio(1); \
        _Pragma("unroll") for (int kk = 0; kk < 2; ++kk) _Pragma("unroll") for (int n = 0; n < 4; ++n) { \
            const bf16x8 b_ = __builtin_shufflevector(tb[kk * 8 + n * 2], tb[kk * 8 + n * 2 + 1], 0, 1, 2, 3, 4, 5, 6, 7); \
            _Pragma("unroll") for (int m = 0; m < 2; ++m) acc[m][n] = __builtin_amdgcn_mfma_f32_16x16x32_bf16(b_, af[m][kk], acc[m][n], 0, 0, 0); \
            if ((n & 1) == 1) { SCHED(); \
                if (kk == 0 && n == 1) { DMA_A(((s) + 2) % 3, tn_, 1); } \
                if (kk == 0 && n == 3) { DMA_B(((s) + 2) % 3, tn_, 2); } \
                if (kk == 1 && n == 1) { DMA_B(((s) + 2) % 3, tn_, 3); } \
                SCHED(); } } \
        __builtin_amdgcn_s_setprio(0); \
        SCHED(); WAIT_V(6); BAR(); } while (0)
    for (int i = 0; i < 63; i += 3) {
        STEP(0, g + 2 * i); STEP(1, g + 2 * i + 2); STEP(2, g + 2 * i + 4);
    }
    STEP(0, g + 126);
    WAIT_V(0);
    if (g == 0) BAR();
    BAR();
    constexpr int EP = 272;
    const bool bad = AGT_LOAD(tmo) == call;
    if (g == 1) {
#pragma unroll
        for (int m = 0; m < 2; ++m)
#pragma unroll
            for (int n = 0; n < 4; ++n) *(LAS f32x4*)(lds + 32768 + ((wq * 8 + m * 4 + n) * 64 + lane) * 16) = acc[m][n];
    }
    WAIT_L(0); BAR();
    if (g == 0) {
#pragma unroll
        for (int m = 0; m < 2; ++m)
#pragma unroll
            for (int n = 0; n < 4; ++n) {
                const f32x4 v = acc[m][n] + *(const LAS f32x4*)(lds + 32768 + ((wq * 8 + m * 4 + n) * 64 + lane) * 16);
                u32x2 w; w.x = cvt_pk_bf16(gelu1(v[0]), gelu1(v[1])); w.y = cvt_pk_bf16(gelu1(v[2]), gelu1(v[3]));
                *(LAS u32x2*)(lds + (wm * 32 + m * 16 + fr) * EP + (wn * 64 + n * 16 + fq * 4) * 2) = w;
            }
    }
    WAIT_L(0); BAR();
#pragma unroll
    for (int i = 0; i < 2; ++i) {
        const int r = (i * 8 + wid) * 4 + (lane >> 4);
        u32x4 v = *(const LAS u32x4*)(lds + r * EP + (lane & 15) * 16);
        if (bad) v = (u32x4){0x7fc07fc0u, 0x7fc07fc0u, 0x7fc07fc0u, 0x7fc07fc0u};
        store_out(out + (size_t)r * WPITCH + (lane & 15) * 16, v);
    }
#undef STAGE
#undef DMA_A
#undef DMA_B
#undef CIDX
#undef CHECK
#undef POLL
#undef STEP
}

__device__ __forceinline__ void push_role(LAS unsigned char* lds, const Params& P, unsigned call, int pw) {
    const int tid = threadIdx.x, wid = __builtin_amdgcn_readfirstlane(tid >> 6), lane = tid & 63;
    const int rank = P.rank, cls = rank >> 2, idx = rank & 3;
    const int di = pw % 7, j = pw / 7;
    const bool same = di < 3;
    const int q = same ? (cls * 4 + ((idx + 1 + di) & 3)) : ((1 - cls) * 4 + (di - 3));
    const int qidx = q & 3;
    const size_t par = WS_STAGE + (size_t)(call & 1u) * PAR_BYTES;
    const int slot = same ? ((idx - qidx - 1) & 3) : idx;
    const size_t xdst_off = par + (same ? (size_t)slot * U16 * UNITB : (size_t)(3 * U16 + slot * V16) * UNITB);
    const size_t wdst_off = par + XS_BYTES + (size_t)slot * WSQ_BYTES;
    const unsigned xbytes = (unsigned)(same ? U16 : V16) * UNITB;
    const char* xsrc = P.x + (same ? 0 : (size_t)U16 * UNITB);
    const char* wsrc = P.w + qidx * 512;
    char* wsq = P.ws[q];
    unsigned* flag = (unsigned*)(wsq + WS_FLAG) + rank * FROW;
    const unsigned nbt = same ? NB_S : NB_X;
    const unsigned njd = NJ;
    unsigned* tick = (unsigned*)(P.ws[rank] + WS_TICK) + di * 32;
    const unsigned tbase = (call - 1u) * nbt - 3u * njd;
    volatile LAS unsigned* tk = (volatile LAS unsigned*)lds;
    LAS unsigned* cnt = (LAS unsigned*)(lds + 32);
    volatile LAS unsigned* done = (volatile LAS unsigned*)(lds + 64);
    if (tid < 8) { cnt[tid] = 0u; tk[tid] = tid < 3 ? (((unsigned)tid + 1u) << 20 | ((unsigned)j + (unsigned)tid * njd)) : 0u; }
    if (tid == 8) *done = 0u;
    __syncthreads();
    int c = 0, kt0 = 0, row0 = 0, pbase = same ? PBASE_S1 : PBASE_X, nx0 = same ? PBASE_S1 : V16 * 8;
    unsigned cb0 = 0;
#define LOCATE(tt) do { for (;;) { const unsigned nbc_ = (unsigned)nbatch(pbase << CH_LOGF(c)); if ((tt) < cb0 + nbc_) break; \
            cb0 += nbc_; kt0 += 1 << CH_LOGF(c); ++c; \
            if (c == NCH1) { kt0 = 0; row0 = U1 * 64; pbase = PBASE_S2; nx0 = PBASE_S2; } } } while (0)
#define PIECE(tt, soff, doff, w) do { const int lg_ = CH_LOGF(c), ptot_ = pbase << lg_, nx_ = nx0 << lg_; const int i_ = (int)((tt) - cb0) * PB + wid; \
            soff = 0xffffff00u; doff = 0xffffff00u; w = false; \
            if (i_ < nx_) { const unsigned o_ = (unsigned)i_ * 1024u + (unsigned)lane * 16u; \
                const unsigned row_ = (unsigned)row0 + (o_ >> (7 + lg_)), col_ = o_ & ((128u << lg_) - 1u); soff = doff = row_ * XPITCH + (unsigned)kt0 * 128u + col_; } \
            else if (i_ < ptot_) { const unsigned k_ = (unsigned)kt0 * 64u + (unsigned)(i_ - nx_) * 2u + (unsigned)(lane >> 5); \
                soff = k_ * WPITCH + (unsigned)(lane & 31) * 16u; doff = k_ * QPITCH + (unsigned)(lane & 31) * 16u; w = true; } } while (0)
    auto desc = [](const void* p, unsigned bytes) { const unsigned long long a = (unsigned long long)p; u32x4 r;
        r.x = __builtin_amdgcn_readfirstlane((unsigned)a); r.y = __builtin_amdgcn_readfirstlane((unsigned)(a >> 32) & 0xffffu); r.z = __builtin_amdgcn_readfirstlane(bytes); r.w = 0x00020000u; return r; };
    const u32x4 dxs = desc(xsrc, xbytes), dxd = desc(wsq + xdst_off, xbytes), dws = desc(wsrc, (unsigned)KDIM * WPITCH - (unsigned)qidx * 512u), dwd = desc(wsq + wdst_off, (unsigned)WSQ_BYTES);
#define LD(dst, w, off) asm volatile("buffer_load_dwordx4 %0, %1, %2, 0 offen" : "=v"(dst) : "v"(off), "s"((w) ? dws : dxs) : "memory")
#define ST(src, w, off) asm volatile("buffer_store_dwordx4 %0, %1, %2, 0 offen sc0 sc1\n\ts_nop 1" :: "v"(src), "v"(off), "s"((w) ? dwd : dxd) : "memory")
#define FETCH_ASYNC(dst) do { unsigned long long ex_; asm volatile("s_mov_b64 %1, exec\n\ts_mov_b64 exec, 1\n\tglobal_atomic_add %0, %2, %3, %4 sc0\n\ts_mov_b64 exec, %1" \
        : "=&v"(dst), "=&s"(ex_) : "v"(0u), "v"(1u), "s"(tick) : "memory"); } while (0)
#define FLAG_ADD(idx) do { unsigned long long ex_; asm volatile("s_mov_b64 %0, exec\n\ts_mov_b64 exec, 1\n\tglobal_atomic_add %1, %2, %3 sc1\n\ts_mov_b64 exec, %0" \
        : "=&s"(ex_) : "v"((unsigned)(idx) * 4u), "v"(1u), "s"(flag) : "memory"); } while (0)
#define COMPLETE(k, ch) do { unsigned old_ = 0; if (lane == 0) old_ = __hip_atomic_fetch_add(cnt + ((k) & 7u), 1u, __ATOMIC_RELAXED, __HIP_MEMORY_SCOPE_WORKGROUP); \
        if (__builtin_amdgcn_readfirstlane(old_) == 7u) { FLAG_ADD(ch); if (lane == 0) { cnt[(k) & 7u] = 0u; *done = (k) + 1u; } } } while (0)
    auto loop = [&](auto leader) {
        constexpr bool LEADER = decltype(leader)::value;
        u32x4 va, vb; unsigned oa, ob; bool wa, wb;
        unsigned tfa = 0, tfb = 0;
        unsigned it = 0;
        int cc, pc = 0;
        { LOCATE((unsigned)j); cc = c; unsigned sf; PIECE((unsigned)j, sf, oa, wa); LD(va, wa, sf); }
#define ITER(cv, co, cw, nv, no, nw, tfnew, tfold) \
        unsigned w_; while (((w_ = tk[(it + 1u) & 7u]) >> 20) != it + 2u) __builtin_amdgcn_s_sleep(1); \
        const unsigned tn = __builtin_amdgcn_readfirstlane(w_) & 0xfffffu; \
        const bool nlive = tn < nbt; \
        int nc = 0; \
        if (nlive) { LOCATE(tn); nc = c; } \
        unsigned sf; PIECE(tn, sf, no, nw); if (!nlive) sf = 0xffffff00u; \
        LD(nv, nw, sf); \
        if constexpr (LEADER) { WAIT_V(1); asm volatile("" : "+v"(tfold)); FETCH_ASYNC(tfnew); \
            if (it >= 1u) { while (*done + 3u < it) __builtin_amdgcn_s_sleep(1); \
                const unsigned tv_ = __builtin_amdgcn_readfirstlane(tfold) - tbase; \
                if (lane == 0) tk[(it + 2u) & 7u] = (it + 3u) << 20 | (tv_ < 0xfffffu ? tv_ : 0xfffffu); } \
        } else { WAIT_V(1); } \
        if (it >= 1u) COMPLETE(it - 1u, pc); \
        ST(cv, cw, co); \
        pc = cc; cc = nc; ++it; \
        if (!nlive) break;
        for (;;) {
            { ITER(va, oa, wa, vb, ob, wb, tfa, tfb) }
            { ITER(vb, ob, wb, va, oa, wa, tfb, tfa) }
        }
#undef ITER
        WAIT_V(0);
        asm volatile("" :: "v"(va), "v"(vb), "v"(tfa), "v"(tfb));
        COMPLETE(it - 1u, pc);
    };
    if (wid == 0) loop(std::true_type{}); else loop(std::false_type{});
#undef COMPLETE
#undef FLAG_ADD
#undef FETCH_ASYNC
#undef LD
#undef ST
#undef LOCATE
#undef PIECE
}

__device__ __forceinline__ void signal_done(const Params& P, int crank) {
    WAIT_V(0);
    BAR();
    if (threadIdx.x == 0) __hip_atomic_fetch_add((unsigned*)(P.ws[crank] + WS_DONE), 1u, __ATOMIC_RELAXED, __HIP_MEMORY_SCOPE_SYSTEM);
}

__global__ __launch_bounds__(512, 2) void ag_kernel(Params P) {
    extern __shared__ __attribute__((aligned(16))) unsigned char shm[];
    LAS unsigned char* lds = (LAS unsigned char*)shm;
    const int tid = threadIdx.x, lane = tid & 63;
    const int rank = P.rank, cls = rank >> 2, idx = rank & 3;
    char* myws = P.ws[rank];
    const int b = blockIdx.x;
    volatile LAS unsigned* cw = (volatile LAS unsigned*)(lds + LDS_CTL + 16);
    unsigned call;
    {
        if (tid == 0) *cw = __hip_atomic_fetch_add((unsigned*)(myws + WS_CTR) + b * 16, 1u, __ATOMIC_RELAXED, __HIP_MEMORY_SCOPE_AGENT) + 1u;
        __syncthreads();
        call = __builtin_amdgcn_readfirstlane(*cw);
    }
    const size_t par = WS_STAGE + (size_t)(call & 1u) * PAR_BYTES;
    const char* xs = myws + par;
    const unsigned* flags = (const unsigned*)(myws + WS_FLAG);
    const unsigned needv = call;
    if (b < NPUSH) {
        push_role(lds, P, call, b);
    } else if (b >= GEMM0) {
        const int g = b - GEMM0;
        constexpr int NA = 4 * RBA, NB = 4 * RBB;
        const int x = g & 7, s = g >> 3;
        const int a0 = x * NA / 8, a1 = (x + 1) * NA / 8, b0 = x * NB / 8, b1 = (x + 1) * NB / 8;
        const int na = a1 - a0;
        const int type = s < na ? 0 : 1;
        int t = type == 0 ? a0 + s : b0 + (s - na);
        if (type == 0) {
            constexpr int NLOC = 4 * (16 / NU), PERL = NLOC / 8;
            if (NA % 8 == 0 && NLOC % 8 == 0) t = s < PERL ? x * PERL + s : NLOC + x * (NA / 8 - PERL) + (s - PERL);
        }
        if (type == 0 || s - na < b1 - b0) {
            const int rb = t >> 2, cb = t & 3;
            TileDesc d;
            const int crank = (1 - cls) * 4 + cb;
            d.flag = flags; d.call1 = 0u; d.pbase = 0; d.anywait = false;
#pragma unroll
            for (int u = 0; u < NU; ++u) {
                int ui = rb * NU + u;
                const int nun = type == 0 ? UA : UB;
                const bool valid = ui < nun;
                if (!valid) ui = nun - 1;
                int src, su; const char* xp;
                const int nown = type == 0 ? 16 : U16;
                if (ui < nown) { src = rank; su = ui; xp = P.x + (size_t)ui * UNITB; }
                else if (ui < nown + 3 * U1) { const int sl = (ui - nown) / U1; su = (ui - nown) % U1; src = cls * 4 + ((idx + 1 + sl) & 3); xp = xs + (size_t)(sl * U16 + su) * UNITB; }
                else { int sl = (ui - nown - 3 * U1) / V16; const int jj = (ui - nown - 3 * U1) % V16; if (type == 1 && sl >= cb) ++sl;
                       src = (1 - cls) * 4 + sl; su = U16 + jj; xp = xs + (size_t)(3 * U16 + sl * V16 + jj) * UNITB; }
                d.xa[u] = xp;
                char* ob = type == 0 ? P.out[rank] + cb * 512 : P.out[crank] + idx * 512;
                d.out[u] = valid ? ob + (size_t)(src * MSH + su * 64) * WPITCH : nullptr;
                if (src != rank) { d.anywait = true; if (lane / (64 / (NU + 1)) == u) { d.flag = flags + src * FROW; d.call1 = needv; d.pbase = (src >> 2) == cls ? PBASE_S1 : PBASE_X; } }
            }
            d.throttle = type == 1;
            if (type == 0) { d.wb = P.w + cb * 512; d.wpitch = WPITCH; d.wbytes = KDIM * WPITCH - cb * 512; }
            else { d.wb = xs + XS_BYTES + (size_t)cb * WSQ_BYTES; d.wpitch = QPITCH; d.wbytes = (unsigned)WSQ_BYTES; d.anywait = true;
                   if (lane / (64 / (NU + 1)) == NU) { d.flag = flags + crank * FROW; d.call1 = needv; d.pbase = PBASE_X; } }
            gemm_tile(lds, d, P, call);
            if (type == 1) signal_done(P, crank);
        }
    }
    if (U2 > 0) {
        const int x = b & 7;
        for (;;) {
            __syncthreads();
            if (tid == 0) *cw = __hip_atomic_fetch_add((unsigned*)(myws + WS_QUEUE) + x * 32, 1u, __ATOMIC_RELAXED, __HIP_MEMORY_SCOPE_AGENT) - (call - 1u) * (unsigned)(NT2X + NWG / 8);
            __syncthreads();
            const unsigned tk = __builtin_amdgcn_readfirstlane(*cw);
            if (tk >= (unsigned)NT2X) break;
            const int cbk = 2 * x + (int)(tk & 1u), un = (int)(tk >> 1);
            const int sl = un / (U2 > 0 ? U2 : 1), su = U1 + un % (U2 > 0 ? U2 : 1);
            const int src = cls * 4 + ((idx + 1 + sl) & 3);
            const char* xa = xs + (size_t)(sl * U16 + su) * UNITB;
            const unsigned* xflag = flags + src * FROW + NCH1;
            const size_t orow = (size_t)(src * MSH + su * 64) * WPITCH;
            const bool own = cbk < 8;
            const int cb = (cbk - 8) >> 1, hf = cbk & 1, crank = own ? rank : (1 - cls) * 4 + cb;
            small_tile(lds, xa, own ? P.w + cbk * 256 : xs + XS_BYTES + (size_t)cb * WSQ_BYTES + hf * 256, own ? WPITCH : QPITCH,
                       own ? KDIM * WPITCH - cbk * 256 : (unsigned)WSQ_BYTES - hf * 256, P.out[crank] + orow + (own ? cbk * 256 : idx * 512 + hf * 256),
                       xflag, needv, flags + crank * FROW, own ? 0u : needv, P, call);
            if (!own) signal_done(P, crank);
        }
    }
    if (b == 0) {
        if (tid < 64) {
            unsigned* tmo = (unsigned*)(myws + WS_TMO);
            const bool ok = wave_wait((const unsigned*)(myws + WS_DONE), call * (unsigned)(4 * RBB + 4 * 3 * U2 * 2), tmo, call);
            if (!ok && lane == 0) *(unsigned short*)(P.out[rank] + (size_t)rank * MSH * WPITCH) = 0x7fc0u;
        }
    }
}

}

extern "C" void kernel_launch(int rank, int nranks, ncclComm_t comm, hipStream_t stream,
                              void* const* d_in, const int* in_sizes, int n_in,
                              void* d_out, int out_size, const int* in_dtypes, int out_dtype,
                              void* const* d_out_all, void* const* d_ws_all, size_t ws_bytes) {
    using namespace agk;
    (void)comm; (void)in_sizes; (void)n_in; (void)out_size; (void)in_dtypes; (void)out_dtype; (void)d_out;
    if (nranks != NR || ws_bytes < WS_NEED || rank < 0 || rank >= NR) { fprintf(stderr, "unsupported configuration\n"); return; }
    static bool init[64] = {};
    int dev = 0; HIP_CHECK(hipGetDevice(&dev));
    if (!init[dev & 63]) {
        HIP_CHECK(hipFuncSetAttribute((const void*)ag_kernel, hipFuncAttributeMaxDynamicSharedMemorySize, LDS_BYTES));
        init[dev & 63] = true;
    }
    Params P;
    P.x = (const char*)d_in[0]; P.w = (const char*)d_in[1];
    for (int r = 0; r < NR; ++r) { P.out[r] = (char*)d_out_all[r]; P.ws[r] = (char*)d_ws_all[r]; }
    P.rank = rank;
    hipLaunchKernelGGL(ag_kernel, dim3(NWG), dim3(512), LDS_BYTES, stream, P);
    HIP_CHECK(hipGetLastError());
}
```
